```python
import numpy as np

kernel_name = "gpu_gemm_ar_m4096_k4096_n32768_bf16_relu"


ABSMAX_RTOL = 0.008

COMPUTE_DTYPE = "bfloat16"
EPILOGUE = "relu"
IN_QUANT = None
OUT_QUANT = None
IN_DTYPES = {"x": "bfloat16", "w_mat": "bfloat16"}
WORLD_SIZE = 8

def _to_bf16(x):
    u = np.ascontiguousarray(x, dtype=np.float32).view(np.uint32)
    return ((u + 0x7FFF + ((u >> 16) & 1)) & 0xFFFF0000).view(np.float32)

def setup_inputs():

    x = _to_bf16(np.random.randn(4096, 4096).astype(np.float32))
    w_mat = _to_bf16((np.random.randn(4096, 32768) / np.sqrt(4096)).astype(np.float32))
    return {"x": x, "w_mat": w_mat}

def _epilogue(y):
    return np.maximum(y, 0.0).astype(np.float32)

def reference(x, w_mat):
    y = (x @ w_mat).astype(np.float32)
    return _epilogue(y)
```

```cpp
#include <hip/hip_runtime.h>
#include <rccl/rccl.h>
#include <cstdio>

#define HIP_CHECK(expr) do { hipError_t _e = (expr); if (_e != hipSuccess) { \
    fprintf(stderr, "HIP error %s at %s:%d\n", hipGetErrorString(_e), __FILE__, __LINE__); \
    return; } } while (0)

#define LAS __attribute__((address_space(3)))
#define GAS __attribute__((address_space(1)))
typedef unsigned short u16;
typedef unsigned long long u64;
typedef short bf16x8 __attribute__((ext_vector_type(8)));
typedef float f32x4 __attribute__((ext_vector_type(4)));
typedef unsigned u32x4 __attribute__((ext_vector_type(4)));
typedef unsigned u32x2 __attribute__((ext_vector_type(2)));

constexpr int NR = 8;
constexpr int GM = 4096, GN = 32768, KS = 512, NB = 4096;
constexpr int HTB = 16384;
constexpr int LDS_BYTES = 8 * HTB;
constexpr size_t OFF_CTR = 0;
constexpr size_t OFF_TMO = 32768;
constexpr size_t OFF_STAMP = 65536;
constexpr size_t OFF_INF = 512u << 10;
constexpr size_t OFF_OUTF = 1u << 20;
constexpr size_t OFF_XI = 4u << 20;
constexpr size_t OFF_WI = 36u << 20;
constexpr size_t WS_NEED = 68u << 20;
constexpr u64 DEADLINE = 50000000ull;

struct Args { const char* x; const char* w; char* out[NR]; char* ws[NR]; int rank; };

__device__ __forceinline__ int lds_byte(int r, int c) { const int st = (r >> 4) * 2 + (c >> 5), rr = r & 15, cc = c & 31, ob = rr * 64 + cc * 2; return st * 1024 + (ob ^ (((ob >> 9) & 1) << 5)); }
__device__ __forceinline__ void stage_rc(int b, int& R, int& C) { const int st = b / 1024, sb = b % 1024, swz = sb ^ (((sb >> 9) & 1) << 5); R = (st >> 1) * 16 + swz / 64; C = (st & 1) * 32 + (swz % 64) / 2; }
__device__ __forceinline__ unsigned cvt_pk_bf16(float lo, float hi) { unsigned r; asm volatile("v_cvt_pk_bf16_f32 %0, %1, %2" : "=v"(r) : "v"(lo), "v"(hi)); return r; }
__device__ __forceinline__ unsigned relu_pk(unsigned v) { unsigned r; asm("v_pk_max_i16 %0, %1, 0" : "=v"(r) : "v"(v)); return r; }
__device__ __forceinline__ void st_wt(char* p, u32x4 v) { asm volatile("global_store_dwordx4 %0, %1, off sc0 sc1\n\ts_nop 1" :: "v"(p), "v"(v) : "memory"); }
__device__ __forceinline__ void wait_vm0() { __builtin_amdgcn_s_waitcnt(0x0F70); asm volatile("s_waitcnt vmcnt(0)" ::: "memory"); }
__device__ __forceinline__ u64 now() { return __builtin_amdgcn_s_memrealtime(); }

#define STAMP(i) do { } while (0)

__device__ __forceinline__ int slot_of(int s, int p, int kt) { return s == p ? kt : 8 + 7 * kt + ((s - p - 1) & 7); }

__device__ __forceinline__ void job_load(const Args& a, int dest, int g, int kt, int j, int lane, u32x4 (&v)[16]) {
    const int panel = g * 8 + ((j & 15) >> 1), half = j & 1;
    if (j < 16) {
        const int sb = lane * 16, swz = sb ^ (((sb >> 9) & 1) << 5);
        const char* src = a.x + (size_t)(panel * 256 + half * 128 + (swz >> 6)) * (KS * 2) + kt * 128 + (swz & 63);
#pragma unroll
        for (int i = 0; i < 16; ++i) v[i] = *(const u32x4*)(src + (i >> 1) * 16 * (KS * 2) + (i & 1) * 64);
    } else {
        const char* src = a.w + (size_t)(kt * 64 + (lane & 3) * 8) * (GN * 2) + (size_t)(dest * NB + panel * 256 + half * 128 + (lane >> 2) * 8) * 2;
#pragma unroll
        for (int i = 0; i < 16; ++i) v[i] = *(const u32x4*)(src + (size_t)((i >> 3) * 32 + (i & 7)) * (GN * 2));
    }
}
__device__ __forceinline__ void job_store(const Args& a, int dest, int g, int kt, int j, int lane, const u32x4 (&v)[16]) {
    const int panel = g * 8 + ((j & 15) >> 1), half = j & 1, t = slot_of(a.rank, dest, kt);
    char* dst = a.ws[dest] + (j < 16 ? OFF_XI : OFF_WI) + (size_t)(panel * 64 + t) * 32768 + half * 16384 + lane * 16;
    if (j < 16) {
#pragma unroll
        for (int i = 0; i < 16; ++i) st_wt(dst + i * 1024, v[i]);
    } else {
#pragma unroll
        for (int kh = 0; kh < 2; ++kh)
#pragma unroll
            for (int i = 0; i < 8; ++i) {
                u32x4 o;
#pragma unroll
                for (int d = 0; d < 4; ++d) o[d] = __builtin_amdgcn_perm(v[kh * 8 + 2 * d + 1][i >> 1], v[kh * 8 + 2 * d][i >> 1], (i & 1) ? 0x07060302u : 0x05040100u);
                st_wt(dst + kh * 8192 + i * 1024, o);
            }
    }
}
__device__ __forceinline__ void job_flag(const Args& a, int dest, int g, int kt, int j, int lane, unsigned call) {
    const int panel = g * 8 + ((j & 15) >> 1), half = j & 1, t = slot_of(a.rank, dest, kt);
    unsigned* f = (unsigned*)(a.ws[dest] + OFF_INF) + (((j >> 4) * 16 + panel) * 64 + t) * 2 + half;
    if (lane == 0) __hip_atomic_store(f, call, __ATOMIC_RELAXED, __HIP_MEMORY_SCOPE_SYSTEM);
}

__device__ __forceinline__ bool expired(unsigned* tmo, unsigned call, u64 t0) {
    if (__builtin_amdgcn_readfirstlane(__hip_atomic_load(tmo, __ATOMIC_RELAXED, __HIP_MEMORY_SCOPE_AGENT)) == call || now() - t0 > DEADLINE) {
        __hip_atomic_store(tmo, call, __ATOMIC_RELAXED, __HIP_MEMORY_SCOPE_AGENT);
        return true;
    }
    return false;
}

__global__ __launch_bounds__(512, 2) void gemm_ar_kernel(const Args a) {
    extern __shared__ __attribute__((aligned(16))) unsigned char shm[];
    LAS unsigned char* lds = (LAS unsigned char*)shm;
    const int tid = threadIdx.x, wid = __builtin_amdgcn_readfirstlane(tid >> 6), lane = tid & 63;
    const int wr = wid >> 2, wc = wid & 3, fr = lane & 15, fq = lane >> 4;
    const int b = blockIdx.x, xcd = b & 7, idx = b >> 3, rank = a.rank;
    char* const myws = a.ws[rank];
    const u64 t0 = now();
    STAMP(0);
    unsigned* const ctr = (unsigned*)(myws + OFF_CTR) + b * 32 + wid;
    unsigned* const tmo = (unsigned*)(myws + OFF_TMO);
    const unsigned call = __builtin_amdgcn_readfirstlane(__hip_atomic_load(ctr, __ATOMIC_RELAXED, __HIP_MEMORY_SCOPE_AGENT)) + 1u;
    if (lane == 0) __hip_atomic_store(ctr, call, __ATOMIC_RELAXED, __HIP_MEMORY_SCOPE_AGENT);

    if (idx >= 28) {
        if (wid < 7) {
            const int dest = (rank + 1 + wid) & 7, j = (idx - 28) * 8 + xcd;
            u32x4 v0[16], v1[16];
            job_load(a, dest, 0, 0, j, lane, v0);
            wait_vm0();
            for (int r = 0; r < 16; r += 2) {
                job_store(a, dest, r >> 3, r & 7, j, lane, v0);
                job_load(a, dest, r >> 3, (r & 7) + 1, j, lane, v1);
                wait_vm0();
                job_flag(a, dest, r >> 3, r & 7, j, lane, call);
                job_store(a, dest, r >> 3, (r & 7) + 1, j, lane, v1);
                if (r + 2 < 16) job_load(a, dest, (r + 2) >> 3, (r + 2) & 7, j, lane, v0);
                wait_vm0();
                job_flag(a, dest, r >> 3, (r & 7) + 1, j, lane, call);
            }
        }
        STAMP(1);
    } else if (idx >= 8 && idx < 16) {
        const int q = (idx - 8) * 8 + xcd, j = q & 31, g = q >> 5;
        u32x4 v0[16];
        job_load(a, rank, g, wid, j, lane, v0);
        wait_vm0();
        job_store(a, rank, g, wid, j, lane, v0);
        wait_vm0();
        job_flag(a, rank, g, wid, j, lane, call);
        STAMP(1);
    }

    const int quad = idx >> 3;
    const int pm = (quad >> 1) * 8 + (xcd & 1) * 4 + (idx & 3), pn = (quad & 1) * 8 + (xcd >> 1) * 2 + ((idx >> 2) & 1);
    const char* const fX = myws + OFF_INF + (size_t)pm * 512 + lane * 8;
    const char* const fW = myws + OFF_INF + (size_t)(16 + pn) * 512 + lane * 8;
    bool bad = false;
    auto poll = [&](int need) -> int {
        for (;;) {
            u32x2 fa, fb;
            asm volatile("global_load_dwordx2 %0, %2, off sc0 sc1\n\tglobal_load_dwordx2 %1, %3, off sc0 sc1\n\ts_waitcnt vmcnt(0)"
                         : "=&v"(fa), "=&v"(fb) : "v"(fX), "v"(fW) : "memory");
            const u64 miss = __builtin_amdgcn_ballot_w64(fa.x < call || fa.y < call || fb.x < call || fb.y < call);
            const int upto = miss ? __builtin_ctzll(miss) : 64;
            if (upto >= need) return upto;
            if (expired(tmo, call, t0)) { bad = true; return 64; }
            __builtin_amdgcn_s_sleep(4);
        }
    };
    unsigned voffA[2], voffB[2];
#pragma unroll
    for (int i = 0; i < 2; ++i) {
        int R, C; stage_rc(tid * 16 + i * 8192, R, C);
        voffA[i] = tid * 16 + i * 8192;
        const int w_ = R >> 5, n_ = (R >> 4) & 1, q_ = (R & 15) >> 2, j_ = R & 3;
        voffB[i] = (w_ >> 1) * 16384 + (C >> 5) * 8192 + (4 * n_ + j_) * 1024 + ((w_ & 1) * 8 + q_) * 64 + ((C >> 3) & 3) * 16;
    }
    constexpr size_t kstep = 32768, hstepA = 16384, hstepB = 256;
    const unsigned ldsw = (unsigned)wid * 1024u;
    const int aoff = lds_byte(wr * 64 + fr, fq * 8), boff = lds_byte(wc * 32 + fr, fq * 8);
#define G_SA(b, h) (((b) * 2 + (h)) * HTB)
#define G_SB(b, h) ((4 + (b) * 2 + (h)) * HTB)
#define G_STAGE(bufoff, gbase, voff) do { _Pragma("unroll") for (int _i = 0; _i < 2; ++_i) \
        __builtin_amdgcn_global_load_lds((const unsigned*)((const char*)(gbase) + (voff)[_i]), (LAS unsigned*)(lds + (bufoff) + ldsw + _i * 8192), 16, 0, 17); } while (0)
#define G_LDA(dst, b, h) do { _Pragma("unroll") for (int m = 0; m < 4; ++m) _Pragma("unroll") for (int k = 0; k < 2; ++k) dst[m][k] = *(const LAS bf16x8*)(lds + G_SA(b, h) + aoff + m * 2048 + k * 1024); } while (0)
#define G_LDB(dst, b, h) do { _Pragma("unroll") for (int n = 0; n < 2; ++n) _Pragma("unroll") for (int k = 0; k < 2; ++k) dst[n][k] = *(const LAS bf16x8*)(lds + G_SB(b, h) + boff + n * 2048 + k * 1024); } while (0)
#define G_MMA(ai, bj, At, Bt) do { __builtin_amdgcn_s_setprio(1); _Pragma("unroll") for (int m = 0; m < 4; ++m) _Pragma("unroll") for (int n = 0; n < 2; ++n) _Pragma("unroll") for (int k = 0; k < 2; ++k) \
        acc[ai][bj][m][n] = __builtin_amdgcn_mfma_f32_16x16x32_bf16(Bt[n][k], At[m][k], acc[ai][bj][m][n], 0, 0, 0); __builtin_amdgcn_s_setprio(0); } while (0)
#define G_WAIT_V(n) asm volatile("s_waitcnt vmcnt(" #n ")" ::: "memory")
#define G_WAIT_L(n) asm volatile("s_waitcnt lgkmcnt(" #n ")" ::: "memory")
#define G_BAR __builtin_amdgcn_s_barrier()
#define G_SCHED __builtin_amdgcn_sched_barrier(0)
    f32x4 acc[2][2][4][2];
#pragma unroll
    for (int x = 0; x < 2; ++x)
#pragma unroll
        for (int y = 0; y < 2; ++y)
#pragma unroll
            for (int m = 0; m < 4; ++m)
#pragma unroll
                for (int n = 0; n < 2; ++n) acc[x][y][m][n] = (f32x4){0.f, 0.f, 0.f, 0.f};
    bf16x8 At[4][2], B0[2][2], B1[2][2];
    const char* const cA = myws + OFF_XI + (size_t)pm * 64 * kstep;
    const char* const cB = myws + OFF_WI + (size_t)pn * 64 * kstep;
    int ready = poll(2);
    STAMP(2);
    wait_vm0();
    G_STAGE(G_SB(0, 0), cB, voffB); G_STAGE(G_SA(0, 0), cA, voffA); G_STAGE(G_SB(0, 1), cB + hstepB, voffB); G_STAGE(G_SA(0, 1), cA + hstepA, voffA);
    if (wr == 1) G_BAR;
    G_WAIT_V(4); G_BAR;
    G_STAGE(G_SB(1, 0), cB + kstep, voffB); G_STAGE(G_SA(1, 0), cA + kstep, voffA); G_STAGE(G_SB(1, 1), cB + hstepB + kstep, voffB);
    G_WAIT_V(6); G_BAR;
    for (int t = 0; t < 64; t += 2) {
        const bool last = (t == 62);
        if (!last && ready < t + 4) ready = poll(t + 4);
        const char* a1 = cA + (size_t)(t + 1) * kstep;
        const char* a2 = last ? cA : cA + (size_t)(t + 2) * kstep; const char* b2 = last ? cB : cB + (size_t)(t + 2) * kstep;
        const char* a3 = a2 + kstep; const char* b3 = b2 + kstep;
        G_LDB(B0, 0, 0); G_SCHED; G_LDA(At, 0, 0); G_STAGE(G_SA(1, 1), a1 + hstepA, voffA);
        G_WAIT_L(8); G_BAR; G_WAIT_L(0); G_MMA(0, 0, At, B0); G_BAR; G_SCHED;
        G_LDB(B1, 0, 1); G_STAGE(G_SB(0, 0), b2, voffB);
        G_BAR; G_WAIT_L(0); G_MMA(0, 1, At, B1); G_BAR;
        G_LDA(At, 0, 1); G_STAGE(G_SA(0, 0), a2, voffA);
        G_BAR; G_WAIT_L(0); G_MMA(1, 0, At, B0); G_BAR; G_SCHED;
        G_STAGE(G_SB(0, 1), b2 + hstepB, voffB);
        G_WAIT_V(6); G_BAR; G_MMA(1, 1, At, B1); G_BAR;
        G_LDB(B0, 1, 0); G_SCHED; G_LDA(At, 1, 0); G_STAGE(G_SA(0, 1), a2 + hstepA, voffA);
        G_WAIT_L(8); G_BAR; G_WAIT_L(0); G_MMA(0, 0, At, B0); G_BAR; G_SCHED;
        G_LDB(B1, 1, 1); G_STAGE(G_SB(1, 0), b3, voffB);
        G_BAR; G_WAIT_L(0); G_MMA(0, 1, At, B1); G_BAR;
        G_LDA(At, 1, 1); G_STAGE(G_SA(1, 0), a3, voffA);
        G_BAR; G_WAIT_L(0); G_MMA(1, 0, At, B0); G_BAR; G_SCHED;
        G_STAGE(G_SB(1, 1), b3 + hstepB, voffB);
        G_WAIT_V(6); G_BAR; G_MMA(1, 1, At, B1); G_BAR;
    }
    G_WAIT_V(0);
    if (wr == 0) G_BAR;
    G_BAR;
    STAMP(3);

    if (__builtin_amdgcn_readfirstlane(__hip_atomic_load(tmo, __ATOMIC_RELAXED, __HIP_MEMORY_SCOPE_AGENT)) == call) bad = true;
    u32x4 o[2][2][4];
#pragma unroll
    for (int ai = 0; ai < 2; ++ai)
#pragma unroll
        for (int bj = 0; bj < 2; ++bj)
#pragma unroll
            for (int m = 0; m < 4; ++m) {
                const f32x4 v0 = acc[ai][bj][m][0], v1 = acc[ai][bj][m][1];
                u32x4 w; w[0] = relu_pk(cvt_pk_bf16(v0[0], v0[1])); w[1] = relu_pk(cvt_pk_bf16(v0[2], v0[3]));
                w[2] = relu_pk(cvt_pk_bf16(v1[0], v1[1])); w[3] = relu_pk(cvt_pk_bf16(v1[2], v1[3]));
                if (bad) w = (u32x4){0x7fc07fc0u, 0x7fc07fc0u, 0x7fc07fc0u, 0x7fc07fc0u};
                o[ai][bj][m] = w;
            }
    const size_t obase = (size_t)(pm * 256 + wr * 64 + fr) * (GN * 2) + (size_t)(rank * NB + pn * 256 + wc * 64 + fq * 8) * 2;
    for (int i = 0; i < 8; ++i) {
        const int dest = (rank + 1 + ((i + b) & 7)) & 7;
        char* p = a.out[dest] + obase;
#pragma unroll
        for (int ai = 0; ai < 2; ++ai)
#pragma unroll
            for (int m = 0; m < 4; ++m)
#pragma unroll
                for (int bj = 0; bj < 2; ++bj) st_wt(p + (size_t)(ai * 128 + m * 16) * (GN * 2) + bj * 64, o[ai][bj][m]);
    }
    wait_vm0();
    STAMP(4);
    const int unit = b * 8 + wid;
    if (lane < 7) {
        const int dest = (rank + 1 + lane) & 7;
        char* wsd = a.ws[0];
#pragma unroll
        for (int p = 1; p < 8; ++p) wsd = dest == p ? a.ws[p] : wsd;
        __hip_atomic_store((u64*)(wsd + OFF_OUTF) + rank * 2048 + unit, (u64)call << 32, __ATOMIC_RELAXED, __HIP_MEMORY_SCOPE_SYSTEM);
    }
    {
        const int src = (rank + 1 + (lane < 7 ? lane : 0)) & 7;
        const u64* f = (const u64*)(myws + OFF_OUTF) + src * 2048 + unit;
        for (;;) {
            const u64 v = __hip_atomic_load(f, __ATOMIC_RELAXED, __HIP_MEMORY_SCOPE_SYSTEM);
            if (__builtin_amdgcn_ballot_w64((unsigned)(v >> 32) < call) == 0) break;
            if (expired(tmo, call, t0)) { if (lane == 0) *(u16*)(a.out[rank] + obase) = 0x7fc0; break; }
            __builtin_amdgcn_s_sleep(8);
        }
    }
    STAMP(5);
}

extern "C" void kernel_launch(int rank, int nranks, ncclComm_t comm,
                              hipStream_t stream,
                              void* const* d_in, const int* in_sizes, int n_in,
                              void* d_out, int out_size,
                              const int* in_dtypes, int out_dtype,
                              void* const* d_out_all,
                              void* const* d_ws_all, size_t ws_bytes) {
    if (nranks != NR || (unsigned)rank >= (unsigned)NR || ws_bytes < WS_NEED || n_in < 2) { fprintf(stderr, "unsupported configuration\n"); return; }
    static bool attr[NR] = {};
    if (!attr[rank]) { HIP_CHECK(hipFuncSetAttribute((const void*)gemm_ar_kernel, hipFuncAttributeMaxDynamicSharedMemorySize, LDS_BYTES)); attr[rank] = true; }
    Args a;
    a.x = (const char*)d_in[0]; a.w = (const char*)d_in[1]; a.rank = rank;
    for (int p = 0; p < NR; ++p) { a.out[p] = (char*)d_out_all[p]; a.ws[p] = (char*)d_ws_all[p]; }
    hipLaunchKernelGGL(gemm_ar_kernel, dim3(256), dim3(512), LDS_BYTES, stream, a);
    HIP_CHECK(hipGetLastError());
}
```

```cpp
#include <hip/hip_runtime.h>
#include <rccl/rccl.h>
#include <cstdio>

#define HIP_CHECK(expr) do { hipError_t _e = (expr); if (_e != hipSuccess) { \
    fprintf(stderr, "HIP error %s at %s:%d\n", hipGetErrorString(_e), __FILE__, __LINE__); \
    return; } } while (0)

#define LAS __attribute__((address_space(3)))
#define GAS __attribute__((address_space(1)))
typedef unsigned short u16;
typedef unsigned long long u64;
typedef short bf16x8 __attribute__((ext_vector_type(8)));
typedef float f32x4 __attribute__((ext_vector_type(4)));
typedef unsigned u32x4 __attribute__((ext_vector_type(4)));
typedef unsigned u32x2 __attribute__((ext_vector_type(2)));

constexpr int NR = 8;
constexpr int GM = 4096, GN = 32768, KS = 512, NB = 4096;
constexpr int HTB = 16384;
constexpr int C_ARR = 1024, C_EXC = C_ARR + 16384 + 128, C_WAVE = C_EXC + 1024, EXC_CAP = 512;
constexpr int LDS_SEM = 8 * C_WAVE;
constexpr int LDS_BYTES = LDS_SEM + 64;
constexpr size_t OFF_CTR = 0;
constexpr size_t OFF_TMO = 32768;
constexpr size_t OFF_TICK = 32768 + 256;
constexpr size_t OFF_STAMP = 65536;
constexpr size_t OFF_INF = 512u << 10;
constexpr size_t OFF_OUTF = 1u << 20;
constexpr size_t OFF_XI = 4u << 20;
constexpr size_t OFF_WI = 36u << 20;
constexpr size_t OFF_OC = 68u << 20;
constexpr size_t WS_NEED = OFF_OC + (256u << 20);
constexpr u64 DEADLINE = 50000000ull;

#define NPUSH 32
#define LIM1 4
#define LIM2 2
#define GATE 480
#define WIRE_ALIGN 64
struct Args { const char* x; const char* w; char* out[NR]; char* ws[NR]; int rank; };

__device__ __forceinline__ int lds_byte(int r, int c) { const int st = (r >> 4) * 2 + (c >> 5), rr = r & 15, cc = c & 31, ob = rr * 64 + cc * 2; return st * 1024 + (ob ^ (((ob >> 9) & 1) << 5)); }
__device__ __forceinline__ void stage_rc(int b, int& R, int& C) { const int st = b / 1024, sb = b % 1024, swz = sb ^ (((sb >> 9) & 1) << 5); R = (st >> 1) * 16 + swz / 64; C = (st & 1) * 32 + (swz % 64) / 2; }
__device__ __forceinline__ unsigned cvt_pk_bf16(float lo, float hi) { unsigned r; asm volatile("v_cvt_pk_bf16_f32 %0, %1, %2" : "=v"(r) : "v"(lo), "v"(hi)); return r; }
__device__ __forceinline__ unsigned relu_pk(unsigned v) { unsigned r; asm("v_pk_max_i16 %0, %1, 0" : "=v"(r) : "v"(v)); return r; }
__device__ __forceinline__ void st_wt(char* p, u32x4 v) { asm volatile("global_store_dwordx4 %0, %1, off sc0 sc1\n\ts_nop 1" :: "v"(p), "v"(v) : "memory"); }
__device__ __forceinline__ void wait_vm0() { __builtin_amdgcn_s_waitcnt(0x0F70); asm volatile("s_waitcnt vmcnt(0)" ::: "memory"); }
__device__ __forceinline__ void bar() { asm volatile("" ::: "memory"); __builtin_amdgcn_s_barrier(); asm volatile("" ::: "memory"); }
__device__ __forceinline__ u64 now() { return __builtin_amdgcn_s_memrealtime(); }

#define STAMP(i) do { } while (0)
#define STAMP_FLUSH() do { } while (0)

__device__ __forceinline__ int slot_of(int s, int p, int kt) { return s == p ? kt : 8 + 7 * kt + ((s - p - 1) & 7); }

__device__ __forceinline__ void job_load(const Args& a, int dest, int g, int kt, int j, int lane, u32x4 (&v)[16]) {
    const int panel = g * 8 + ((j & 15) >> 1), half = j & 1;
    if (j < 16) {
        const int sb = lane * 16, swz = sb ^ (((sb >> 9) & 1) << 5);
        const char* src = a.x + (size_t)(panel * 256 + half * 128 + (swz >> 6)) * (KS * 2) + kt * 128 + (swz & 63);
#pragma unroll
        for (int i = 0; i < 16; ++i) v[i] = *(const u32x4*)(src + (i >> 1) * 16 * (KS * 2) + (i & 1) * 64);
    } else {
        const char* src = a.w + (size_t)(kt * 64 + (lane & 3) * 8) * (GN * 2) + (size_t)(dest * NB + panel * 256 + half * 128 + (lane >> 2) * 8) * 2;
#pragma unroll
        for (int i = 0; i < 16; ++i) v[i] = *(const u32x4*)(src + (size_t)((i >> 3) * 32 + (i & 7)) * (GN * 2));
    }
}
__device__ __forceinline__ void job_store(const Args& a, int dest, int g, int kt, int j, int lane, const u32x4 (&v)[16]) {
    const int panel = g * 8 + ((j & 15) >> 1), half = j & 1, t = slot_of(a.rank, dest, kt);
    char* dst = a.ws[dest] + (j < 16 ? OFF_XI : OFF_WI) + (size_t)(panel * 64 + t) * 32768 + half * 16384 + lane * 16;
    if (j < 16) {
#pragma unroll
        for (int i = 0; i < 16; ++i) st_wt(dst + i * 1024, v[i]);
    } else {
#pragma unroll
        for (int kh = 0; kh < 2; ++kh)
#pragma unroll
            for (int i = 0; i < 8; ++i) {
                u32x4 o;
#pragma unroll
                for (int d = 0; d < 4; ++d) o[d] = __builtin_amdgcn_perm(v[kh * 8 + 2 * d + 1][i >> 1], v[kh * 8 + 2 * d][i >> 1], (i & 1) ? 0x07060302u : 0x05040100u);
                st_wt(dst + kh * 8192 + i * 1024, o);
            }
    }
}
__device__ __forceinline__ void job_flag(const Args& a, int dest, int g, int kt, int j, int lane, unsigned call) {
    const int panel = g * 8 + ((j & 15) >> 1), half = j & 1, t = slot_of(a.rank, dest, kt);
    unsigned* f = (unsigned*)(a.ws[dest] + OFF_INF) + (((j >> 4) * 16 + panel) * 64 + t) * 2 + half;
    if (lane == 0) __hip_atomic_store(f, call, __ATOMIC_RELAXED, __HIP_MEMORY_SCOPE_SYSTEM);
}

#define OUT_OFF(i) ((size_t)(((i) >> 3) * 128 + (((i) >> 1) & 3) * 16) * (GN * 2) + ((i) & 1) * 64)
constexpr unsigned INFO_RAW = 1u << 31;
__device__ __forceinline__ int hi_off(int nch) { return 1024 + nch * 16; }
__device__ __forceinline__ int exc_off(int nch) { return 1024 + nch * 16 + ((nch + 3) & ~3) * 4; }
__device__ __forceinline__ int wire_bytes(unsigned info) { return (exc_off(info & 1023) + (int)((info >> 10) & 1023) * 2 + (WIRE_ALIGN - 1)) & ~(WIRE_ALIGN - 1); }
__device__ __forceinline__ unsigned pk_max_u16(unsigned x, unsigned y) { unsigned r; asm("v_pk_max_u16 %0, %1, %2" : "=v"(r) : "v"(x), "v"(y)); return r; }
__device__ __forceinline__ unsigned pk_sub_u16(unsigned x, unsigned y) { unsigned r; asm("v_pk_sub_u16 %0, %1, %2" : "=v"(r) : "v"(x), "v"(y)); return r; }
__device__ __forceinline__ unsigned pk_add_u16(unsigned x, unsigned y) { unsigned r; asm("v_pk_add_u16 %0, %1, %2" : "=v"(r) : "v"(x), "v"(y)); return r; }
__device__ __forceinline__ unsigned lanes_below(u64 bal) { return __builtin_amdgcn_mbcnt_hi((unsigned)(bal >> 32), __builtin_amdgcn_mbcnt_lo((unsigned)bal, 0u)); }
__device__ __forceinline__ void wait_lds() { asm volatile("s_waitcnt lgkmcnt(0)" ::: "memory"); }

__device__ __forceinline__ unsigned encode_unit(u32x4 (&W)[16], LAS unsigned char* cw, int lane) {
    unsigned mx = 0;
#pragma unroll
    for (int i = 0; i < 16; ++i) mx = pk_max_u16(mx, pk_max_u16(pk_max_u16(W[i][0], W[i][1]), pk_max_u16(W[i][2], W[i][3])));
    mx = max(mx & 0xffffu, mx >> 16);
#pragma unroll
    for (int o = 32; o >= 1; o >>= 1) mx = max(mx, (unsigned)__shfl_xor((int)mx, o));
    const unsigned M = __builtin_amdgcn_readfirstlane(mx), base = M > 1024u ? M - 1023u : 1u, base2 = base * 0x10001u;
    u32x4 mk = {0u, 0u, 0u, 0u};
    unsigned cnt = 0;
#pragma unroll
    for (int i = 0; i < 64; ++i) {
        const unsigned w = W[i >> 2][i & 3];
#pragma unroll
        for (int h = 0; h < 2; ++h) {
            const bool nz = h ? w > 0xffffu : (w & 0xffffu) != 0u;
            const u64 bal = __builtin_amdgcn_ballot_w64(nz);
            const unsigned pos = lanes_below(bal);
            if (nz) *(LAS u16*)(cw + C_ARR + cnt + 2 * pos) = (u16)(h ? w >> 16 : w);
            mk[i >> 4] = 2u * mk[i >> 4] + (nz ? 1u : 0u);
            cnt += 2u * (unsigned)__builtin_popcountll(bal);
        }
    }
    if (lane < 16) *(LAS u16*)(cw + C_ARR + cnt + 2 * lane) = (u16)base;
    *(LAS u32x4*)(cw + lane * 16) = mk;
    if (lane == 0) *(LAS u16*)(cw + C_EXC) = (u16)base;
    const int nch = (int)(cnt + 31u) >> 5;
    unsigned nexc = 1; bool over = false;
    unsigned hi[8];
#pragma unroll
    for (int it = 0; it < 8; ++it) {
        hi[it] = 0;
        if (it * 64 < nch) {
            const int c = it * 64 + lane;
            asm volatile("" ::: "memory");
            const u32x4 x = *(const LAS u32x4*)(cw + C_ARR + c * 32), y = *(const LAS u32x4*)(cw + C_ARR + c * 32 + 16);
            unsigned d[8];
#pragma unroll
            for (int q = 0; q < 4; ++q) { d[q] = pk_sub_u16(x[q], base2); d[4 + q] = pk_sub_u16(y[q], base2); }
            u32x4 lo; unsigned H[4];
#pragma unroll
            for (int q = 0; q < 4; ++q) { lo[q] = __builtin_amdgcn_perm(d[2 * q + 1], d[2 * q], 0x06040200u); H[q] = __builtin_amdgcn_perm(d[2 * q + 1], d[2 * q], 0x07050301u); }
            hi[it] = (H[0] & 0x03030303u) | ((H[1] & 0x03030303u) << 2) | ((H[2] & 0x03030303u) << 4) | ((H[3] & 0x03030303u) << 6);
            unsigned e = ((H[0] >> 7) & 0x01010101u) | ((H[1] >> 6) & 0x02020202u) | ((H[2] >> 5) & 0x04040404u) | ((H[3] >> 4) & 0x08080808u);
            if (c >= nch) e = 0;
            for (;;) {
                const u64 bal = __builtin_amdgcn_ballot_w64(e != 0u);
                if (!bal) break;
                const unsigned slot = nexc + lanes_below(bal);
                if (e) {
                    const int p = __builtin_ctz(e), q = p & 7, j = p >> 3;
                    e &= e - 1u;
                    const unsigned Hq = q == 0 ? H[0] : q == 1 ? H[1] : q == 2 ? H[2] : H[3];
                    const unsigned k = 64u - (((Hq >> (8 * j)) & 0xffu) >> 2);
                    if (k > 7u) over = true;
                    if (slot < EXC_CAP) *(LAS u16*)(cw + C_EXC + 2 * slot) = (u16)((unsigned)(c * 16 + q * 4 + j) | (k << 13));
                }
                nexc += (unsigned)__builtin_popcountll(bal);
            }
            asm volatile("" ::: "memory");
            *(LAS u32x4*)(cw + C_ARR + c * 16) = lo;
        }
    }
    if (nexc > EXC_CAP || __builtin_amdgcn_ballot_w64(over)) return INFO_RAW;
    asm volatile("" ::: "memory");
#pragma unroll
    for (int it = 0; it < 8; ++it) if (it * 64 < nch) { if (it * 64 + lane < nch) *(LAS unsigned*)(cw + hi_off(nch) + (it * 64 + lane) * 4) = hi[it]; }
    const u32x4 ex = *(const LAS u32x4*)(cw + C_EXC + lane * 16);
    *(LAS u32x4*)(cw + exc_off(nch) + lane * 16) = ex;
    asm volatile("" ::: "memory");
#pragma unroll
    for (int i = 0; i < 12; ++i) W[i] = *(const LAS u32x4*)(cw + i * 1024 + lane * 16);
    wait_lds();
    return (unsigned)nch | (nexc << 10);
}

__device__ __forceinline__ void decode_unit(const char* slot, unsigned info, char* outp, LAS unsigned char* cw, int lane) {
    if (info & INFO_RAW) {
#pragma unroll
        for (int g = 0; g < 4; ++g) {
            u32x4 r0, r1, r2, r3;
            asm volatile("global_load_dwordx4 %0, %4, off sc0 sc1\n\tglobal_load_dwordx4 %1, %4, off offset:1024 sc0 sc1\n\t"
                         "global_load_dwordx4 %2, %4, off offset:2048 sc0 sc1\n\tglobal_load_dwordx4 %3, %4, off offset:3072 sc0 sc1\n\ts_waitcnt vmcnt(0)"
                         : "=&v"(r0), "=&v"(r1), "=&v"(r2), "=&v"(r3) : "v"(slot + g * 4096 + lane * 16) : "memory");
            *(GAS u32x4*)(outp + OUT_OFF(g * 4)) = r0; *(GAS u32x4*)(outp + OUT_OFF(g * 4 + 1)) = r1;
            *(GAS u32x4*)(outp + OUT_OFF(g * 4 + 2)) = r2; *(GAS u32x4*)(outp + OUT_OFF(g * 4 + 3)) = r3;
        }
        return;
    }
    const int nch = (int)min(info & 1023u, 512u), nexc = (int)min((info >> 10) & 1023u, (unsigned)EXC_CAP);
    const int last = max(nch - 1, 0);
    u32x4 mk; unsigned e0, e1; unsigned base2 = 0;
    for (int p = 0; p == 0 || p < nch; p += 320) {
        u32x4 L[5]; unsigned h[5]; unsigned ol[5], oh[5];
#pragma unroll
        for (int i = 0; i < 5; ++i) { const int c = min(p + i * 64 + lane, last); ol[i] = 1024 + c * 16; oh[i] = hi_off(nch) + c * 4; }
        const unsigned om = lane * 16, oe = exc_off(nch) + lane * 2;
        asm volatile("global_load_dwordx4 %0, %13, %25 sc0 sc1\n\tglobal_load_ushort %1, %14, %25 sc0 sc1\n\tglobal_load_ushort %2, %14, %25 offset:128 sc0 sc1\n\t"
                     "global_load_dwordx4 %3, %15, %25 sc0 sc1\n\tglobal_load_dwordx4 %4, %16, %25 sc0 sc1\n\tglobal_load_dwordx4 %5, %17, %25 sc0 sc1\n\t"
                     "global_load_dwordx4 %6, %18, %25 sc0 sc1\n\tglobal_load_dwordx4 %7, %19, %25 sc0 sc1\n\t"
                     "global_load_dword %8, %20, %25 sc0 sc1\n\tglobal_load_dword %9, %21, %25 sc0 sc1\n\tglobal_load_dword %10, %22, %25 sc0 sc1\n\t"
                     "global_load_dword %11, %23, %25 sc0 sc1\n\tglobal_load_dword %12, %24, %25 sc0 sc1\n\ts_waitcnt vmcnt(0)"
                     : "=&v"(mk), "=&v"(e0), "=&v"(e1), "=&v"(L[0]), "=&v"(L[1]), "=&v"(L[2]), "=&v"(L[3]), "=&v"(L[4]),
                       "=&v"(h[0]), "=&v"(h[1]), "=&v"(h[2]), "=&v"(h[3]), "=&v"(h[4])
                     : "v"(om), "v"(oe), "v"(ol[0]), "v"(ol[1]), "v"(ol[2]), "v"(ol[3]), "v"(ol[4]),
                       "v"(oh[0]), "v"(oh[1]), "v"(oh[2]), "v"(oh[3]), "v"(oh[4]), "s"(slot) : "memory");
        base2 = __builtin_amdgcn_readfirstlane(e0) * 0x10001u;
#pragma unroll
        for (int i = 0; i < 5; ++i) if (p + i * 64 < nch) {
            u32x4 x, y;
#pragma unroll
            for (int q = 0; q < 4; ++q) {
                const unsigned Hq = (h[i] >> (2 * q)) & 0x03030303u;
                const unsigned v0 = pk_add_u16(__builtin_amdgcn_perm(Hq, L[i][q], 0x05010400u), base2), v1 = pk_add_u16(__builtin_amdgcn_perm(Hq, L[i][q], 0x07030602u), base2);
                if (q < 2) { x[2 * q] = v0; x[2 * q + 1] = v1; } else { y[2 * q - 4] = v0; y[2 * q - 3] = v1; }
            }
            *(LAS u32x4*)(cw + C_ARR + (p + i * 64 + lane) * 32) = x; *(LAS u32x4*)(cw + C_ARR + (p + i * 64 + lane) * 32 + 16) = y;
        }
    }
    auto patch = [&](unsigned e) { __hip_atomic_fetch_sub((LAS unsigned*)(cw + C_ARR + ((e & 0x1ffeu) << 1)), (e >> 13) << (10 + 16 * (e & 1u)), __ATOMIC_RELAXED, __HIP_MEMORY_SCOPE_WORKGROUP); };
    if (lane > 0 && lane < nexc) patch(e0);
    if (lane + 64 < nexc) patch(e1);
    for (int r = 128; r < nexc; r += 64) {
        unsigned e;
        asm volatile("global_load_ushort %0, %1, %2 sc0 sc1\n\ts_waitcnt vmcnt(0)" : "=&v"(e) : "v"((unsigned)(exc_off(nch) + (r + lane) * 2)), "s"(slot) : "memory");
        if (r + lane < nexc) patch(e);
    }
    asm volatile("" ::: "memory");
    unsigned cnt = 0;
#pragma unroll
    for (int i = 0; i < 16; ++i) {
        u32x4 o;
#pragma unroll
        for (int d = 0; d < 4; ++d) {
            unsigned v[2];
#pragma unroll
            for (int hh = 0; hh < 2; ++hh) {
                unsigned m2; const bool nz = __builtin_add_overflow(mk[i >> 2], mk[i >> 2], &m2); mk[i >> 2] = m2;
                const u64 bal = __builtin_amdgcn_ballot_w64(nz);
                const unsigned x = *(const LAS u16*)(cw + C_ARR + cnt + 2 * lanes_below(bal));
                v[hh] = nz ? x : 0u;
                cnt += 2u * (unsigned)__builtin_popcountll(bal);
            }
            o[d] = v[0] | (v[1] << 16);
        }
        *(GAS u32x4*)(outp + OUT_OFF(i)) = o;
    }
    asm volatile("" ::: "memory");
}

__device__ __forceinline__ bool expired(unsigned* tmo, unsigned call, u64 t0) {
    if (__builtin_amdgcn_readfirstlane(__hip_atomic_load(tmo, __ATOMIC_RELAXED, __HIP_MEMORY_SCOPE_AGENT)) == call || now() - t0 > DEADLINE) {
        __hip_atomic_store(tmo, call, __ATOMIC_RELAXED, __HIP_MEMORY_SCOPE_AGENT);
        return true;
    }
    return false;
}

__global__ __launch_bounds__(512, 2) void gemm_ar_kernel(const Args a) {
    extern __shared__ __attribute__((aligned(16))) unsigned char shm[];
    LAS unsigned char* lds = (LAS unsigned char*)shm;
    const int tid = threadIdx.x, wid = __builtin_amdgcn_readfirstlane(tid >> 6), lane = tid & 63;
    const int wr = wid >> 2, wc = wid & 3, fr = lane & 15, fq = lane >> 4;
    const int b = blockIdx.x, xcd = b & 7, idx = b >> 3, rank = a.rank;
    char* const myws = a.ws[rank];
    const u64 t0 = now();
    STAMP(0);
    unsigned* const ctr = (unsigned*)(myws + OFF_CTR) + b * 32 + wid;
    unsigned* const tmo = (unsigned*)(myws + OFF_TMO);
    const unsigned call = __builtin_amdgcn_readfirstlane(__hip_atomic_load(ctr, __ATOMIC_RELAXED, __HIP_MEMORY_SCOPE_AGENT)) + 1u;
    if (lane == 0) __hip_atomic_store(ctr, call, __ATOMIC_RELAXED, __HIP_MEMORY_SCOPE_AGENT);

    if (tid == 0) *(LAS unsigned*)(lds + LDS_SEM) = 0u;
    if (idx >= 32 - NPUSH / 8) {
        const int pw = (idx - (32 - NPUSH / 8)) * 8 + xcd;
        LAS int* tk = (LAS int*)lds;
        int q0 = pw, q1 = NPUSH + pw, n = 0;
        if (wid < 7) {
            u32x4 v0[16], v1[16];
#define DEST(q) ((rank + 1 + (wid + (q)) % 7) & 7)
            job_load(a, DEST(q0), q0 >> 8, (q0 >> 5) & 7, q0 & 31, lane, v0);
            wait_vm0();
            for (;;) {
                job_store(a, DEST(q0), q0 >> 8, (q0 >> 5) & 7, q0 & 31, lane, v0);
                if (q1 < 512) job_load(a, DEST(q1), q1 >> 8, (q1 >> 5) & 7, q1 & 31, lane, v1);
                wait_vm0();
                job_flag(a, DEST(q0), q0 >> 8, (q0 >> 5) & 7, q0 & 31, lane, call);
                STAMP(8 + (q0 >> 6));
                bar();
                if (q1 >= 512) break;
                q0 = __builtin_amdgcn_readfirstlane(tk[n & 1]); ++n;
                job_store(a, DEST(q1), q1 >> 8, (q1 >> 5) & 7, q1 & 31, lane, v1);
                if (q0 < 512) job_load(a, DEST(q0), q0 >> 8, (q0 >> 5) & 7, q0 & 31, lane, v0);
                wait_vm0();
                job_flag(a, DEST(q1), q1 >> 8, (q1 >> 5) & 7, q1 & 31, lane, call);
                STAMP(8 + (q1 >> 6));
                bar();
                if (q0 >= 512) break;
                q1 = __builtin_amdgcn_readfirstlane(tk[n & 1]); ++n;
            }
        } else {
            unsigned* tq = (unsigned*)(myws + OFF_TICK);
            const unsigned base = (call - 1u) * (512u - NPUSH) - 2u * NPUSH;
            for (;;) {
                int q2 = 512;
                if (q1 < 512) {
                    unsigned d = 0;
                    if (lane == 0) d = __hip_atomic_fetch_add(tq, 1u, __ATOMIC_RELAXED, __HIP_MEMORY_SCOPE_AGENT);
                    d = __builtin_amdgcn_readfirstlane(d) - base;
                    q2 = d < 512u ? (int)d : 512;
                    tk[n & 1] = q2;
                    asm volatile("s_waitcnt lgkmcnt(0)" ::: "memory");
                }
                bar();
                if (q1 >= 512) break;
                q1 = q2; ++n;
            }
        }
        bar();
        STAMP(1);
    } else if (idx >= 8 && idx < 16) {
        const int q = (idx - 8) * 8 + xcd, j = q & 31, g = q >> 5;
        u32x4 v0[16];
        job_load(a, rank, g, wid, j, lane, v0);
        wait_vm0();
        job_store(a, rank, g, wid, j, lane, v0);
        wait_vm0();
        job_flag(a, rank, g, wid, j, lane, call);
        STAMP(1);
    }

    const int quad = idx >> 3;
    const int pm = (quad >> 1) * 8 + (xcd & 1) * 4 + (idx & 3), pn = (quad & 1) * 8 + (xcd >> 1) * 2 + ((idx >> 2) & 1);
    const char* const fX = myws + OFF_INF + (size_t)pm * 512 + lane * 8;
    const char* const fW = myws + OFF_INF + (size_t)(16 + pn) * 512 + lane * 8;
    bool bad = false;
    auto poll = [&](int need) -> int {
        for (;;) {
            u32x2 fa, fb;
            asm volatile("global_load_dwordx2 %0, %2, off sc0 sc1\n\tglobal_load_dwordx2 %1, %3, off sc0 sc1\n\ts_waitcnt vmcnt(0)"
                         : "=&v"(fa), "=&v"(fb) : "v"(fX), "v"(fW) : "memory");
            const u64 miss = __builtin_amdgcn_ballot_w64(fa.x < call || fa.y < call || fb.x < call || fb.y < call);
            const int upto = miss ? __builtin_ctzll(miss) : 64;
            if (upto >= need) return upto;
            if (expired(tmo, call, t0)) { bad = true; return 64; }
            __builtin_amdgcn_s_sleep(4);
        }
    };
    unsigned voffA[2], voffB[2];
#pragma unroll
    for (int i = 0; i < 2; ++i) {
        int R, C; stage_rc(tid * 16 + i * 8192, R, C);
        voffA[i] = tid * 16 + i * 8192;
        const int w_ = R >> 5, n_ = (R >> 4) & 1, q_ = (R & 15) >> 2, j_ = R & 3;
        voffB[i] = (w_ >> 1) * 16384 + (C >> 5) * 8192 + (4 * n_ + j_) * 1024 + ((w_ & 1) * 8 + q_) * 64 + ((C >> 3) & 3) * 16;
    }
    constexpr size_t kstep = 32768, hstepA = 16384, hstepB = 256;
    const unsigned ldsw = (unsigned)wid * 1024u;
    const int aoff = lds_byte(wr * 64 + fr, fq * 8), boff = lds_byte(wc * 32 + fr, fq * 8);
#define G_SA(b, h) (((b) * 2 + (h)) * HTB)
#define G_SB(b, h) ((4 + (b) * 2 + (h)) * HTB)
#define G_STAGE(bufoff, gbase, voff) do { _Pragma("unroll") for (int _i = 0; _i < 2; ++_i) \
        __builtin_amdgcn_global_load_lds((const unsigned*)((const char*)(gbase) + (voff)[_i]), (LAS unsigned*)(lds + (bufoff) + ldsw + _i * 8192), 16, 0, 17); } while (0)
#define G_LDA(dst, b, h) do { _Pragma("unroll") for (int m = 0; m < 4; ++m) _Pragma("unroll") for (int k = 0; k < 2; ++k) dst[m][k] = *(const LAS bf16x8*)(lds + G_SA(b, h) + aoff + m * 2048 + k * 1024); } while (0)
#define G_LDB(dst, b, h) do { _Pragma("unroll") for (int n = 0; n < 2; ++n) _Pragma("unroll") for (int k = 0; k < 2; ++k) dst[n][k] = *(const LAS bf16x8*)(lds + G_SB(b, h) + boff + n * 2048 + k * 1024); } while (0)
#define G_MMA(ai, bj, At, Bt) do { __builtin_amdgcn_s_setprio(1); _Pragma("unroll") for (int m = 0; m < 4; ++m) _Pragma("unroll") for (int n = 0; n < 2; ++n) _Pragma("unroll") for (int k = 0; k < 2; ++k) \
        acc[ai][bj][m][n] = __builtin_amdgcn_mfma_f32_16x16x32_bf16(Bt[n][k], At[m][k], acc[ai][bj][m][n], 0, 0, 0); __builtin_amdgcn_s_setprio(0); } while (0)
#define G_WAIT_V(n) asm volatile("s_waitcnt vmcnt(" #n ")" ::: "memory")
#define G_WAIT_L(n) asm volatile("s_waitcnt lgkmcnt(" #n ")" ::: "memory")
#define G_BAR __builtin_amdgcn_s_barrier()
#define G_SCHED __builtin_amdgcn_sched_barrier(0)
    f32x4 acc[2][2][4][2];
#pragma unroll
    for (int x = 0; x < 2; ++x)
#pragma unroll
        for (int y = 0; y < 2; ++y)
#pragma unroll
            for (int m = 0; m < 4; ++m)
#pragma unroll
                for (int n = 0; n < 2; ++n) acc[x][y][m][n] = (f32x4){0.f, 0.f, 0.f, 0.f};
    bf16x8 At[4][2], B0[2][2], B1[2][2];
    const char* const cA = myws + OFF_XI + (size_t)pm * 64 * kstep;
    const char* const cB = myws + OFF_WI + (size_t)pn * 64 * kstep;
    int ready = poll(2);
    STAMP(2);
    wait_vm0();
    G_STAGE(G_SB(0, 0), cB, voffB); G_STAGE(G_SA(0, 0), cA, voffA); G_STAGE(G_SB(0, 1), cB + hstepB, voffB); G_STAGE(G_SA(0, 1), cA + hstepA, voffA);
    if (wr == 1) G_BAR;
    G_WAIT_V(4); G_BAR;
    G_STAGE(G_SB(1, 0), cB + kstep, voffB); G_STAGE(G_SA(1, 0), cA + kstep, voffA); G_STAGE(G_SB(1, 1), cB + hstepB + kstep, voffB);
    G_WAIT_V(6); G_BAR;
    for (int t = 0; t < 64; t += 2) {
        const bool last = (t == 62);
        if (!last && ready < t + 4) ready = poll(t + 4);
        const char* a1 = cA + (size_t)(t + 1) * kstep;
        const char* a2 = last ? cA : cA + (size_t)(t + 2) * kstep; const char* b2 = last ? cB : cB + (size_t)(t + 2) * kstep;
        const char* a3 = a2 + kstep; const char* b3 = b2 + kstep;
        G_LDB(B0, 0, 0); G_SCHED; G_LDA(At, 0, 0); G_STAGE(G_SA(1, 1), a1 + hstepA, voffA);
        G_WAIT_L(8); G_BAR; G_WAIT_L(0); G_MMA(0, 0, At, B0); G_BAR; G_SCHED;
        G_LDB(B1, 0, 1); G_STAGE(G_SB(0, 0), b2, voffB);
        G_BAR; G_WAIT_L(0); G_MMA(0, 1, At, B1); G_BAR;
        G_LDA(At, 0, 1); G_STAGE(G_SA(0, 0), a2, voffA);
        G_BAR; G_WAIT_L(0); G_MMA(1, 0, At, B0); G_BAR; G_SCHED;
        G_STAGE(G_SB(0, 1), b2 + hstepB, voffB);
        G_WAIT_V(6); G_BAR; G_MMA(1, 1, At, B1); G_BAR;
        G_LDB(B0, 1, 0); G_SCHED; G_LDA(At, 1, 0); G_STAGE(G_SA(0, 1), a2 + hstepA, voffA);
        G_WAIT_L(8); G_BAR; G_WAIT_L(0); G_MMA(0, 0, At, B0); G_BAR; G_SCHED;
        G_LDB(B1, 1, 1); G_STAGE(G_SB(1, 0), b3, voffB);
        G_BAR; G_WAIT_L(0); G_MMA(0, 1, At, B1); G_BAR;
        G_LDA(At, 1, 1); G_STAGE(G_SA(1, 0), a3, voffA);
        G_BAR; G_WAIT_L(0); G_MMA(1, 0, At, B0); G_BAR; G_SCHED;
        G_STAGE(G_SB(1, 1), b3 + hstepB, voffB);
        G_WAIT_V(6); G_BAR; G_MMA(1, 1, At, B1); G_BAR;
    }
    G_WAIT_V(0);
    if (wr == 0) G_BAR;
    G_BAR;
    STAMP(3);

    if (__builtin_amdgcn_readfirstlane(__hip_atomic_load(tmo, __ATOMIC_RELAXED, __HIP_MEMORY_SCOPE_AGENT)) == call) bad = true;
    u32x4 W[16];
#pragma unroll
    for (int ai = 0; ai < 2; ++ai)
#pragma unroll
        for (int m = 0; m < 4; ++m)
#pragma unroll
            for (int bj = 0; bj < 2; ++bj) {
                const f32x4 v0 = acc[ai][bj][m][0], v1 = acc[ai][bj][m][1];
                u32x4 w; w[0] = relu_pk(cvt_pk_bf16(v0[0], v0[1])); w[1] = relu_pk(cvt_pk_bf16(v0[2], v0[3]));
                w[2] = relu_pk(cvt_pk_bf16(v1[0], v1[1])); w[3] = relu_pk(cvt_pk_bf16(v1[2], v1[3]));
                if (bad) w = (u32x4){0x7fc07fc0u, 0x7fc07fc0u, 0x7fc07fc0u, 0x7fc07fc0u};
                W[(ai * 4 + m) * 2 + bj] = w;
            }
    char* const outp = a.out[rank] + (size_t)(pm * 256 + wr * 64 + fr) * (GN * 2) + (size_t)(pn * 256 + wc * 64 + fq * 8) * 2;
#pragma unroll
    for (int i = 0; i < 16; ++i) *(GAS u32x4*)(outp + (size_t)rank * (NB * 2) + OUT_OFF(i)) = W[i];
    LAS unsigned char* const cw = lds + wid * C_WAVE;
    const unsigned info = encode_unit(W, cw, lane);
    const int wire = (info & INFO_RAW) ? 16384 : wire_bytes(info);
    STAMP(4);

    const int unit = b * 8 + wid;
    LAS unsigned* const sem = (LAS unsigned*)(lds + LDS_SEM);
    const unsigned lim = idx < 8 ? LIM1 : LIM2;
    if (idx < 8) {
        const unsigned* tq = (const unsigned*)(myws + OFF_TICK);
        const unsigned base = (call - 1u) * (512u - NPUSH) - 2u * NPUSH;
        while (__builtin_amdgcn_readfirstlane(__hip_atomic_load(tq, __ATOMIC_RELAXED, __HIP_MEMORY_SCOPE_AGENT)) - base < GATE) {
            if (expired(tmo, call, t0)) break;
            __builtin_amdgcn_s_sleep(32);
        }
    }
    STAMP(6);
    const u64* const inflag = (const u64*)(myws + OFF_OUTF) + ((rank + 1 + (lane < 7 ? lane : 0)) & 7) * 2048 + unit;
    int pushed = 0; unsigned todo = 0x7f;
    while (pushed < 7 || todo) {
        bool idle = true;
        if (pushed < 7) {
            unsigned got = 0;
            if (lane == 0) got = __hip_atomic_fetch_add(sem, 1u, __ATOMIC_RELAXED, __HIP_MEMORY_SCOPE_WORKGROUP);
            got = __builtin_amdgcn_readfirstlane(got);
            if (got < lim) {
                const int dest = (rank + 1 + (pushed + unit) % 7) & 7;
                char* const wsd = a.ws[dest];
                char* const slot = wsd + OFF_OC + (size_t)(rank * 2048 + unit) * 16384 + lane * 16;
#pragma unroll
                for (int i = 0; i < 16; ++i) if (i * 1024 < wire) { if (i * 1024 + lane * 16 < wire) st_wt(slot + i * 1024, W[i]); }
                wait_vm0();
                if (lane == 0) __hip_atomic_store((u64*)(wsd + OFF_OUTF) + rank * 2048 + unit, ((u64)call << 32) | info, __ATOMIC_RELAXED, __HIP_MEMORY_SCOPE_SYSTEM);
                ++pushed; idle = false;
                if (pushed == 7) STAMP(7);
            }
            if (lane == 0) __hip_atomic_fetch_sub(sem, 1u, __ATOMIC_RELAXED, __HIP_MEMORY_SCOPE_WORKGROUP);
        }
        if (todo) {
            const u64 f = __hip_atomic_load(inflag, __ATOMIC_RELAXED, __HIP_MEMORY_SCOPE_SYSTEM);
            const unsigned rdy = (unsigned)__builtin_amdgcn_ballot_w64((unsigned)(f >> 32) >= call) & todo;
            if (rdy) {
                const int k = __builtin_ctz(rdy), src = (rank + 1 + k) & 7;
                const unsigned inf = __builtin_amdgcn_readlane((unsigned)f, k);
                decode_unit(myws + OFF_OC + (size_t)(src * 2048 + unit) * 16384, inf, outp + (size_t)src * (NB * 2), cw, lane);
                todo &= ~(1u << k); idle = false;
            }
        }
        if (idle) {
            if (expired(tmo, call, t0)) { if (lane == 0) *(GAS u16*)(outp + (size_t)rank * (NB * 2)) = 0x7fc0; break; }
            __builtin_amdgcn_s_sleep(16);
        }
    }
    STAMP(5);
    STAMP_FLUSH();
}

extern "C" void kernel_launch(int rank, int nranks, ncclComm_t comm,
                              hipStream_t stream,
                              void* const* d_in, const int* in_sizes, int n_in,
                              void* d_out, int out_size,
                              const int* in_dtypes, int out_dtype,
                              void* const* d_out_all,
                              void* const* d_ws_all, size_t ws_bytes) {
    if (nranks != NR || (unsigned)rank >= (unsigned)NR || ws_bytes < WS_NEED || n_in < 2) { fprintf(stderr, "unsupported configuration\n"); return; }
    static bool attr[NR] = {};
    if (!attr[rank]) { HIP_CHECK(hipFuncSetAttribute((const void*)gemm_ar_kernel, hipFuncAttributeMaxDynamicSharedMemorySize, LDS_BYTES)); attr[rank] = true; }
    Args a;
    a.x = (const char*)d_in[0]; a.w = (const char*)d_in[1]; a.rank = rank;
    for (int p = 0; p < NR; ++p) { a.out[p] = (char*)d_out_all[p]; a.ws[p] = (char*)d_ws_all[p]; }
    hipLaunchKernelGGL(gemm_ar_kernel, dim3(256), dim3(512), LDS_BYTES, stream, a);
    HIP_CHECK(hipGetLastError());
}
```

```cpp
#include <hip/hip_runtime.h>
#include <rccl/rccl.h>
#include <cstdio>

#define HIP_CHECK(expr) do { hipError_t _e = (expr); if (_e != hipSuccess) { \
    fprintf(stderr, "HIP error %s at %s:%d\n", hipGetErrorString(_e), __FILE__, __LINE__); \
    return; } } while (0)

#define LAS __attribute__((address_space(3)))
#define GAS __attribute__((address_space(1)))
typedef unsigned short u16;
typedef unsigned long long u64;
typedef short bf16x8 __attribute__((ext_vector_type(8)));
typedef float f32x4 __attribute__((ext_vector_type(4)));
typedef unsigned u32x4 __attribute__((ext_vector_type(4)));
typedef unsigned u32x2 __attribute__((ext_vector_type(2)));

constexpr int NR = 8;
constexpr int GM = 4096, GN = 32768, KS = 512, NB = 4096;
constexpr int HTB = 16384;
constexpr int NG = 18;
constexpr int NIN = 512, NPOS = 2560;
constexpr int C_ARR = 1024, C_EXC = C_ARR + 16384 + 128, C_WAVE = C_EXC + 1024, EXC_CAP = 512;
constexpr int LDS_CTL = 8 * C_WAVE;
constexpr int LDS_BYTES = LDS_CTL + 64;
constexpr size_t OFF_CTR = 0;
constexpr size_t OFF_TMO = 32768;
constexpr size_t OFF_TICK = 32768 + 256;
constexpr size_t OFF_STAMP = 65536;
constexpr size_t OFF_INF = 512u << 10;
constexpr size_t OFF_Q = 1u << 20;
constexpr size_t OFF_XI = 4u << 20;
constexpr size_t OFF_WI = 36u << 20;
constexpr size_t OFF_XW = 68u << 20;
constexpr size_t OFF_WW = 72u << 20;
constexpr size_t OFF_OW = 104u << 20;
constexpr size_t WS_NEED = 136u << 20;
constexpr u64 DEADLINE = 50000000ull;

#define TOK 2
#define WIN 2
#define REL (WIN - 1)
#define DLY 8
#define WIRE_ALIGN 64
struct Args { const char* x; const char* w; char* out[NR]; char* ws[NR]; int rank; };

__device__ __forceinline__ int lds_byte(int r, int c) { const int st = (r >> 4) * 2 + (c >> 5), rr = r & 15, cc = c & 31, ob = rr * 64 + cc * 2; return st * 1024 + (ob ^ (((ob >> 9) & 1) << 5)); }
__device__ __forceinline__ void stage_rc(int b, int& R, int& C) { const int st = b / 1024, sb = b % 1024, swz = sb ^ (((sb >> 9) & 1) << 5); R = (st >> 1) * 16 + swz / 64; C = (st & 1) * 32 + (swz % 64) / 2; }
__device__ __forceinline__ unsigned cvt_pk_bf16(float lo, float hi) { unsigned r; asm volatile("v_cvt_pk_bf16_f32 %0, %1, %2" : "=v"(r) : "v"(lo), "v"(hi)); return r; }
__device__ __forceinline__ unsigned relu_pk(unsigned v) { unsigned r; asm("v_pk_max_i16 %0, %1, 0" : "=v"(r) : "v"(v)); return r; }
__device__ __forceinline__ void st_wt(char* p, u32x4 v) { asm volatile("global_store_dwordx4 %0, %1, off sc0 sc1\n\ts_nop 1" :: "v"(p), "v"(v) : "memory"); }
__device__ __forceinline__ void wait_vm0() { __builtin_amdgcn_s_waitcnt(0x0F70); asm volatile("s_waitcnt vmcnt(0)" ::: "memory"); }
template <int N> __device__ __forceinline__ void wait_vm() { asm volatile("s_waitcnt vmcnt(%0)" :: "n"(N) : "memory"); }
__device__ __forceinline__ void bar() { asm volatile("" ::: "memory"); __builtin_amdgcn_s_barrier(); asm volatile("" ::: "memory"); }
__device__ __forceinline__ u64 now() { return __builtin_amdgcn_s_memrealtime(); }

#define STAMP(i) do { } while (0)
#define STAMP1(i) do { } while (0)
#define LAP(i) do { } while (0)
#define STAMP_FLUSH() do { } while (0)

__device__ __forceinline__ int slot_of(int s, int p, int kt) { return s == p ? kt : 8 + 7 * kt + ((s - p - 1) & 7); }
__device__ __forceinline__ size_t img_off(int g, int j, int t) { return (j < 16 ? OFF_XI : OFF_WI) + (size_t)((g * 8 + ((j & 15) >> 1)) * 64 + t) * 32768 + (j & 1) * 16384; }
__device__ __forceinline__ size_t flag_off(int g, int j, int t) { return OFF_INF + (size_t)((((j >> 4) * 16 + g * 8 + ((j & 15) >> 1)) * 64 + t) * 2 + (j & 1)) * 4; }

#define OUT_OFF(i) ((size_t)(((i) >> 3) * 128 + ((i) & 7) * 8) * (GN * 2))
constexpr unsigned INFO_RAW = 1u << 31;
__device__ __forceinline__ int hi_off(int nch) { return 1024 + nch * 16; }
__device__ __forceinline__ int exc_off(int nch) { return 1024 + nch * 16 + ((nch + 3) & ~3) * 4; }
__device__ __forceinline__ int wire_bytes(unsigned info) { return (exc_off(info & 1023) + (int)((info >> 10) & 1023) * 2 + (WIRE_ALIGN - 1)) & ~(WIRE_ALIGN - 1); }
__device__ __forceinline__ unsigned pk_max_u16(unsigned x, unsigned y) { unsigned r; asm("v_pk_max_u16 %0, %1, %2" : "=v"(r) : "v"(x), "v"(y)); return r; }
__device__ __forceinline__ unsigned pk_sub_u16(unsigned x, unsigned y) { unsigned r; asm("v_pk_sub_u16 %0, %1, %2" : "=v"(r) : "v"(x), "v"(y)); return r; }
__device__ __forceinline__ unsigned pk_add_u16(unsigned x, unsigned y) { unsigned r; asm("v_pk_add_u16 %0, %1, %2" : "=v"(r) : "v"(x), "v"(y)); return r; }
__device__ __forceinline__ unsigned lanes_below(u64 bal) { return __builtin_amdgcn_mbcnt_hi((unsigned)(bal >> 32), __builtin_amdgcn_mbcnt_lo((unsigned)bal, 0u)); }
__device__ __forceinline__ void wait_lds() { asm volatile("s_waitcnt lgkmcnt(0)" ::: "memory"); }

__device__ __forceinline__ unsigned encode_unit(u32x4 (&W)[16], LAS unsigned char* cw, int lane) {
    unsigned mx = 0;
#pragma unroll
    for (int i = 0; i < 16; ++i) mx = pk_max_u16(mx, pk_max_u16(pk_max_u16(W[i][0], W[i][1]), pk_max_u16(W[i][2], W[i][3])));
    mx = max(mx & 0xffffu, mx >> 16);
#pragma unroll
    for (int o = 32; o >= 1; o >>= 1) mx = max(mx, (unsigned)__shfl_xor((int)mx, o));
    const unsigned M = __builtin_amdgcn_readfirstlane(mx), base = M > 1024u ? M - 1023u : 1u, base2 = base * 0x10001u;
    u32x4 mk = {0u, 0u, 0u, 0u};
    unsigned cnt = 0;
#pragma unroll
    for (int i = 0; i < 64; ++i) {
        const unsigned w = W[i >> 2][i & 3];
#pragma unroll
        for (int h = 0; h < 2; ++h) {
            const bool nz = h ? w > 0xffffu : (w & 0xffffu) != 0u;
            const u64 bal = __builtin_amdgcn_ballot_w64(nz);
            const unsigned pos = lanes_below(bal);
            if (nz) *(LAS u16*)(cw + C_ARR + cnt + 2 * pos) = (u16)(h ? w >> 16 : w);
            mk[i >> 4] = 2u * mk[i >> 4] + (nz ? 1u : 0u);
            cnt += 2u * (unsigned)__builtin_popcountll(bal);
        }
    }
    if (lane < 16) *(LAS u16*)(cw + C_ARR + cnt + 2 * lane) = (u16)base;
    *(LAS u32x4*)(cw + lane * 16) = mk;
    if (lane == 0) *(LAS u16*)(cw + C_EXC) = (u16)base;
    const int nch = (int)(cnt + 31u) >> 5;
    unsigned nexc = 1; bool over = false;
    unsigned hi[8];
#pragma unroll
    for (int it = 0; it < 8; ++it) {
        hi[it] = 0;
        if (it * 64 < nch) {
            const int c = it * 64 + lane;
            asm volatile("" ::: "memory");
            const u32x4 x = *(const LAS u32x4*)(cw + C_ARR + c * 32), y = *(const LAS u32x4*)(cw + C_ARR + c * 32 + 16);
            unsigned d[8];
#pragma unroll
            for (int q = 0; q < 4; ++q) { d[q] = pk_sub_u16(x[q], base2); d[4 + q] = pk_sub_u16(y[q], base2); }
            u32x4 lo; unsigned H[4];
#pragma unroll
            for (int q = 0; q < 4; ++q) { lo[q] = __builtin_amdgcn_perm(d[2 * q + 1], d[2 * q], 0x06040200u); H[q] = __builtin_amdgcn_perm(d[2 * q + 1], d[2 * q], 0x07050301u); }
            hi[it] = (H[0] & 0x03030303u) | ((H[1] & 0x03030303u) << 2) | ((H[2] & 0x03030303u) << 4) | ((H[3] & 0x03030303u) << 6);
            unsigned e = ((H[0] >> 7) & 0x01010101u) | ((H[1] >> 6) & 0x02020202u) | ((H[2] >> 5) & 0x04040404u) | ((H[3] >> 4) & 0x08080808u);
            if (c >= nch) e = 0;
            for (;;) {
                const u64 bal = __builtin_amdgcn_ballot_w64(e != 0u);
                if (!bal) break;
                const unsigned slot = nexc + lanes_below(bal);
                if (e) {
                    const int p = __builtin_ctz(e), q = p & 7, j = p >> 3;
                    e &= e - 1u;
                    const unsigned Hq = q == 0 ? H[0] : q == 1 ? H[1] : q == 2 ? H[2] : H[3];
                    const unsigned k = 64u - (((Hq >> (8 * j)) & 0xffu) >> 2);
                    if (k > 7u) over = true;
                    if (slot < EXC_CAP) *(LAS u16*)(cw + C_EXC + 2 * slot) = (u16)((unsigned)(c * 16 + q * 4 + j) | (k << 13));
                }
                nexc += (unsigned)__builtin_popcountll(bal);
            }
            asm volatile("" ::: "memory");
            *(LAS u32x4*)(cw + C_ARR + c * 16) = lo;
        }
    }
    if (nexc > EXC_CAP || __builtin_amdgcn_ballot_w64(over)) return INFO_RAW;
    asm volatile("" ::: "memory");
#pragma unroll
    for (int it = 0; it < 8; ++it) if (it * 64 < nch) { if (it * 64 + lane < nch) *(LAS unsigned*)(cw + hi_off(nch) + (it * 64 + lane) * 4) = hi[it]; }
    const u32x4 ex = *(const LAS u32x4*)(cw + C_EXC + lane * 16);
    *(LAS u32x4*)(cw + exc_off(nch) + lane * 16) = ex;
    asm volatile("" ::: "memory");
#pragma unroll
    for (int i = 0; i < 12; ++i) W[i] = *(const LAS u32x4*)(cw + i * 1024 + lane * 16);
    wait_lds();
    return (unsigned)nch | (nexc << 10);
}

__device__ __forceinline__ void in_job(const Args& a, char* myws, int dest, int g, int kt, int j, int lane, unsigned call) {
    const int panel = g * 8 + ((j & 15) >> 1), half = j & 1, rank = a.rank;
    u32x4 v[16];
    if (j < 16) {
        const int sb = lane * 16, swz = sb ^ (((sb >> 9) & 1) << 5);
        const char* src = a.x + (size_t)(panel * 256 + half * 128 + (swz >> 6)) * (KS * 2) + kt * 128 + (swz & 63);
#pragma unroll
        for (int i = 0; i < 16; ++i) v[i] = *(const u32x4*)(src + (i >> 1) * 16 * (KS * 2) + (i & 1) * 64);
    } else {
        const char* src = a.w + (size_t)(kt * 64 + (lane & 3) * 8) * (GN * 2) + (size_t)(dest * NB + panel * 256 + half * 128 + (lane >> 2) * 8) * 2;
        u32x4 r[16];
#pragma unroll
        for (int i = 0; i < 16; ++i) r[i] = *(const u32x4*)(src + (size_t)((i >> 3) * 32 + (i & 7)) * (GN * 2));
#pragma unroll
        for (int kh = 0; kh < 2; ++kh)
#pragma unroll
            for (int i = 0; i < 8; ++i)
#pragma unroll
                for (int d = 0; d < 4; ++d) v[kh * 8 + i][d] = __builtin_amdgcn_perm(r[kh * 8 + 2 * d + 1][i >> 1], r[kh * 8 + 2 * d][i >> 1], (i & 1) ? 0x07060302u : 0x05040100u);
    }
    const bool own = j < 16 || dest == rank, wire = j < 16 || dest != rank;
    if (wire) {
        char* dst = myws + (j < 16 ? OFF_XW : OFF_WW + (size_t)dest * (256 * 16384)) + (size_t)(g * 128 + kt * 16 + (j & 15)) * 16384 + lane * 16;
#pragma unroll
        for (int i = 0; i < 16; ++i) st_wt(dst + i * 1024, v[i]);
        wait_vm0();
        const int p = j < 16 ? (rank + 1 + lane) & 7 : dest;
        if (lane < (j < 16 ? 7 : 1))
            __hip_atomic_store((u64*)(a.ws[p] + OFF_Q) + rank * NPOS + g * 256 + kt * 32 + j, ((u64)call << 32) | INFO_RAW, __ATOMIC_RELAXED, __HIP_MEMORY_SCOPE_SYSTEM);
    }
    if (own) {
        char* dst = myws + img_off(g, j, kt) + lane * 16;
#pragma unroll
        for (int i = 0; i < 16; ++i) st_wt(dst + i * 1024, v[i]);
        wait_vm0();
        if (lane == 0) __hip_atomic_store((unsigned*)(myws + flag_off(g, j, kt)), call, __ATOMIC_RELAXED, __HIP_MEMORY_SCOPE_SYSTEM);
    }
}

__device__ __forceinline__ void decode_unit(unsigned info, char* outp, LAS unsigned char* cw, int lane) {
    const int nch = (int)(info & 1023u), nexc = (int)((info >> 10) & 1023u);
    u32x4 mk = *(const LAS u32x4*)(cw + lane * 16), L[8]; unsigned h[8], e[8];
#pragma unroll
    for (int i = 0; i < 8; ++i) {
        if (i * 64 < nch) { L[i] = *(const LAS u32x4*)(cw + 1024 + (i * 64 + lane) * 16); h[i] = *(const LAS unsigned*)(cw + hi_off(nch) + (i * 64 + lane) * 4); }
        if (i * 64 < nexc || i == 0) e[i] = *(const LAS u16*)(cw + exc_off(nch) + (i * 64 + lane) * 2);
    }
    wait_lds();
    const unsigned base2 = __builtin_amdgcn_readfirstlane(e[0]) * 0x10001u;
#pragma unroll
    for (int i = 0; i < 8; ++i) if (i * 64 < nch) {
        u32x4 x, y;
#pragma unroll
        for (int q = 0; q < 4; ++q) {
            const unsigned Hq = (h[i] >> (2 * q)) & 0x03030303u;
            const unsigned v0 = pk_add_u16(__builtin_amdgcn_perm(Hq, L[i][q], 0x05010400u), base2), v1 = pk_add_u16(__builtin_amdgcn_perm(Hq, L[i][q], 0x07030602u), base2);
            if (q < 2) { x[2 * q] = v0; x[2 * q + 1] = v1; } else { y[2 * q - 4] = v0; y[2 * q - 3] = v1; }
        }
        *(LAS u32x4*)(cw + C_ARR + (i * 64 + lane) * 32) = x; *(LAS u32x4*)(cw + C_ARR + (i * 64 + lane) * 32 + 16) = y;
    }
#pragma unroll
    for (int i = 0; i < 8; ++i) if (i * 64 < nexc) {
        if (i * 64 + lane < nexc && i * 64 + lane > 0)
            __hip_atomic_fetch_sub((LAS unsigned*)(cw + C_ARR + ((e[i] & 0x1ffeu) << 1)), (e[i] >> 13) << (10 + 16 * (e[i] & 1u)), __ATOMIC_RELAXED, __HIP_MEMORY_SCOPE_WORKGROUP);
    }
    asm volatile("" ::: "memory");
    unsigned cnt = 0;
#pragma unroll
    for (int i = 0; i < 16; ++i) {
        u32x4 o;
#pragma unroll
        for (int d = 0; d < 4; ++d) {
            unsigned v[2];
#pragma unroll
            for (int hh = 0; hh < 2; ++hh) {
                unsigned m2; const bool nz = __builtin_add_overflow(mk[i >> 2], mk[i >> 2], &m2); mk[i >> 2] = m2;
                const u64 bal = __builtin_amdgcn_ballot_w64(nz);
                const unsigned x = *(const LAS u16*)(cw + C_ARR + cnt + 2 * lanes_below(bal));
                v[hh] = nz ? x : 0u;
                cnt += 2u * (unsigned)__builtin_popcountll(bal);
            }
            o[d] = v[0] | (v[1] << 16);
        }
        st_wt(outp + OUT_OFF(i), o);
    }
}

__device__ __forceinline__ bool expired(unsigned* tmo, unsigned call, u64 t0) {
    if (__builtin_amdgcn_readfirstlane(__hip_atomic_load(tmo, __ATOMIC_RELAXED, __HIP_MEMORY_SCOPE_AGENT)) == call || now() - t0 > DEADLINE) {
        __hip_atomic_store(tmo, call, __ATOMIC_RELAXED, __HIP_MEMORY_SCOPE_AGENT);
        return true;
    }
    return false;
}

__device__ __forceinline__ void tile_of(int ti, int xcd, int& pm, int& pn) { pm = (ti >> 4) * 8 + (xcd & 1) * 4 + (ti & 3); pn = ((ti >> 3) & 1) * 8 + (xcd >> 1) * 2 + ((ti >> 2) & 1); }
__device__ __forceinline__ size_t out_off(int src, int pm, int pn, int w, int lane) { return (size_t)(pm * 256 + (w >> 2) * 64 + (lane >> 3)) * (GN * 2) + (size_t)(src * NB + pn * 256 + (w & 3) * 64) * 2 + (lane & 7) * 16; }

__global__ __launch_bounds__(512, 2) void gemm_ar_kernel(const Args a) {
    extern __shared__ __attribute__((aligned(16))) unsigned char shm[];
    LAS unsigned char* lds = (LAS unsigned char*)shm;
    const int tid = threadIdx.x, wid = __builtin_amdgcn_readfirstlane(tid >> 6), lane = tid & 63;
    const int b = blockIdx.x, xcd = b & 7, idx = b >> 3, rank = a.rank;
    char* const myws = a.ws[rank];
    char* const myout = a.out[rank];
    const u64 t0 = now();
    STAMP(0);
    unsigned* const ctr = (unsigned*)(myws + OFF_CTR) + b * 32 + wid;
    unsigned* const tmo = (unsigned*)(myws + OFF_TMO);
    const unsigned call = __builtin_amdgcn_readfirstlane(__hip_atomic_load(ctr, __ATOMIC_RELAXED, __HIP_MEMORY_SCOPE_AGENT)) + 1u;
    if (lane == 0) __hip_atomic_store(ctr, call, __ATOMIC_RELAXED, __HIP_MEMORY_SCOPE_AGENT);
    LAS unsigned char* const cw = lds + wid * C_WAVE;

    const int pk = (idx - NG + 7 * 8 + xcd) % 7, peer = (rank + 1 + pk) & 7, ord = xcd * 2 + (idx >= NG + 7);
    if (idx >= NG) { if (tid < 2) *(LAS unsigned*)(lds + LDS_CTL + tid * 4) = 0u; wait_lds(); bar(); }
    {
        int q, qe = 0, delay;
        if (idx >= NG) { q = (pk + 1) * 512 + wid * 32 + 16 + ord; delay = wid; }
        else if (idx < 8) { const int u = idx * 8 + xcd; q = (u >> 5) * 256 + wid * 32 + (u & 31); delay = (u >> 5) * 8 + wid; }
        else { q = ((idx - 8) * 8 + xcd) * 8 + wid; qe = 896; delay = 10; }
        for (int i = 0; i < delay; ++i) __builtin_amdgcn_s_sleep(DLY);
        do {
            const int r = qe ? (q % 7 + 1) * 512 + 256 + ((q / 7) >> 4) * 32 + 16 + ((q / 7) & 15) : q;
            in_job(a, myws, (rank + (r >> 9)) & 7, (r >> 8) & 1, (r >> 5) & 7, r & 31, lane, call);
            q += 640;
        } while (q < qe);
    }
    STAMP(1);

    if (idx >= NG) {
        const char* const pws = a.ws[peer];
        const u64* const queue = (const u64*)(myws + OFF_Q) + peer * NPOS;
        unsigned* const tq = (unsigned*)(myws + OFF_TICK) + pk * 64;
        const unsigned tbase = (call - 1u) * NPOS - 128u;
        const unsigned ctl = LDS_CTL;
        unsigned pos = wid * 16 + ord;
        bool bad = false;
        while (pos < (unsigned)NPOS) {
            unsigned info;
            for (;;) {
                const u64 e = __hip_atomic_load(queue + pos, __ATOMIC_RELAXED, __HIP_MEMORY_SCOPE_SYSTEM);
                info = __builtin_amdgcn_readfirstlane((unsigned)e);
                if (__builtin_amdgcn_readfirstlane((unsigned)(e >> 32)) >= call) break;
                if (expired(tmo, call, t0)) { bad = true; break; }
                __builtin_amdgcn_s_sleep(4);
            }
            if (bad) break;
            LAP(24); STAMP1(2);
            if ((info & 1023u) > 512u || ((info >> 10) & 1023u) > (unsigned)EXC_CAP) info = INFO_RAW;
            const int g = pos >> 8, kt = (pos >> 5) & 7, j = pos & 31;
            const char* const src = pws + (pos >= NIN ? OFF_OW + (size_t)(pos - NIN) * 16384
                                           : (j < 16 ? OFF_XW : OFF_WW + (size_t)rank * (256 * 16384)) + (size_t)(g * 128 + kt * 16 + (j & 15)) * 16384) + lane * 16;
            const int wire = (info & INFO_RAW) ? 16384 : wire_bytes(info);
            unsigned my = 0;
            if (lane == 0) asm volatile("ds_add_rtn_u32 %0, %1, %2\n\ts_waitcnt lgkmcnt(0)" : "=v"(my) : "v"(ctl), "v"(1u) : "memory");
            my = __builtin_amdgcn_readfirstlane(my);
            for (;;) {
                unsigned rel;
                asm volatile("ds_read_b32 %0, %1 offset:4\n\ts_waitcnt lgkmcnt(0)" : "=v"(rel) : "v"(ctl) : "memory");
                if ((int)(__builtin_amdgcn_readfirstlane(rel) + TOK - my) > 0) break;
                if (now() - t0 > DEADLINE) break;
                __builtin_amdgcn_s_sleep(1);
            }
            LAP(25);
            for (int o = 0; o < wire; o += 1024) {
                if (o + lane * 16 < wire) __builtin_amdgcn_global_load_lds((const unsigned*)(src + o), (LAS unsigned*)(cw + o), 16, 0, 17);
                wait_vm<WIN - 1>();
            }
            wait_vm<REL>();
            if (lane == 0) asm volatile("ds_add_u32 %0, %1 offset:4" :: "v"(ctl), "v"(1u) : "memory");
            wait_vm0();
            LAP(26); STAMP1(3);
            int t = 0;
            char* dst;
            if (pos >= NIN) { int pm, pn; tile_of((pos - NIN) >> 6, ((pos - NIN) >> 3) & 7, pm, pn); dst = myout + out_off(peer, pm, pn, pos & 7, lane); }
            else { t = slot_of(peer, rank, kt); dst = myws + img_off(g, j, t) + lane * 16; }
            if (info & INFO_RAW) {
                u32x4 v[16];
#pragma unroll
                for (int i = 0; i < 16; ++i) v[i] = *(const LAS u32x4*)(cw + i * 1024 + lane * 16);
#pragma unroll
                for (int i = 0; i < 16; ++i) st_wt(dst + (pos >= NIN ? OUT_OFF(i) : (size_t)i * 1024), v[i]);
            } else decode_unit(info, dst, cw, lane);
            LAP(27);
            unsigned nx = 0;
            if (lane == 0) nx = __hip_atomic_fetch_add(tq, 1u, __ATOMIC_RELAXED, __HIP_MEMORY_SCOPE_AGENT);
            wait_vm0();
            if (pos < NIN && lane == 0) __hip_atomic_store((unsigned*)(myws + flag_off(g, j, t)), call, __ATOMIC_RELAXED, __HIP_MEMORY_SCOPE_SYSTEM);
            LAP(28); STAMP(8 + (pos >> 8));
            pos = __builtin_amdgcn_readfirstlane(nx) - tbase;
        }
        if (bad && lane == 0) *(GAS u16*)(myout + (size_t)rank * (NB * 2)) = 0x7fc0;
        STAMP(5);
        STAMP_FLUSH();
        return;
    }

    const int wr = wid >> 2, wc = wid & 3, fr = lane & 15, fq = lane >> 4;
    bool bad = false;
    unsigned voffA[2], voffB[2];
#pragma unroll
    for (int i = 0; i < 2; ++i) {
        int R, C; stage_rc(tid * 16 + i * 8192, R, C);
        voffA[i] = tid * 16 + i * 8192;
        const int w_ = R >> 5, n_ = (R >> 4) & 1, q_ = (R & 15) >> 2, j_ = R & 3;
        voffB[i] = (w_ >> 1) * 16384 + (C >> 5) * 8192 + (4 * n_ + j_) * 1024 + ((w_ & 1) * 8 + q_) * 64 + ((C >> 3) & 3) * 16;
    }
    constexpr size_t kstep = 32768, hstepA = 16384, hstepB = 256;
    const unsigned ldsw = (unsigned)wid * 1024u;
    const int aoff = lds_byte(wr * 64 + fr, fq * 8), boff = lds_byte(wc * 32 + fr, fq * 8);
#define G_SA(b, h) (((b) * 2 + (h)) * HTB)
#define G_SB(b, h) ((4 + (b) * 2 + (h)) * HTB)
#define G_STAGE(bufoff, gbase, voff) do { _Pragma("unroll") for (int _i = 0; _i < 2; ++_i) \
        __builtin_amdgcn_global_load_lds((const unsigned*)((const char*)(gbase) + (voff)[_i]), (LAS unsigned*)(lds + (bufoff) + ldsw + _i * 8192), 16, 0, 17); } while (0)
#define G_LDA(dst, b, h) do { _Pragma("unroll") for (int m = 0; m < 4; ++m) _Pragma("unroll") for (int k = 0; k < 2; ++k) dst[m][k] = *(const LAS bf16x8*)(lds + G_SA(b, h) + aoff + m * 2048 + k * 1024); } while (0)
#define G_LDB(dst, b, h) do { _Pragma("unroll") for (int n = 0; n < 2; ++n) _Pragma("unroll") for (int k = 0; k < 2; ++k) dst[n][k] = *(const LAS bf16x8*)(lds + G_SB(b, h) + boff + n * 2048 + k * 1024); } while (0)
#define G_MMA(ai, bj, At, Bt) do { __builtin_amdgcn_s_setprio(1); _Pragma("unroll") for (int m = 0; m < 4; ++m) _Pragma("unroll") for (int n = 0; n < 2; ++n) _Pragma("unroll") for (int k = 0; k < 2; ++k) \
        acc[ai][bj][m][n] = __builtin_amdgcn_mfma_f32_16x16x32_bf16(Bt[n][k], At[m][k], acc[ai][bj][m][n], 0, 0, 0); __builtin_amdgcn_s_setprio(0); } while (0)
#define G_WAIT_V(n) asm volatile("s_waitcnt vmcnt(" #n ")" ::: "memory")
#define G_WAIT_L(n) asm volatile("s_waitcnt lgkmcnt(" #n ")" ::: "memory")
#define G_BAR __builtin_amdgcn_s_barrier()
#define G_SCHED __builtin_amdgcn_sched_barrier(0)
    for (int ti = idx; ti < 32; ti += NG) {
    int pm, pn; tile_of(ti, xcd, pm, pn);
    const char* const fX = myws + OFF_INF + (size_t)pm * 512 + lane * 8;
    const char* const fW = myws + OFF_INF + (size_t)(16 + pn) * 512 + lane * 8;
    auto poll = [&](int need) -> int {
        for (;;) {
            u32x2 fa, fb;
            asm volatile("global_load_dwordx2 %0, %2, off sc0 sc1\n\tglobal_load_dwordx2 %1, %3, off sc0 sc1\n\ts_waitcnt vmcnt(0)"
                         : "=&v"(fa), "=&v"(fb) : "v"(fX), "v"(fW) : "memory");
            const u64 miss = __builtin_amdgcn_ballot_w64(fa.x < call || fa.y < call || fb.x < call || fb.y < call);
            const int upto = miss ? __builtin_ctzll(miss) : 64;
            if (upto >= need) return upto;
            if (expired(tmo, call, t0)) { bad = true; return 64; }
            __builtin_amdgcn_s_sleep(4);
        }
    };
    f32x4 acc[2][2][4][2];
#pragma unroll
    for (int x = 0; x < 2; ++x)
#pragma unroll
        for (int y = 0; y < 2; ++y)
#pragma unroll
            for (int m = 0; m < 4; ++m)
#pragma unroll
                for (int n = 0; n < 2; ++n) acc[x][y][m][n] = (f32x4){0.f, 0.f, 0.f, 0.f};
    bf16x8 At[4][2], B0[2][2], B1[2][2];
    const char* const cA = myws + OFF_XI + (size_t)pm * 64 * kstep;
    const char* const cB = myws + OFF_WI + (size_t)pn * 64 * kstep;
    int ready = poll(2);
    STAMP1(2);
    wait_vm0();
    G_STAGE(G_SB(0, 0), cB, voffB); G_STAGE(G_SA(0, 0), cA, voffA); G_STAGE(G_SB(0, 1), cB + hstepB, voffB); G_STAGE(G_SA(0, 1), cA + hstepA, voffA);
    if (wr == 1) G_BAR;
    G_WAIT_V(4); G_BAR;
    G_STAGE(G_SB(1, 0), cB + kstep, voffB); G_STAGE(G_SA(1, 0), cA + kstep, voffA); G_STAGE(G_SB(1, 1), cB + hstepB + kstep, voffB);
    G_WAIT_V(6); G_BAR;
    for (int t = 0; t < 64; t += 2) {
        const bool last = (t == 62);
        if (!last && ready < t + 4) ready = poll(t + 4);
        const char* a1 = cA + (size_t)(t + 1) * kstep;
        const char* a2 = last ? cA : cA + (size_t)(t + 2) * kstep; const char* b2 = last ? cB : cB + (size_t)(t + 2) * kstep;
        const char* a3 = a2 + kstep; const char* b3 = b2 + kstep;
        G_LDB(B0, 0, 0); G_SCHED; G_LDA(At, 0, 0); G_STAGE(G_SA(1, 1), a1 + hstepA, voffA);
        G_WAIT_L(8); G_BAR; G_WAIT_L(0); G_MMA(0, 0, At, B0); G_BAR; G_SCHED;
        G_LDB(B1, 0, 1); G_STAGE(G_SB(0, 0), b2, voffB);
        G_BAR; G_WAIT_L(0); G_MMA(0, 1, At, B1); G_BAR;
        G_LDA(At, 0, 1); G_STAGE(G_SA(0, 0), a2, voffA);
        G_BAR; G_WAIT_L(0); G_MMA(1, 0, At, B0); G_BAR; G_SCHED;
        G_STAGE(G_SB(0, 1), b2 + hstepB, voffB);
        G_WAIT_V(6); G_BAR; G_MMA(1, 1, At, B1); G_BAR;
        G_LDB(B0, 1, 0); G_SCHED; G_LDA(At, 1, 0); G_STAGE(G_SA(0, 1), a2 + hstepA, voffA);
        G_WAIT_L(8); G_BAR; G_WAIT_L(0); G_MMA(0, 0, At, B0); G_BAR; G_SCHED;
        G_LDB(B1, 1, 1); G_STAGE(G_SB(1, 0), b3, voffB);
        G_BAR; G_WAIT_L(0); G_MMA(0, 1, At, B1); G_BAR;
        G_LDA(At, 1, 1); G_STAGE(G_SA(1, 0), a3, voffA);
        G_BAR; G_WAIT_L(0); G_MMA(1, 0, At, B0); G_BAR; G_SCHED;
        G_STAGE(G_SB(1, 1), b3 + hstepB, voffB);
        G_WAIT_V(6); G_BAR; G_MMA(1, 1, At, B1); G_BAR;
    }
    G_WAIT_V(0);
    if (wr == 0) G_BAR;
    G_BAR;
    STAMP(ti < NG ? 3 : 6);

    if (__builtin_amdgcn_readfirstlane(__hip_atomic_load(tmo, __ATOMIC_RELAXED, __HIP_MEMORY_SCOPE_AGENT)) == call) bad = true;
    int ln = lane;
    asm volatile("" : "+v"(ln) :: "memory");
#pragma unroll
    for (int ai = 0; ai < 2; ++ai)
#pragma unroll
        for (int m = 0; m < 4; ++m)
#pragma unroll
            for (int bj = 0; bj < 2; ++bj) {
                const f32x4 v0 = acc[ai][bj][m][0], v1 = acc[ai][bj][m][1];
                u32x4 w; w[0] = relu_pk(cvt_pk_bf16(v0[0], v0[1])); w[1] = relu_pk(cvt_pk_bf16(v0[2], v0[3]));
                w[2] = relu_pk(cvt_pk_bf16(v1[0], v1[1])); w[3] = relu_pk(cvt_pk_bf16(v1[2], v1[3]));
                if (bad) w = (u32x4){0x7fc07fc0u, 0x7fc07fc0u, 0x7fc07fc0u, 0x7fc07fc0u};
                *(LAS u32x4*)(cw + (ai * 64 + m * 16 + (ln & 15)) * 128 + (((bj * 4 + (ln >> 4)) ^ (ln & 7)) << 4)) = w;
            }
    asm volatile("" ::: "memory");
    u32x4 W[16];
#pragma unroll
    for (int i = 0; i < 16; ++i) W[i] = *(const LAS u32x4*)(cw + i * 1024 + (ln >> 3) * 128 + (((ln & 7) ^ (ln >> 3)) << 4));
    wait_lds();
    char* const outp = myout + out_off(rank, pm, pn, wid, ln);
#pragma unroll
    for (int i = 0; i < 16; ++i) st_wt(outp + OUT_OFF(i), W[i]);
    const unsigned info = encode_unit(W, cw, ln);
    const int wire = (info & INFO_RAW) ? 16384 : wire_bytes(info);
    char* const slot = myws + OFF_OW + (size_t)((ti * 8 + xcd) * 8 + wid) * 16384 + ln * 16;
#pragma unroll
    for (int i = 0; i < 16; ++i) if (i * 1024 < wire) { if (i * 1024 + ln * 16 < wire) st_wt(slot + i * 1024, W[i]); }
    if (ln == 0) *(LAS unsigned*)(lds + LDS_CTL + wid * 4) = info;
    wait_lds();
    wait_vm0();
    bar();
    if (wid == 0 && ln < 56) {
        const unsigned inf = *(const LAS unsigned*)(lds + LDS_CTL + (ln & 7) * 4);
        __hip_atomic_store((u64*)(a.ws[(rank + 1 + (ln >> 3)) & 7] + OFF_Q) + rank * NPOS + NIN + (ti * 8 + xcd) * 8 + (ln & 7), ((u64)call << 32) | inf, __ATOMIC_RELAXED, __HIP_MEMORY_SCOPE_SYSTEM);
    }
    STAMP(ti < NG ? 4 : 7);
    }
    STAMP(5);
    STAMP_FLUSH();
}

extern "C" void kernel_launch(int rank, int nranks, ncclComm_t comm,
                              hipStream_t stream,
                              void* const* d_in, const int* in_sizes, int n_in,
                              void* d_out, int out_size,
                              const int* in_dtypes, int out_dtype,
                              void* const* d_out_all,
                              void* const* d_ws_all, size_t ws_bytes) {
    if (nranks != NR || (unsigned)rank >= (unsigned)NR || ws_bytes < WS_NEED || n_in < 2) { fprintf(stderr, "unsupported configuration\n"); return; }
    static bool attr[NR] = {};
    if (!attr[rank]) { HIP_CHECK(hipFuncSetAttribute((const void*)gemm_ar_kernel, hipFuncAttributeMaxDynamicSharedMemorySize, LDS_BYTES)); attr[rank] = true; }
    Args a;
    a.x = (const char*)d_in[0]; a.w = (const char*)d_in[1]; a.rank = rank;
    for (int p = 0; p < NR; ++p) { a.out[p] = (char*)d_out_all[p]; a.ws[p] = (char*)d_ws_all[p]; }
    hipLaunchKernelGGL(gemm_ar_kernel, dim3(256), dim3(512), LDS_BYTES, stream, a);
    HIP_CHECK(hipGetLastError());
}
```

```cpp
#include <hip/hip_runtime.h>
#include <rccl/rccl.h>
#include <cstdio>

#define HIP_CHECK(expr) do { hipError_t _e = (expr); if (_e != hipSuccess) { \
    fprintf(stderr, "HIP error %s at %s:%d\n", hipGetErrorString(_e), __FILE__, __LINE__); \
    return; } } while (0)

#define LAS __attribute__((address_space(3)))
#define GAS __attribute__((address_space(1)))
typedef unsigned short u16;
typedef unsigned long long u64;
typedef short bf16x8 __attribute__((ext_vector_type(8)));
typedef float f32x4 __attribute__((ext_vector_type(4)));
typedef unsigned u32x4 __attribute__((ext_vector_type(4)));
typedef unsigned u32x2 __attribute__((ext_vector_type(2)));

constexpr int NR = 8;
constexpr int GM = 4096, GN = 32768, KS = 512, NB = 4096;
constexpr int HTB = 16384;
constexpr int NG = 18;
constexpr int NIN = 512, NPOS = 2560;
constexpr unsigned LDS_TAB = 64, LDS_CW = 256, C_WAVE = 16384 + 128, LDS_TL = LDS_CW + 8 * C_WAVE, C_TAIL = 3584, ESC_CAP = 3560;
constexpr unsigned LDS_CTL = LDS_TL + 8 * C_TAIL + 64;
constexpr int LDS_BYTES = LDS_CTL + 64;
static_assert(LDS_TL >= 8 * HTB && LDS_BYTES <= 163840, "LDS layout");
constexpr size_t OFF_CTR = 0;
constexpr size_t OFF_TMO = 32768;
constexpr size_t OFF_TICK = 32768 + 256;
constexpr size_t OFF_STAMP = 65536;
constexpr size_t OFF_INF = 512u << 10;
constexpr size_t OFF_Q = 1u << 20;
constexpr size_t OFF_XI = 4u << 20;
constexpr size_t OFF_WI = 36u << 20;
constexpr size_t OFF_XW = 68u << 20;
constexpr size_t OFF_WW = 72u << 20;
constexpr size_t OFF_OW = 104u << 20;
constexpr size_t WS_NEED = 136u << 20;
constexpr u64 DEADLINE = 50000000ull;

#define TOK 2
#define WIN 2
#define REL (WIN - 1)
#define DLY1 2
#define DLY2 3
#define WIRE_ALIGN 64
struct Args { const char* x; const char* w; char* myws; char* myout; int rank; char* ws[NR]; };

__device__ __forceinline__ int lds_byte(int r, int c) { const int st = (r >> 4) * 2 + (c >> 5), rr = r & 15, cc = c & 31, ob = rr * 64 + cc * 2; return st * 1024 + (ob ^ (((ob >> 9) & 1) << 5)); }
__device__ __forceinline__ void stage_rc(int b, int& R, int& C) { const int st = b / 1024, sb = b % 1024, swz = sb ^ (((sb >> 9) & 1) << 5); R = (st >> 1) * 16 + swz / 64; C = (st & 1) * 32 + (swz % 64) / 2; }
__device__ __forceinline__ unsigned cvt_pk_bf16(float lo, float hi) { unsigned r; asm volatile("v_cvt_pk_bf16_f32 %0, %1, %2" : "=v"(r) : "v"(lo), "v"(hi)); return r; }
__device__ __forceinline__ unsigned relu_pk(unsigned v) { unsigned r; asm("v_pk_max_i16 %0, %1, 0" : "=v"(r) : "v"(v)); return r; }
__device__ __forceinline__ void st_wt(char* p, u32x4 v) { asm volatile("global_store_dwordx4 %0, %1, off sc0 sc1\n\ts_nop 1" :: "v"(p), "v"(v) : "memory"); }
__device__ __forceinline__ void wait_vm0() { __builtin_amdgcn_s_waitcnt(0x0F70); asm volatile("s_waitcnt vmcnt(0)" ::: "memory"); }
template <int N> __device__ __forceinline__ void wait_vm() { asm volatile("s_waitcnt vmcnt(%0)" :: "n"(N) : "memory"); }
__device__ __forceinline__ void bar() { asm volatile("" ::: "memory"); __builtin_amdgcn_s_barrier(); asm volatile("" ::: "memory"); }
__device__ __forceinline__ u64 now() { return __builtin_amdgcn_s_memrealtime(); }

#define STAMP(i) do { } while (0)
#define STAMP1(i) do { } while (0)
#define STAMPD(i) do { } while (0)
#define DBG_ARGS
#define DBG_PASS
#define LAP(i) do { } while (0)
#define STAMP_FLUSH() do { } while (0)

__device__ __forceinline__ int slot_of(int s, int p, int kt) { return s == p ? kt : 8 + 7 * kt + ((s - p - 1) & 7); }
__device__ __forceinline__ size_t img_off(int g, int j, int t) { return (j < 16 ? OFF_XI : OFF_WI) + (size_t)((g * 8 + ((j & 15) >> 1)) * 64 + t) * 32768 + (j & 1) * 16384; }
__device__ __forceinline__ size_t flag_off(int g, int j, int t) { return OFF_INF + (size_t)((((j >> 4) * 16 + g * 8 + ((j & 15) >> 1)) * 64 + t) * 2 + (j & 1)) * 4; }

#define OUT_OFF(i) ((size_t)(((i) >> 3) * 128 + ((i) & 7) * 8) * (GN * 2))
constexpr unsigned INFO_RAW = 1u << 31;
constexpr int NW_CAP = 384;
template <bool OUT> __device__ __forceinline__ int cls_off(int nch) { return OUT ? 1024 + nch * 16 : 8192; }
template <bool OUT> __device__ __forceinline__ int cls_bytes(int nch) { return OUT ? (nch * 2 + 3) & ~3 : 2048; }
template <bool OUT> __device__ __forceinline__ int wire_bytes(unsigned info) {
    const int nch = info & 1023; return (cls_off<OUT>(nch) + cls_bytes<OUT>(nch) + (int)((info >> 10) & 511) * 4 + (WIRE_ALIGN - 1)) & ~(WIRE_ALIGN - 1); }
__device__ __forceinline__ unsigned pk_min_u16(unsigned x, unsigned y) { unsigned r; asm("v_pk_min_u16 %0, %1, %2" : "=v"(r) : "v"(x), "s"(y)); return r; }
__device__ __forceinline__ unsigned pk_rsub(unsigned s, unsigned y) { unsigned r; asm("v_pk_sub_u16 %0, %1, %2" : "=v"(r) : "s"(s), "v"(y)); return r; }
__device__ __forceinline__ unsigned sum_u16(unsigned x, unsigned acc) { unsigned r; asm("v_sad_u16 %0, %1, 0, %2" : "=v"(r) : "v"(x), "v"(acc)); return r; }
template <int S> __device__ __forceinline__ unsigned lshl_add(unsigned x, unsigned y) { unsigned r; asm("v_lshl_add_u32 %0, %1, %2, %3" : "=v"(r) : "v"(x), "n"(S), "v"(y)); return r; }
template <int S> __device__ __forceinline__ unsigned lshl_or(unsigned x, unsigned y) { unsigned r; asm("v_lshl_or_b32 %0, %1, %2, %3" : "=v"(r) : "v"(x), "n"(S), "v"(y)); return r; }
__device__ __forceinline__ unsigned perm(unsigned hi, unsigned lo, unsigned sel) { return __builtin_amdgcn_perm(hi, lo, sel); }
__device__ __forceinline__ void wait_lds() { asm volatile("s_waitcnt lgkmcnt(0)" ::: "memory"); }
__device__ __forceinline__ void fence() { asm volatile("" ::: "memory"); }
template <int C, int RM> __device__ __forceinline__ unsigned dpp0(unsigned x) { return (unsigned)__builtin_amdgcn_update_dpp(0, (int)x, C, RM, 0xf, false); }
__device__ __forceinline__ unsigned scan_add(unsigned x) {
    x += dpp0<0x111, 0xf>(x); x += dpp0<0x112, 0xf>(x); x += dpp0<0x114, 0xf>(x); x += dpp0<0x118, 0xf>(x); x += dpp0<0x142, 0xa>(x); x += dpp0<0x143, 0xc>(x); return x; }
__device__ __forceinline__ unsigned scan_max(unsigned x) {
    x = max(x, dpp0<0x111, 0xf>(x)); x = max(x, dpp0<0x112, 0xf>(x)); x = max(x, dpp0<0x114, 0xf>(x)); x = max(x, dpp0<0x118, 0xf>(x)); x = max(x, dpp0<0x142, 0xa>(x)); x = max(x, dpp0<0x143, 0xc>(x)); return x; }
__device__ __forceinline__ unsigned last_lane(unsigned x) { return (unsigned)__builtin_amdgcn_readlane((int)x, 63); }
template <class T> __device__ __forceinline__ LAS T* ldsp(unsigned a) { return (LAS T*)(size_t)a; }

template <bool OUT, class GET, class MID>
__device__ __forceinline__ unsigned encode_values(GET get, MID mid, int nch, unsigned T, const u32x4& mk, unsigned tail, char* dst, int lane) {
    const unsigned T2 = T * 0x10001u;
#pragma unroll
    for (int i = 0; i < 4; ++i) if (i < 3 || lane < 32) *ldsp<u32x4>(tail + i * 1024 + lane * 16) = (u32x4){0u, 0u, 0u, 0u};
    fence();
    u32x4 Lb[8], Ln[8]; unsigned tot = 0, direct = 0;
#pragma unroll
    for (int it = 0; it < 8; ++it) if (!OUT || it * 64 < nch) {
        u32x4 x, y; get(it, x, y);
        unsigned h = 0, h0 = 0;
#pragma unroll
        for (int q = 0; q < 4; ++q) {
            const unsigned w0 = q < 2 ? x[2 * q] : y[2 * q - 4], w1 = q < 2 ? x[2 * q + 1] : y[2 * q - 3];
            const unsigned r0 = pk_rsub(T2, __builtin_amdgcn_alignbit(w0, w0, 31)), r1 = pk_rsub(T2, __builtin_amdgcn_alignbit(w1, w1, 31));
            const unsigned ec = perm(pk_min_u16(r1, 0x07ff07ffu), pk_min_u16(r0, 0x07ff07ffu), 0x07050301u);
            Ln[it][q] = perm(r1, r0, 0x07050301u) + perm(0x01000000u, 0x00010207u, ec) - 0x03030303u;
            tot = __builtin_amdgcn_sad_u8(Ln[it][q], 0u, tot);
            Lb[it][q] = perm(r1, r0, 0x06040200u);
            if (OUT) { const unsigned b0 = perm(0u, 0x01000100u, ec); Lb[it][q] |= b0; h0 |= b0 << q; h |= perm(0u, 0x01010000u, ec) << q; }
            else h |= perm(0u, 0x03020100u, ec) << (2 * q);
        }
        if (OUT) { direct += __builtin_popcount(h | h0); if (it * 64 + lane < nch) *ldsp<u16>(tail + (it * 64 + lane) * 2) = (u16)(h | (h >> 12)); }
        else { direct += __builtin_popcount((h | (h >> 1)) & 0x55555555u); *ldsp<unsigned>(tail + (it * 64 + lane) * 4) = h; }
    }
    const unsigned inc = scan_add(tot), nw = (last_lane(inc) + 33u) >> 5;
    const unsigned nesc = (OUT ? (unsigned)((nch + 63) >> 6) * 1024u : 8192u) - last_lane(scan_add(direct));
    mid();
    if (nw > (unsigned)NW_CAP || nesc > (unsigned)ESC_CAP) return INFO_RAW;
    if (OUT) st_wt(dst, mk);
    unsigned pos = (tail + cls_bytes<OUT>(nch)) * 8u + 1u + inc - tot;
#pragma unroll
    for (int it = 0; it < 8; ++it) if (!OUT || it * 64 < nch) {
        if (!OUT || it * 64 + lane < nch) st_wt(dst + (OUT ? 1024 : 0) + it * 1024, Lb[it]);
#pragma unroll
        for (int e = 0; e < 16; ++e) {
            pos += (Ln[it][e >> 2] >> (8 * (e & 3))) & 0xffu;
            asm volatile("ds_or_b32 %0, %1" :: "v"((pos >> 3) & ~3u), "v"(0x80000000u >> (pos & 31u)) : "memory");
        }
    }
    const unsigned info = (T >> 4) << 19 | nw << 10 | (OUT ? (unsigned)nch : 0u);
    const int rest = wire_bytes<OUT>(info) - cls_off<OUT>(nch);
    wait_lds();
#pragma unroll
    for (int i = 0; i < 4; ++i) if (i * 1024 < rest) { if (i * 1024 + lane * 16 < rest) st_wt(dst + cls_off<OUT>(nch) + i * 1024, *ldsp<u32x4>(tail + i * 1024 + lane * 16)); }
    return info;
}
__device__ __forceinline__ unsigned top_of(unsigned sum2, unsigned n) {
    return ((unsigned)__builtin_amdgcn_readfirstlane((int)((float)sum2 * __builtin_amdgcn_rcpf((float)n))) + 2u * (276u + 128u) + 8u) & 0xfff0u; }

template <class MID> __device__ __forceinline__ unsigned encode_in(const u32x4 (&v)[16], unsigned tail, char* dst, int lane, MID mid) {
    unsigned s = 0;
#pragma unroll
    for (int i = 0; i < 16; ++i) s = sum_u16(__builtin_amdgcn_alignbit(v[i][i & 3], v[i][i & 3], 31), s);
    return encode_values<false>([&](int it, u32x4& x, u32x4& y) { x = v[2 * it]; y = v[2 * it + 1]; }, mid, 512, top_of(last_lane(scan_add(s)), 2048u), v[0], tail, dst, lane);
}

__device__ __forceinline__ unsigned encode_out(const u32x4 (&W)[16], unsigned cw, unsigned tail, char* dst, int lane) {
    u32x4 mk; unsigned s = 0;
#pragma unroll
    for (int j = 0; j < 4; ++j) {
        unsigned t[2] = {0u, 0u};
#pragma unroll
        for (int d = 15; d >= 0; --d) { const unsigned w = W[j * 4 + (d >> 2)][d & 3]; t[d >> 3] = lshl_or<2>(t[d >> 3], pk_min_u16(w, 0x00010001u)); if (j == 0) s = sum_u16(w, s); }
        mk[j] = perm(t[1], t[0], 0x05040100u) | (perm(t[1], t[0], 0x07060302u) << 1);
    }
    unsigned end[8], run = cw;
    unsigned base[16];
#pragma unroll
    for (int k = 0; k < 8; ++k) {
        const unsigned m = mk[k >> 1] >> (16 * (k & 1));
        end[k] = scan_add(((unsigned)__builtin_popcount(m & 0xffu) | (unsigned)__builtin_popcount(m & 0xff00u) << 16) * 2u);
    }
#pragma unroll
    for (int k = 0; k < 8; ++k) { const unsigned t = last_lane(end[k]); base[2 * k] = run; run += t & 0xffffu; base[2 * k + 1] = run; run += t >> 16; }
    const unsigned n = (run - cw) >> 1;
    const int nch = (int)(n + 15u) >> 4;
    const unsigned T = top_of(4u * last_lane(scan_add(s)), max(base[4] - cw, 1u));
    {
        const unsigned pad = (((T - 0x200u) >> 1) & 0x7fffu) * 0x10001u, at = cw + (min(n, 8191u) >> 10) * 2048u + lane * 16;
        *ldsp<u32x4>(at) = (u32x4){pad, pad, pad, pad}; *ldsp<u32x4>(at + 1024) = (u32x4){pad, pad, pad, pad};
    }
    fence();
#pragma unroll
    for (int i = 0; i < 16; ++i) {
        unsigned a = base[i] + ((i & 1) ? end[i >> 1] >> 16 : end[i >> 1] & 0xffffu);
#pragma unroll
        for (int e = 7; e >= 0; --e) {
            const unsigned nz = (unsigned)__builtin_amdgcn_sbfe((int)mk[i >> 2], (i & 3) * 8 + e, 1);
            a = lshl_add<1>(nz, a);
            if (e & 1) asm volatile("ds_write_b16_d16_hi %0, %1" :: "v"(a & nz), "v"(W[i][e >> 1]) : "memory");
            else asm volatile("ds_write_b16 %0, %1" :: "v"(a & nz), "v"(W[i][e >> 1]) : "memory");
        }
    }
    wait_lds();
    return encode_values<true>([&](int it, u32x4& x, u32x4& y) { x = *ldsp<u32x4>(cw + (it * 64 + lane) * 32); y = *ldsp<u32x4>(cw + (it * 64 + lane) * 32 + 16); },
                               []() {}, nch, T, mk, tail, dst, lane);
}

template <bool OUT, class PUT>
__device__ __forceinline__ void decode_values(PUT put, unsigned info, unsigned cw, unsigned esc, int lane) {
    const int nch = OUT ? (int)(info & 1023u) : 512;
    const unsigned T2 = ((info >> 19) & 0xfffu) * 0x100010u;
    u32x4 L[8]; unsigned h[8];
#pragma unroll
    for (int it = 0; it < 8; ++it) if (!OUT || it * 64 < nch) {
        L[it] = *ldsp<u32x4>(cw + (OUT ? 1024 : 0) + (it * 64 + lane) * 16);
        h[it] = OUT ? *ldsp<u16>(cw + cls_off<OUT>(nch) + (it * 64 + lane) * 2) : *ldsp<unsigned>(cw + cls_off<OUT>(nch) + (it * 64 + lane) * 4);
    }
    {
        const unsigned str = cw + cls_off<OUT>(nch) + cls_bytes<OUT>(nch), nw = (info >> 10) & 511u, bits = nw * 32u - 2u;
        const unsigned lg = nw > 248u ? 9u : nw > 124u ? 8u : 7u, S = (bits + (1u << lg) - 1u) >> lg, msk = ((1u << S) - 1u) << (31u - S);
        unsigned at = esc + 4u, before = 1u;
        for (unsigned s0 = 0; s0 < (1u << lg); s0 += 128u) {
            unsigned x[2], a[2], l[2];
#pragma unroll
            for (int k = 0; k < 2; ++k) {
                const unsigned o = (s0 + lane * 2u + k) * S, i = min(o >> 5, nw - 1u);
                const unsigned d0 = *ldsp<unsigned>(str + i * 4u), d1 = *ldsp<unsigned>(str + i * 4u + 4u);
                a[k] = o + 1u;
                x[k] = __builtin_amdgcn_alignbit(d0, i + 1u < nw ? d1 : 0u, ~o) & msk;
                if (o >= bits) x[k] = 0u;
                l[k] = x[k] ? a[k] + 31u - (unsigned)__builtin_ctz(x[k]) : 0u;
            }
            const unsigned c0 = __builtin_popcount(x[0]), c = c0 + __builtin_popcount(x[1]), ci = scan_add(c), li = scan_max(max(l[0], l[1]));
            const unsigned prev = max(before, dpp0<0x138, 0xf>(li));
            unsigned p = at + ci - c;
#pragma unroll
            for (int k = 0; k < 2; ++k) {
                unsigned w = x[k], carry = a[k] - (k ? max(prev, l[0]) : prev);
                while (w) {
                    const unsigned z = __builtin_clz(w), len = z + carry;
                    w = (w << z) + 0x80000000u; carry = 0u;
                    *ldsp<unsigned char>(p) = (unsigned char)(len + perm(0x020202fcu, 0x03030300u, min(len, 7u)));
                    ++p;
                }
            }
            at = min(at + last_lane(ci), esc + 4u + ESC_CAP); before = max(before, last_lane(li));
        }
    }
    u32x4 C[8], M[8]; unsigned cnt = 0;
#pragma unroll
    for (int it = 0; it < 8; ++it) if (!OUT || it * 64 < nch) {
        if (OUT) { h[it] = (h[it] | (h[it] << 12)) << 1; if (it * 64 + lane >= nch) h[it] = ~0u; }
#pragma unroll
        for (int q = 0; q < 4; ++q) {
            C[it][q] = OUT ? (L[it][q] & 0x01010101u) | ((h[it] >> q) & 0x02020202u) : (h[it] >> (2 * q)) & 0x03030303u;
            M[it][q] = perm(0u, 0x000000ffu, C[it][q]);
            cnt += __builtin_popcount(M[it][q]);
        }
    }
    cnt >>= 3;
    unsigned p = esc + 3u + scan_add(cnt) - cnt;
    fence();
#pragma unroll
    for (int it = 0; it < 8; ++it) if (!OUT || it * 64 < nch) {
        u32x4 x, y;
#pragma unroll
        for (int q = 0; q < 4; ++q) {
            const unsigned f = (M[it][q] & 0x01010101u) + (p & 3u), t = lshl_add<8>(f, f), sel = lshl_add<16>(t, t), ad = p & ~3u;
            const unsigned d0 = *ldsp<unsigned>(ad), d1 = *ldsp<unsigned>(ad + 4u);
            p = ad + (sel >> 24);
            const unsigned E = (perm(d1, d0, sel) & M[it][q]) | C[it][q], lo = OUT ? L[it][q] & 0xfefefefeu : L[it][q];
            const unsigned q0 = pk_rsub(T2, perm(E, lo, 0x05010400u)), q1 = pk_rsub(T2, perm(E, lo, 0x07030602u));
            const unsigned w0 = __builtin_amdgcn_alignbit(q0, q0, 1), w1 = __builtin_amdgcn_alignbit(q1, q1, 1);
            if (q < 2) { x[2 * q] = w0; x[2 * q + 1] = w1; } else { y[2 * q - 4] = w0; y[2 * q - 3] = w1; }
        }
        put(it, x, y);
    }
}

__device__ __forceinline__ void decode_in(unsigned info, unsigned cw, unsigned esc, char* dst, int lane) {
    decode_values<false>([&](int it, const u32x4& x, const u32x4& y) { st_wt(dst + it * 2048, x); st_wt(dst + it * 2048 + 1024, y); }, info, cw, esc, lane);
}
__device__ __forceinline__ void make_table(int n) {
    unsigned s[4], c = 0;
#pragma unroll
    for (int e = 0; e < 4; ++e) { const bool nz = (n >> e) & 1; s[e] = nz ? 0x0100u + c * 0x0202u : 0x0c0cu; c += nz; }
    *ldsp<u32x2>(LDS_TAB + n * 8) = (u32x2){s[0] | s[1] << 16, s[2] | s[3] << 16};
}
__device__ __forceinline__ void decode_out(unsigned info, unsigned cw, unsigned esc, char* outp, int lane) {
    const u32x4 mk = *ldsp<u32x4>(cw + lane * 16);
    decode_values<true>([&](int it, const u32x4& x, const u32x4& y) { *ldsp<u32x4>(cw + (it * 64 + lane) * 32) = x; *ldsp<u32x4>(cw + (it * 64 + lane) * 32 + 16) = y; }, info, cw, esc, lane);
    unsigned nl[16], nh[16], cl[16], st[8], base[16], run = cw;
#pragma unroll
    for (int k = 0; k < 8; ++k) {
        unsigned c[2];
#pragma unroll
        for (int j = 0; j < 2; ++j) {
            const int i = 2 * k + j;
            nl[i] = (((mk[i >> 2] & 0x0f0f0f0fu) << 3) >> (8 * (i & 3))) & 0xffu; nh[i] = (((mk[i >> 2] & 0xf0f0f0f0u) >> 1) >> (8 * (i & 3))) & 0xffu;
            cl[i] = __builtin_popcount(nl[i]); c[j] = cl[i] + __builtin_popcount(nh[i]);
        }
        const unsigned own = (c[0] | c[1] << 16) * 2u, inc = scan_add(own), t = last_lane(inc);
        st[k] = inc - own;
        base[2 * k] = run; run += t & 0xffffu; base[2 * k + 1] = run; run += t >> 16;
    }
    fence();
#pragma unroll
    for (int i = 0; i < 16; ++i) {
        const unsigned a0 = base[i] + ((i & 1) ? st[i >> 1] >> 16 : st[i >> 1] & 0xffffu), a1 = a0 + cl[i] * 2u;
        u32x4 o;
#pragma unroll
        for (int j = 0; j < 2; ++j) {
            const unsigned a = j ? a1 : a0, ad = a & ~3u;
            const unsigned d0 = *ldsp<unsigned>(ad), d1 = *ldsp<unsigned>(ad + 4u), d2 = *ldsp<unsigned>(ad + 8u);
            const u32x2 sel = *ldsp<u32x2>(LDS_TAB + (j ? nh[i] : nl[i]));
            const unsigned t0 = __builtin_amdgcn_alignbyte(d1, d0, a), t1 = __builtin_amdgcn_alignbyte(d2, d1, a);
            o[2 * j] = perm(t1, t0, sel.x); o[2 * j + 1] = perm(t1, t0, sel.y);
        }
        st_wt(outp + OUT_OFF(i), o);
    }
}

__device__ __forceinline__ void in_job(const Args& a, char* myws, int dest, int g, int kt, int j, int lane, unsigned seen, unsigned tail, bool push DBG_ARGS) {
    const int panel = g * 8 + ((j & 15) >> 1), half = j & 1, rank = a.rank;
    char* pq = a.ws[j < 16 ? (rank + 1 + lane) & 7 : dest];
    u32x4 v[16];
    if (j < 16) {
        const int sb = lane * 16, swz = sb ^ (((sb >> 9) & 1) << 5);
        const char* src = a.x + (size_t)(panel * 256 + half * 128 + (swz >> 6)) * (KS * 2) + kt * 128 + (swz & 63);
#pragma unroll
        for (int i = 0; i < 16; ++i) v[i] = *(const u32x4*)(src + (i >> 1) * 16 * (KS * 2) + (i & 1) * 64);
        __builtin_amdgcn_sched_barrier(0); wait_vm0();
    } else {
        const char* src = a.w + (size_t)(kt * 64 + (lane & 3) * 8) * (GN * 2) + (size_t)(dest * NB + panel * 256 + half * 128 + (lane >> 2) * 8) * 2;
        u32x4 r[16];
#pragma unroll
        for (int i = 0; i < 16; ++i) r[i] = *(const u32x4*)(src + (size_t)((i >> 3) * 32 + (i & 7)) * (GN * 2));
        __builtin_amdgcn_sched_barrier(0); wait_vm0();
#pragma unroll
        for (int kh = 0; kh < 2; ++kh)
#pragma unroll
            for (int i = 0; i < 8; ++i)
#pragma unroll
                for (int d = 0; d < 4; ++d) v[kh * 8 + i][d] = __builtin_amdgcn_perm(r[kh * 8 + 2 * d + 1][i >> 1], r[kh * 8 + 2 * d][i >> 1], (i & 1) ? 0x07060302u : 0x05040100u);
    }
    asm volatile("" : "+v"(pq));
    asm volatile("" : "+v"(seen));
    STAMPD(29);
    const unsigned call = seen + 1u;
    const bool img = dest == rank || push, wire = j < 16 || !img;
    char* const iws = j < 16 ? myws : pq;
    const int t = slot_of(rank, dest, kt);
    if (img) {
        char* dst = iws + img_off(g, j, t) + lane * 16;
#pragma unroll
        for (int i = 0; i < 16; ++i) st_wt(dst + i * 1024, v[i]);
    }
    auto done = [&]() { STAMPD(30); if (img) { wait_vm0(); if (lane == 0) __hip_atomic_store((GAS unsigned*)(iws + flag_off(g, j, t)), call, __ATOMIC_RELAXED, __HIP_MEMORY_SCOPE_SYSTEM); } };
    if (!wire) { done(); return; }
    char* dst = myws + (j < 16 ? OFF_XW : OFF_WW + (size_t)dest * (256 * 16384)) + (size_t)(g * 128 + kt * 16 + (j & 15)) * 16384 + lane * 16;
    const unsigned info = encode_in(v, tail, dst, lane, done);
    if (info & INFO_RAW) {
#pragma unroll
        for (int i = 0; i < 16; ++i) st_wt(dst + i * 1024, v[i]);
    }
    STAMPD(31);
    wait_vm0();
    if (lane < (j < 16 ? 7 : 1))
        __hip_atomic_store((GAS u64*)(pq + OFF_Q) + rank * NPOS + g * 256 + kt * 32 + j, ((u64)call << 32) | info, __ATOMIC_RELAXED, __HIP_MEMORY_SCOPE_SYSTEM);
}

__device__ __forceinline__ bool expired(unsigned* tmo, unsigned call, u64 t0) {
    if (__builtin_amdgcn_readfirstlane(__hip_atomic_load(tmo, __ATOMIC_RELAXED, __HIP_MEMORY_SCOPE_AGENT)) == call || now() - t0 > DEADLINE) {
        __hip_atomic_store(tmo, call, __ATOMIC_RELAXED, __HIP_MEMORY_SCOPE_AGENT);
        return true;
    }
    return false;
}

__device__ __forceinline__ void tile_of(int ti, int xcd, int& pm, int& pn) { pm = (ti >> 4) * 8 + (xcd & 1) * 4 + (ti & 3); pn = ((ti >> 3) & 1) * 8 + (xcd >> 1) * 2 + ((ti >> 2) & 1); }
__device__ __forceinline__ size_t out_off(int src, int pm, int pn, int w, int lane) { return (size_t)(pm * 256 + (w >> 2) * 64 + (lane >> 3)) * (GN * 2) + (size_t)(src * NB + pn * 256 + (w & 3) * 64) * 2 + (lane & 7) * 16; }

__global__ __launch_bounds__(512, 2) void gemm_ar_kernel(const Args a) {
    extern __shared__ __attribute__((aligned(16))) unsigned char shm[];
    LAS unsigned char* lds = (LAS unsigned char*)shm;
    const int tid = threadIdx.x, wid = __builtin_amdgcn_readfirstlane(tid >> 6), lane = tid & 63;
    const int b = blockIdx.x, xcd = b & 7, idx = b >> 3, rank = a.rank;
    char* const myws = a.myws;
    char* const myout = a.myout;
    const u64 t0 = now();
    STAMP(0);
    unsigned* const ctr = (unsigned*)(myws + OFF_CTR) + b * 32 + wid;
    unsigned* const tmo = (unsigned*)(myws + OFF_TMO);
    const unsigned seen = __hip_atomic_load(ctr, __ATOMIC_RELAXED, __HIP_MEMORY_SCOPE_AGENT);
    const unsigned cw = LDS_CW + wid * C_WAVE, tl = LDS_TL + wid * C_TAIL;

    const int pk = (idx - NG + 7 * 8 + xcd) % 7, peer = (rank + 1 + pk) & 7, ord = xcd * 2 + (idx >= NG + 7);
    if (idx >= NG) { if (tid < 2) *ldsp<unsigned>(LDS_CTL + tid * 4) = tid ? 0u : 8u; if (tid < 16) make_table(tid); wait_lds(); bar(); }
    {
        int q, qe, delay;
        if (idx >= NG) { q = 7 - wid; qe = 1; delay = 0; }
        else if (idx < 8) { q = wid; qe = 4; delay = wid * DLY1; }
        else { q = wid * 80 + (idx - 8) * 8 + xcd; qe = 1936; delay = wid * DLY2; }
        for (int i = 0; i < delay; ++i) __builtin_amdgcn_s_sleep(8);
        for (; q < qe; q += 640) {
            int r;
            if (idx >= NG) r = (pk + 1) * 512 + 16 + ord;
            else if (idx < 8) r = wid * 64 + (idx ^ 2) * 8 + xcd;
            else {
                const int kt = q < 784 ? 1 + q / 112 : 8 + (q - 784) / 144, rem = q < 784 ? q % 112 : (q - 784) % 144 - 32;
                r = kt * 32 + (rem < 0 ? rem + 32 : (rem % 7 + 1) * 512 + 16 + rem / 7);
            }
            in_job(a, myws, (rank + (r >> 9)) & 7, (r >> 8) & 1, (r >> 5) & 7, r & 31, lane, seen, tl, idx >= NG DBG_PASS);
        }
    }
    const unsigned call = __builtin_amdgcn_readfirstlane(seen) + 1u;
    if (lane == 0) __hip_atomic_store(ctr, call, __ATOMIC_RELAXED, __HIP_MEMORY_SCOPE_AGENT);
    STAMP(1);

    if (idx >= NG) {
        const char* const pws = a.ws[peer];
        const u64* const queue = (const u64*)(myws + OFF_Q) + peer * NPOS;
        unsigned* const tq = (unsigned*)(myws + OFF_TICK) + pk * 64;
        const unsigned tbase = (call - 1u) * (NPOS - 16) - 144u;
        const unsigned ctl = LDS_CTL;
        unsigned pos = (wid + (wid > 0)) * 16 + ord;
        bool bad = false;
        while (pos < (unsigned)NPOS) {
            unsigned info;
            for (;;) {
                const u64 e = __hip_atomic_load(queue + pos, __ATOMIC_RELAXED, __HIP_MEMORY_SCOPE_SYSTEM);
                info = __builtin_amdgcn_readfirstlane((unsigned)e);
                if (__builtin_amdgcn_readfirstlane((unsigned)(e >> 32)) >= call) break;
                if (expired(tmo, call, t0)) { bad = true; break; }
                __builtin_amdgcn_s_sleep(4);
            }
            if (bad) break;
            LAP(24); STAMP1(2);
            if ((info & 1023u) > 512u || ((info >> 10) & 511u) - 1u >= (unsigned)NW_CAP) info = INFO_RAW;
            const int g = pos >> 8, kt = (pos >> 5) & 7, j = pos & 31;
            const char* const src = pws + (pos >= NIN ? OFF_OW + (size_t)(pos - NIN) * 16384
                                           : (j < 16 ? OFF_XW : OFF_WW + (size_t)rank * (256 * 16384)) + (size_t)(g * 128 + kt * 16 + (j & 15)) * 16384) + lane * 16;
            const int wire = (info & INFO_RAW) ? 16384 : pos >= NIN ? wire_bytes<true>(info) : wire_bytes<false>(info);
            unsigned my = wid;
            if (lane == 0 && pos >= 144u) asm volatile("ds_add_rtn_u32 %0, %1, %2\n\ts_waitcnt lgkmcnt(0)" : "+v"(my) : "v"(ctl), "v"(1u) : "memory");
            my = __builtin_amdgcn_readfirstlane(my);
            for (;;) {
                unsigned rel;
                asm volatile("ds_read_b32 %0, %1 offset:4\n\ts_waitcnt lgkmcnt(0)" : "=v"(rel) : "v"(ctl) : "memory");
                if ((int)(__builtin_amdgcn_readfirstlane(rel) + TOK - my) > 0) break;
                if (now() - t0 > DEADLINE) break;
                __builtin_amdgcn_s_sleep(1);
            }
            LAP(25); STAMP1(7);
            for (int o = 0; o < wire; o += 1024) {
                if (o + lane * 16 < wire) __builtin_amdgcn_global_load_lds((const unsigned*)(src + o), ldsp<unsigned>(cw + o), 16, 0, 17);
                wait_vm<WIN - 1>();
            }
            wait_vm<REL>();
            if (lane == 0) asm volatile("ds_add_u32 %0, %1 offset:4" :: "v"(ctl), "v"(1u) : "memory");
            wait_vm0();
            LAP(26); STAMP1(3);
            int t = 0;
            char* dst;
            if (pos >= NIN) { int pm, pn; tile_of((pos - NIN) >> 6, ((pos - NIN) >> 3) & 7, pm, pn); dst = myout + out_off(peer, pm, pn, pos & 7, lane); }
            else { t = slot_of(peer, rank, kt); dst = myws + img_off(g, j, t) + lane * 16; }
            if (info & INFO_RAW) {
                u32x4 v[16];
#pragma unroll
                for (int i = 0; i < 16; ++i) v[i] = *ldsp<u32x4>(cw + i * 1024 + lane * 16);
#pragma unroll
                for (int i = 0; i < 16; ++i) st_wt(dst + (pos >= NIN ? OUT_OFF(i) : (size_t)i * 1024), v[i]);
            } else if (pos >= NIN) decode_out(info, cw, tl, dst, lane);
            else decode_in(info, cw, tl, dst, lane);
            LAP(27); STAMP1(4);
            unsigned nx = 0;
            if (lane == 0) nx = __hip_atomic_fetch_add(tq, 1u, __ATOMIC_RELAXED, __HIP_MEMORY_SCOPE_AGENT);
            wait_vm0();
            if (pos < NIN && lane == 0) __hip_atomic_store((unsigned*)(myws + flag_off(g, j, t)), call, __ATOMIC_RELAXED, __HIP_MEMORY_SCOPE_SYSTEM);
            LAP(28); STAMP(8 + (pos >> 8)); STAMP1(6);
            pos = __builtin_amdgcn_readfirstlane(nx) - tbase;
        }
        if (bad && lane == 0) *(GAS u16*)(myout + (size_t)rank * (NB * 2)) = 0x7fc0;
        STAMP(5);
        STAMP_FLUSH();
        return;
    }

    const int wr = wid >> 2, wc = wid & 3, fr = lane & 15, fq = lane >> 4;
    bool bad = false;
    unsigned voffA[2], voffB[2];
#pragma unroll
    for (int i = 0; i < 2; ++i) {
        int R, C; stage_rc(tid * 16 + i * 8192, R, C);
        voffA[i] = tid * 16 + i * 8192;
        const int w_ = R >> 5, n_ = (R >> 4) & 1, q_ = (R & 15) >> 2, j_ = R & 3;
        voffB[i] = (w_ >> 1) * 16384 + (C >> 5) * 8192 + (4 * n_ + j_) * 1024 + ((w_ & 1) * 8 + q_) * 64 + ((C >> 3) & 3) * 16;
    }
    constexpr size_t kstep = 32768, hstepA = 16384, hstepB = 256;
    const unsigned ldsw = (unsigned)wid * 1024u;
    const int aoff = lds_byte(wr * 64 + fr, fq * 8), boff = lds_byte(wc * 32 + fr, fq * 8);
#define G_SA(b, h) (((b) * 2 + (h)) * HTB)
#define G_SB(b, h) ((4 + (b) * 2 + (h)) * HTB)
#define G_STAGE(bufoff, gbase, voff) do { _Pragma("unroll") for (int _i = 0; _i < 2; ++_i) \
        __builtin_amdgcn_global_load_lds((const unsigned*)((const char*)(gbase) + (voff)[_i]), (LAS unsigned*)(lds + (bufoff) + ldsw + _i * 8192), 16, 0, 17); } while (0)
#define G_LDA(dst, b, h) do { _Pragma("unroll") for (int m = 0; m < 4; ++m) _Pragma("unroll") for (int k = 0; k < 2; ++k) dst[m][k] = *(const LAS bf16x8*)(lds + G_SA(b, h) + aoff + m * 2048 + k * 1024); } while (0)
#define G_LDB(dst, b, h) do { _Pragma("unroll") for (int n = 0; n < 2; ++n) _Pragma("unroll") for (int k = 0; k < 2; ++k) dst[n][k] = *(const LAS bf16x8*)(lds + G_SB(b, h) + boff + n * 2048 + k * 1024); } while (0)
#define G_MMA(ai, bj, At, Bt) do { __builtin_amdgcn_s_setprio(1); _Pragma("unroll") for (int m = 0; m < 4; ++m) _Pragma("unroll") for (int n = 0; n < 2; ++n) _Pragma("unroll") for (int k = 0; k < 2; ++k) \
        acc[ai][bj][m][n] = __builtin_amdgcn_mfma_f32_16x16x32_bf16(Bt[n][k], At[m][k], acc[ai][bj][m][n], 0, 0, 0); __builtin_amdgcn_s_setprio(0); } while (0)
#define G_WAIT_V(n) asm volatile("s_waitcnt vmcnt(" #n ")" ::: "memory")
#define G_WAIT_L(n) asm volatile("s_waitcnt lgkmcnt(" #n ")" ::: "memory")
#define G_BAR __builtin_amdgcn_s_barrier()
#define G_SCHED __builtin_amdgcn_sched_barrier(0)
    for (int ti = idx; ti < 32; ti += NG) {
    int pm, pn; tile_of(ti, xcd, pm, pn);
    const char* const fX = myws + OFF_INF + (size_t)pm * 512 + lane * 8;
    const char* const fW = myws + OFF_INF + (size_t)(16 + pn) * 512 + lane * 8;
    auto poll = [&](int need) -> int {
        for (;;) {
            u32x2 fa, fb;
            asm volatile("global_load_dwordx2 %0, %2, off sc0 sc1\n\tglobal_load_dwordx2 %1, %3, off sc0 sc1\n\ts_waitcnt vmcnt(0)"
                         : "=&v"(fa), "=&v"(fb) : "v"(fX), "v"(fW) : "memory");
            const u64 miss = __builtin_amdgcn_ballot_w64(fa.x < call || fa.y < call || fb.x < call || fb.y < call);
            const int upto = miss ? __builtin_ctzll(miss) : 64;
            if (upto >= need) return upto;
            if (expired(tmo, call, t0)) { bad = true; return 64; }
            __builtin_amdgcn_s_sleep(4);
        }
    };
    f32x4 acc[2][2][4][2];
#pragma unroll
    for (int x = 0; x < 2; ++x)
#pragma unroll
        for (int y = 0; y < 2; ++y)
#pragma unroll
            for (int m = 0; m < 4; ++m)
#pragma unroll
                for (int n = 0; n < 2; ++n) acc[x][y][m][n] = (f32x4){0.f, 0.f, 0.f, 0.f};
    bf16x8 At[4][2], B0[2][2], B1[2][2];
    const char* const cA = myws + OFF_XI + (size_t)pm * 64 * kstep;
    const char* const cB = myws + OFF_WI + (size_t)pn * 64 * kstep;
    int ready = poll(2);
    STAMP1(2);
    wait_vm0();
    G_STAGE(G_SB(0, 0), cB, voffB); G_STAGE(G_SA(0, 0), cA, voffA); G_STAGE(G_SB(0, 1), cB + hstepB, voffB); G_STAGE(G_SA(0, 1), cA + hstepA, voffA);
    if (wr == 1) G_BAR;
    G_WAIT_V(4); G_BAR;
    G_STAGE(G_SB(1, 0), cB + kstep, voffB); G_STAGE(G_SA(1, 0), cA + kstep, voffA); G_STAGE(G_SB(1, 1), cB + hstepB + kstep, voffB);
    G_WAIT_V(6); G_BAR;
    for (int t = 0; t < 64; t += 2) {
        const bool last = (t == 62);
        if (!last && ready < t + 4) ready = poll(t + 4);
        const char* a1 = cA + (size_t)(t + 1) * kstep;
        const char* a2 = last ? cA : cA + (size_t)(t + 2) * kstep; const char* b2 = last ? cB : cB + (size_t)(t + 2) * kstep;
        const char* a3 = a2 + kstep; const char* b3 = b2 + kstep;
        G_LDB(B0, 0, 0); G_SCHED; G_LDA(At, 0, 0); G_STAGE(G_SA(1, 1), a1 + hstepA, voffA);
        G_WAIT_L(8); G_BAR; G_WAIT_L(0); G_MMA(0, 0, At, B0); G_BAR; G_SCHED;
        G_LDB(B1, 0, 1); G_STAGE(G_SB(0, 0), b2, voffB);
        G_BAR; G_WAIT_L(0); G_MMA(0, 1, At, B1); G_BAR;
        G_LDA(At, 0, 1); G_STAGE(G_SA(0, 0), a2, voffA);
        G_BAR; G_WAIT_L(0); G_MMA(1, 0, At, B0); G_BAR; G_SCHED;
        G_STAGE(G_SB(0, 1), b2 + hstepB, voffB);
        G_WAIT_V(6); G_BAR; G_MMA(1, 1, At, B1); G_BAR;
        G_LDB(B0, 1, 0); G_SCHED; G_LDA(At, 1, 0); G_STAGE(G_SA(0, 1), a2 + hstepA, voffA);
        G_WAIT_L(8); G_BAR; G_WAIT_L(0); G_MMA(0, 0, At, B0); G_BAR; G_SCHED;
        G_LDB(B1, 1, 1); G_STAGE(G_SB(1, 0), b3, voffB);
        G_BAR; G_WAIT_L(0); G_MMA(0, 1, At, B1); G_BAR;
        G_LDA(At, 1, 1); G_STAGE(G_SA(1, 0), a3, voffA);
        G_BAR; G_WAIT_L(0); G_MMA(1, 0, At, B0); G_BAR; G_SCHED;
        G_STAGE(G_SB(1, 1), b3 + hstepB, voffB);
        G_WAIT_V(6); G_BAR; G_MMA(1, 1, At, B1); G_BAR;
    }
    G_WAIT_V(0);
    if (wr == 0) G_BAR;
    G_BAR;
    STAMP(ti < NG ? 3 : 6);

    int ln = lane;
    asm volatile("" : "+v"(ln) :: "memory");
    char* pq = a.ws[(rank + 1 + (ln >> 3)) & 7];
    if (bad) acc[0][0][0][0][0] = __builtin_nanf("");
#pragma unroll
    for (int ai = 0; ai < 2; ++ai)
#pragma unroll
        for (int m = 0; m < 4; ++m)
#pragma unroll
            for (int bj = 0; bj < 2; ++bj) {
                const f32x4 v0 = acc[ai][bj][m][0], v1 = acc[ai][bj][m][1];
                u32x4 w; w[0] = relu_pk(cvt_pk_bf16(v0[0], v0[1])); w[1] = relu_pk(cvt_pk_bf16(v0[2], v0[3]));
                w[2] = relu_pk(cvt_pk_bf16(v1[0], v1[1])); w[3] = relu_pk(cvt_pk_bf16(v1[2], v1[3]));
                *ldsp<u32x4>(cw + (ai * 64 + m * 16 + (ln & 15)) * 128 + (((bj * 4 + (ln >> 4)) ^ (ln & 7)) << 4)) = w;
            }
    asm volatile("" ::: "memory");
    u32x4 W[16];
#pragma unroll
    for (int i = 0; i < 16; ++i) W[i] = *ldsp<u32x4>(cw + i * 1024 + (ln >> 3) * 128 + (((ln & 7) ^ (ln >> 3)) << 4));
    wait_lds(); wait_vm0();
    asm volatile("" : "+v"(pq));
    char* const outp = myout + out_off(rank, pm, pn, wid, ln);
#pragma unroll
    for (int i = 0; i < 16; ++i) st_wt(outp + OUT_OFF(i), W[i]);
    char* const slot = myws + OFF_OW + (size_t)((ti * 8 + xcd) * 8 + wid) * 16384 + ln * 16;
    const unsigned info = encode_out(W, cw, tl, slot, ln);
    if (info & INFO_RAW) {
#pragma unroll
        for (int i = 0; i < 16; ++i) st_wt(slot + i * 1024, W[i]);
    }
    if (ln == 0) *(LAS unsigned*)(lds + LDS_CTL + wid * 4) = info;
    wait_lds();
    wait_vm0();
    bar();
    if (wid == 0 && ln < 56) {
        const unsigned inf = *(const LAS unsigned*)(lds + LDS_CTL + (ln & 7) * 4);
        __hip_atomic_store((GAS u64*)(pq + OFF_Q) + rank * NPOS + NIN + (ti * 8 + xcd) * 8 + (ln & 7), ((u64)call << 32) | inf, __ATOMIC_RELAXED, __HIP_MEMORY_SCOPE_SYSTEM);
    }
    STAMP(ti < NG ? 4 : 7);
    }
    STAMP(5);
    STAMP_FLUSH();
}

extern "C" void kernel_launch(int rank, int nranks, ncclComm_t comm,
                              hipStream_t stream,
                              void* const* d_in, const int* in_sizes, int n_in,
                              void* d_out, int out_size,
                              const int* in_dtypes, int out_dtype,
                              void* const* d_out_all,
                              void* const* d_ws_all, size_t ws_bytes) {
    if (nranks != NR || (unsigned)rank >= (unsigned)NR || ws_bytes < WS_NEED || n_in < 2) { fprintf(stderr, "unsupported configuration\n"); return; }
    static bool attr[NR] = {};
    if (!attr[rank]) { HIP_CHECK(hipFuncSetAttribute((const void*)gemm_ar_kernel, hipFuncAttributeMaxDynamicSharedMemorySize, LDS_BYTES)); attr[rank] = true; }
    Args a;
    a.x = (const char*)d_in[0]; a.w = (const char*)d_in[1]; a.rank = rank;
    for (int p = 0; p < NR; ++p) a.ws[p] = (char*)d_ws_all[p];
    a.myws = a.ws[rank]; a.myout = (char*)d_out;
    hipLaunchKernelGGL(gemm_ar_kernel, dim3(256), dim3(512), LDS_BYTES, stream, a);
    HIP_CHECK(hipGetLastError());
}
```

```cpp
#include <hip/hip_runtime.h>
#include <rccl/rccl.h>
#include <cstdio>

#define HIP_CHECK(expr) do { hipError_t _e = (expr); if (_e != hipSuccess) { \
    fprintf(stderr, "HIP error %s at %s:%d\n", hipGetErrorString(_e), __FILE__, __LINE__); \
    return; } } while (0)

#define LAS __attribute__((address_space(3)))
#define GAS __attribute__((address_space(1)))
typedef unsigned short u16;
typedef unsigned long long u64;
typedef short bf16x8 __attribute__((ext_vector_type(8)));
typedef float f32x4 __attribute__((ext_vector_type(4)));
typedef unsigned u32x4 __attribute__((ext_vector_type(4)));
typedef unsigned u32x2 __attribute__((ext_vector_type(2)));

constexpr int NR = 8;
constexpr int GM = 4096, GN = 32768, KS = 512, NB = 4096;
constexpr int HTB = 16384;
constexpr int NG = 18;
constexpr int NIN = 512, NPOS = 2560;
constexpr int NQ = NPOS + 64;
constexpr unsigned LDS_TAB = 64, LDS_CW = 256, C_WAVE = 16384 + 128, LDS_TL = LDS_CW + 8 * C_WAVE, C_TAIL = 3584, ESC_CAP = 3560;
constexpr unsigned LDS_CTL = LDS_TL + 8 * C_TAIL + 64;
constexpr int LDS_BYTES = LDS_CTL + 64;
static_assert(LDS_TL >= 8 * HTB && LDS_CTL >= 9 * HTB && LDS_BYTES <= 163840, "LDS layout");
constexpr size_t OFF_CTR = 0;
constexpr size_t OFF_TMO = 32768;
constexpr size_t OFF_TICK = 32768 + 256;
constexpr size_t OFF_STAMP = 65536;
constexpr size_t OFF_INF = 512u << 10;
constexpr size_t OFF_Q = 1u << 20;
constexpr size_t OFF_XI = 4u << 20;
constexpr size_t OFF_WI = 36u << 20;
constexpr size_t OFF_XW = 68u << 20;
constexpr size_t OFF_WW = 72u << 20;
constexpr size_t OFF_OW = 104u << 20;
constexpr size_t WS_NEED = 136u << 20;
constexpr u64 DEADLINE = 50000000ull;

#define TOK 2
#define WIN 2
#define REL (WIN - 1)
#define DLY1 3
#define DLY2 5
#define RAWN 0
#define RWIN 4
#define FREEN 0
#define POLL_SLEEP 6
#define DLY0 4
#define EARLY_AT 3
#define DLY3 0
#define POLL_FAST 1
#define WIRE_ALIGN 64
struct Args { const char* x; const char* w; char* myws; char* myout; int rank; char* ws[NR]; };

__device__ __forceinline__ int lds_byte(int r, int c) { const int st = (r >> 4) * 2 + (c >> 5), rr = r & 15, cc = c & 31, ob = rr * 64 + cc * 2; return st * 1024 + (ob ^ (((ob >> 9) & 1) << 5)); }
__device__ __forceinline__ void stage_rc(int b, int& R, int& C) { const int st = b / 1024, sb = b % 1024, swz = sb ^ (((sb >> 9) & 1) << 5); R = (st >> 1) * 16 + swz / 64; C = (st & 1) * 32 + (swz % 64) / 2; }
__device__ __forceinline__ unsigned cvt_pk_bf16(float lo, float hi) { unsigned r; asm volatile("v_cvt_pk_bf16_f32 %0, %1, %2" : "=v"(r) : "v"(lo), "v"(hi)); return r; }
__device__ __forceinline__ unsigned relu_pk(unsigned v) { unsigned r; asm("v_pk_max_i16 %0, %1, 0" : "=v"(r) : "v"(v)); return r; }
__device__ __forceinline__ void st_wt(char* p, u32x4 v) { asm volatile("global_store_dwordx4 %0, %1, off sc0 sc1\n\ts_nop 1" :: "v"(p), "v"(v) : "memory"); }
__device__ __forceinline__ void wait_vm0() { __builtin_amdgcn_s_waitcnt(0x0F70); asm volatile("s_waitcnt vmcnt(0)" ::: "memory"); }
template <int N> __device__ __forceinline__ void wait_vm() { asm volatile("s_waitcnt vmcnt(%0)" :: "n"(N) : "memory"); }
__device__ __forceinline__ void bar() { asm volatile("" ::: "memory"); __builtin_amdgcn_s_barrier(); asm volatile("" ::: "memory"); }
__device__ __forceinline__ u64 now() { return __builtin_amdgcn_s_memrealtime(); }

#define STAMP(i) do { } while (0)
#define STAMP1(i) do { } while (0)
#define STAMPD(i) do { } while (0)
#define DBG_ARGS
#define DBG_PASS
#define LAP(i) do { } while (0)
#define STAMP_FLUSH() do { } while (0)

__device__ __forceinline__ int slot_of(int s, int p, int kt) { return s == p ? kt : 8 + 7 * kt + ((s - p - 1) & 7); }
__device__ __forceinline__ size_t img_off(int g, int j, int t) { return (j < 16 ? OFF_XI : OFF_WI) + (size_t)((g * 8 + ((j & 15) >> 1)) * 64 + t) * 32768 + (j & 1) * 16384; }
__device__ __forceinline__ size_t flag_off(int g, int j, int t) { return OFF_INF + (size_t)((((j >> 4) * 16 + g * 8 + ((j & 15) >> 1)) * 64 + t) * 2 + (j & 1)) * 4; }

#define OUT_OFF(i) ((size_t)(((i) >> 3) * 128 + ((i) & 7) * 8) * (GN * 2))
constexpr unsigned INFO_RAW = 1u << 31;
constexpr int NW_CAP = 384;
template <bool OUT> __device__ __forceinline__ int cls_off(int nch) { return OUT ? 1024 + nch * 16 : 8192; }
template <bool OUT> __device__ __forceinline__ int cls_bytes(int nch) { return OUT ? (nch * 2 + 3) & ~3 : 2048; }
template <bool OUT> __device__ __forceinline__ int wire_bytes(unsigned info) {
    const int nch = info & 1023; return (cls_off<OUT>(nch) + cls_bytes<OUT>(nch) + (int)((info >> 10) & 511) * 4 + (WIRE_ALIGN - 1)) & ~(WIRE_ALIGN - 1); }
__device__ __forceinline__ unsigned pk_min_u16(unsigned x, unsigned y) { unsigned r; asm("v_pk_min_u16 %0, %1, %2" : "=v"(r) : "v"(x), "s"(y)); return r; }
__device__ __forceinline__ unsigned pk_rsub(unsigned s, unsigned y) { unsigned r; asm("v_pk_sub_u16 %0, %1, %2" : "=v"(r) : "s"(s), "v"(y)); return r; }
__device__ __forceinline__ unsigned sum_u16(unsigned x, unsigned acc) { unsigned r; asm("v_sad_u16 %0, %1, 0, %2" : "=v"(r) : "v"(x), "v"(acc)); return r; }
template <int S> __device__ __forceinline__ unsigned lshl_add(unsigned x, unsigned y) { unsigned r; asm("v_lshl_add_u32 %0, %1, %2, %3" : "=v"(r) : "v"(x), "n"(S), "v"(y)); return r; }
template <int S> __device__ __forceinline__ unsigned lshl_or(unsigned x, unsigned y) { unsigned r; asm("v_lshl_or_b32 %0, %1, %2, %3" : "=v"(r) : "v"(x), "n"(S), "v"(y)); return r; }
__device__ __forceinline__ unsigned perm(unsigned hi, unsigned lo, unsigned sel) { return __builtin_amdgcn_perm(hi, lo, sel); }
__device__ __forceinline__ void wait_lds() { asm volatile("s_waitcnt lgkmcnt(0)" ::: "memory"); }
__device__ __forceinline__ void fence() { asm volatile("" ::: "memory"); }
template <int C, int RM> __device__ __forceinline__ unsigned dpp0(unsigned x) { return (unsigned)__builtin_amdgcn_update_dpp(0, (int)x, C, RM, 0xf, false); }
__device__ __forceinline__ unsigned scan_add(unsigned x) {
    x += dpp0<0x111, 0xf>(x); x += dpp0<0x112, 0xf>(x); x += dpp0<0x114, 0xf>(x); x += dpp0<0x118, 0xf>(x); x += dpp0<0x142, 0xa>(x); x += dpp0<0x143, 0xc>(x); return x; }
__device__ __forceinline__ unsigned scan_max(unsigned x) {
    x = max(x, dpp0<0x111, 0xf>(x)); x = max(x, dpp0<0x112, 0xf>(x)); x = max(x, dpp0<0x114, 0xf>(x)); x = max(x, dpp0<0x118, 0xf>(x)); x = max(x, dpp0<0x142, 0xa>(x)); x = max(x, dpp0<0x143, 0xc>(x)); return x; }
__device__ __forceinline__ unsigned last_lane(unsigned x) { return (unsigned)__builtin_amdgcn_readlane((int)x, 63); }
template <class T> __device__ __forceinline__ LAS T* ldsp(unsigned a) { return (LAS T*)(size_t)a; }

template <bool OUT, class GET, class MID>
__device__ __forceinline__ unsigned encode_values(GET get, MID mid, int nch, unsigned T, const u32x4& mk, unsigned tail, char* dst, int lane) {
    const unsigned T2 = T * 0x10001u;
    u32x4 R0[8], R1[8];
    if (!OUT) {
#pragma unroll
        for (int it = 0; it < 8; ++it) {
            u32x4 x, y, lo; get(it, x, y);
#pragma unroll
            for (int q = 0; q < 4; ++q) {
                const unsigned w0 = q < 2 ? x[2 * q] : y[2 * q - 4], w1 = q < 2 ? x[2 * q + 1] : y[2 * q - 3];
                R0[it][q] = pk_rsub(T2, __builtin_amdgcn_alignbit(w0, w0, 31)); R1[it][q] = pk_rsub(T2, __builtin_amdgcn_alignbit(w1, w1, 31));
                lo[q] = perm(R1[it][q], R0[it][q], 0x06040200u);
            }
            st_wt(dst + it * 1024, lo);
        }
    }
#pragma unroll
    for (int i = 0; i < 4; ++i) if (i < 3 || lane < 32) *ldsp<u32x4>(tail + i * 1024 + lane * 16) = (u32x4){0u, 0u, 0u, 0u};
    fence();
    u32x4 Lb[8], Ln[8]; unsigned tot = 0, direct = 0;
#pragma unroll
    for (int it = 0; it < 8; ++it) if (!OUT || it * 64 < nch) {
        u32x4 x, y; if (OUT) get(it, x, y);
        if (it == EARLY_AT) mid();
        unsigned h = 0, h0 = 0;
#pragma unroll
        for (int q = 0; q < 4; ++q) {
            const unsigned w0 = q < 2 ? x[2 * q] : y[2 * q - 4], w1 = q < 2 ? x[2 * q + 1] : y[2 * q - 3];
            const unsigned r0 = OUT ? pk_rsub(T2, w0 << 1) : R0[it][q], r1 = OUT ? pk_rsub(T2, w1 << 1) : R1[it][q];
            const unsigned ec = perm(pk_min_u16(r1, 0x07ff07ffu), pk_min_u16(r0, 0x07ff07ffu), 0x07050301u);
            Ln[it][q] = perm(r1, r0, 0x07050301u) + perm(0x01000000u, 0x00010207u, ec) - 0x03030303u;
            tot = __builtin_amdgcn_sad_u8(Ln[it][q], 0u, tot);
            if (OUT) { const unsigned b0 = perm(0u, 0x01000100u, ec); Lb[it][q] = perm(r1, r0, 0x06040200u) | b0; h0 |= b0 << q; h |= perm(0u, 0x01010000u, ec) << q; }
            else h |= perm(0u, 0x03020100u, ec) << (2 * q);
        }
        if (OUT) { direct += __builtin_popcount(h | h0); if (it * 64 + lane < nch) *ldsp<u16>(tail + (it * 64 + lane) * 2) = (u16)(h | (h >> 12)); }
        else { direct += __builtin_popcount((h | (h >> 1)) & 0x55555555u); *ldsp<unsigned>(tail + (it * 64 + lane) * 4) = h; }
    }
    const unsigned inc = scan_add(tot), nw = (last_lane(inc) + 33u) >> 5;
    const unsigned nesc = (OUT ? (unsigned)((nch + 63) >> 6) * 1024u : 8192u) - last_lane(scan_add(direct));
    if (nw > (unsigned)NW_CAP || nesc > (unsigned)ESC_CAP) return INFO_RAW;
    if (OUT) st_wt(dst, mk);
    unsigned pos = (tail + cls_bytes<OUT>(nch)) * 8u + 1u + inc - tot;
#pragma unroll
    for (int it = 0; it < 8; ++it) if (!OUT || it * 64 < nch) {
        if (OUT && it * 64 + lane < nch) st_wt(dst + 1024 + it * 1024, Lb[it]);
#pragma unroll
        for (int e = 0; e < 16; ++e) {
            pos += (Ln[it][e >> 2] >> (8 * (e & 3))) & 0xffu;
            asm volatile("ds_or_b32 %0, %1" :: "v"((pos >> 3) & ~3u), "v"(0x80000000u >> (pos & 31u)) : "memory");
        }
    }
    const unsigned info = (T >> 4) << 19 | nw << 10 | (OUT ? (unsigned)nch : 0u);
    const int rest = wire_bytes<OUT>(info) - cls_off<OUT>(nch);
    wait_lds();
#pragma unroll
    for (int i = 0; i < 4; ++i) if (i * 1024 < rest) { if (i * 1024 + lane * 16 < rest) st_wt(dst + cls_off<OUT>(nch) + i * 1024, *ldsp<u32x4>(tail + i * 1024 + lane * 16)); }
    return info;
}
__device__ __forceinline__ unsigned top_of(unsigned sum2, unsigned n) {
    return ((unsigned)__builtin_amdgcn_readfirstlane((int)((float)sum2 * __builtin_amdgcn_rcpf((float)n))) + 2u * (276u + 128u) + 8u) & 0xfff0u; }

template <class MID> __device__ __forceinline__ unsigned encode_in(const u32x4 (&v)[16], unsigned tail, char* dst, int lane, MID mid) {
    unsigned s = 0;
#pragma unroll
    for (int i = 0; i < 16; ++i) s = sum_u16(__builtin_amdgcn_alignbit(v[i][i & 3], v[i][i & 3], 31), s);
    return encode_values<false>([&](int it, u32x4& x, u32x4& y) { x = v[2 * it]; y = v[2 * it + 1]; }, mid, 512, top_of(last_lane(scan_add(s)), 2048u), v[0], tail, dst, lane);
}

__device__ __forceinline__ unsigned encode_out(const u32x4 (&W)[16], unsigned cw, unsigned tail, char* dst, int lane) {
    u32x4 mk; unsigned s = 0;
#pragma unroll
    for (int j = 0; j < 4; ++j) {
        unsigned t[2] = {0u, 0u};
#pragma unroll
        for (int d = 15; d >= 0; --d) { const unsigned w = W[j * 4 + (d >> 2)][d & 3]; t[d >> 3] = lshl_or<2>(t[d >> 3], pk_min_u16(w, 0x00010001u)); if (j == 0) s = sum_u16(w, s); }
        mk[j] = perm(t[1], t[0], 0x05040100u) | (perm(t[1], t[0], 0x07060302u) << 1);
    }
    unsigned end[8], run = cw;
    unsigned base[16];
#pragma unroll
    for (int k = 0; k < 8; ++k) {
        const unsigned m = mk[k >> 1] >> (16 * (k & 1));
        end[k] = scan_add(((unsigned)__builtin_popcount(m & 0xffu) | (unsigned)__builtin_popcount(m & 0xff00u) << 16) * 2u);
    }
#pragma unroll
    for (int k = 0; k < 8; ++k) { const unsigned t = last_lane(end[k]); base[2 * k] = run; run += t & 0xffffu; base[2 * k + 1] = run; run += t >> 16; }
    const unsigned n = (run - cw) >> 1;
    const int nch = (int)(n + 15u) >> 4;
    const unsigned T = top_of(4u * last_lane(scan_add(s)), max(base[4] - cw, 1u));
    {
        const unsigned pad = (((T - 0x200u) >> 1) & 0x7fffu) * 0x10001u, at = cw + (min(n, 8191u) >> 10) * 2048u + lane * 16;
        *ldsp<u32x4>(at) = (u32x4){pad, pad, pad, pad}; *ldsp<u32x4>(at + 1024) = (u32x4){pad, pad, pad, pad};
    }
    fence();
#pragma unroll
    for (int i = 0; i < 16; ++i) {
        unsigned a = base[i] + ((i & 1) ? end[i >> 1] >> 16 : end[i >> 1] & 0xffffu);
#pragma unroll
        for (int e = 7; e >= 0; --e) {
            const unsigned nz = (unsigned)__builtin_amdgcn_sbfe((int)mk[i >> 2], (i & 3) * 8 + e, 1);
            a = lshl_add<1>(nz, a);
            if (e & 1) asm volatile("ds_write_b16_d16_hi %0, %1" :: "v"(a & nz), "v"(W[i][e >> 1]) : "memory");
            else asm volatile("ds_write_b16 %0, %1" :: "v"(a & nz), "v"(W[i][e >> 1]) : "memory");
        }
    }
    wait_lds();
    return encode_values<true>([&](int it, u32x4& x, u32x4& y) { x = *ldsp<u32x4>(cw + (it * 64 + lane) * 32); y = *ldsp<u32x4>(cw + (it * 64 + lane) * 32 + 16); },
                               []() {}, nch, T, mk, tail, dst, lane);
}

template <bool OUT, class PUT>
__device__ __forceinline__ void decode_values(PUT put, unsigned info, unsigned cw, unsigned esc, int lane) {
    const int nch = OUT ? (int)(info & 1023u) : 512;
    const unsigned T2 = ((info >> 19) & 0xfffu) * 0x100010u;
    u32x4 L[8]; unsigned h[8];
#pragma unroll
    for (int it = 0; it < 8; ++it) if (!OUT || it * 64 < nch) {
        L[it] = *ldsp<u32x4>(cw + (OUT ? 1024 : 0) + (it * 64 + lane) * 16);
        h[it] = OUT ? *ldsp<u16>(cw + cls_off<OUT>(nch) + (it * 64 + lane) * 2) : *ldsp<unsigned>(cw + cls_off<OUT>(nch) + (it * 64 + lane) * 4);
    }
    {
        const unsigned str = cw + cls_off<OUT>(nch) + cls_bytes<OUT>(nch), nw = (info >> 10) & 511u, bits = nw * 32u - 2u;
        const unsigned lg = nw > 248u ? 9u : nw > 124u ? 8u : 7u, S = (bits + (1u << lg) - 1u) >> lg, msk = ((1u << S) - 1u) << (31u - S);
        unsigned at = esc + 4u, before = 1u;
        for (unsigned s0 = 0; s0 < (1u << lg); s0 += 128u) {
            unsigned x[2], a[2], l[2];
#pragma unroll
            for (int k = 0; k < 2; ++k) {
                const unsigned o = (s0 + lane * 2u + k) * S, i = min(o >> 5, nw - 1u);
                const unsigned d0 = *ldsp<unsigned>(str + i * 4u), d1 = *ldsp<unsigned>(str + i * 4u + 4u);
                a[k] = o + 1u;
                x[k] = __builtin_amdgcn_alignbit(d0, i + 1u < nw ? d1 : 0u, ~o) & msk;
                if (o >= bits) x[k] = 0u;
                l[k] = x[k] ? a[k] + 31u - (unsigned)__builtin_ctz(x[k]) : 0u;
            }
            const unsigned c0 = __builtin_popcount(x[0]), c = c0 + __builtin_popcount(x[1]), ci = scan_add(c), li = scan_max(max(l[0], l[1]));
            const unsigned prev = max(before, dpp0<0x138, 0xf>(li));
            unsigned p = at + ci - c;
#pragma unroll
            for (int k = 0; k < 2; ++k) {
                unsigned w = x[k], carry = a[k] - (k ? max(prev, l[0]) : prev);
                while (w) {
                    const unsigned z = __builtin_clz(w), len = z + carry;
                    w = (w << z) + 0x80000000u; carry = 0u;
                    *ldsp<unsigned char>(p) = (unsigned char)(len + perm(0x020202fcu, 0x03030300u, min(len, 7u)));
                    ++p;
                }
            }
            at = min(at + last_lane(ci), esc + 4u + ESC_CAP); before = max(before, last_lane(li));
        }
    }
    u32x4 C[8], M[8]; unsigned cnt = 0;
#pragma unroll
    for (int it = 0; it < 8; ++it) if (!OUT || it * 64 < nch) {
        if (OUT) { h[it] = (h[it] | (h[it] << 12)) << 1; if (it * 64 + lane >= nch) h[it] = ~0u; }
#pragma unroll
        for (int q = 0; q < 4; ++q) {
            C[it][q] = OUT ? (L[it][q] & 0x01010101u) | ((h[it] >> q) & 0x02020202u) : (h[it] >> (2 * q)) & 0x03030303u;
            M[it][q] = perm(0u, 0x000000ffu, C[it][q]);
            cnt += __builtin_popcount(M[it][q]);
        }
    }
    cnt >>= 3;
    unsigned p = esc + 3u + scan_add(cnt) - cnt;
    fence();
#pragma unroll
    for (int it = 0; it < 8; ++it) if (!OUT || it * 64 < nch) {
        u32x4 x, y;
#pragma unroll
        for (int q = 0; q < 4; ++q) {
            const unsigned f = (M[it][q] & 0x01010101u) + (p & 3u), t = lshl_add<8>(f, f), sel = lshl_add<16>(t, t), ad = p & ~3u;
            const unsigned d0 = *ldsp<unsigned>(ad), d1 = *ldsp<unsigned>(ad + 4u);
            p = ad + (sel >> 24);
            const unsigned E = (perm(d1, d0, sel) & M[it][q]) | C[it][q], lo = OUT ? L[it][q] & 0xfefefefeu : L[it][q];
            const unsigned q0 = pk_rsub(T2, perm(E, lo, 0x05010400u)), q1 = pk_rsub(T2, perm(E, lo, 0x07030602u));
            const unsigned w0 = __builtin_amdgcn_alignbit(q0, q0, 1), w1 = __builtin_amdgcn_alignbit(q1, q1, 1);
            if (q < 2) { x[2 * q] = w0; x[2 * q + 1] = w1; } else { y[2 * q - 4] = w0; y[2 * q - 3] = w1; }
        }
        put(it, x, y);
    }
}

__device__ __forceinline__ void decode_in(unsigned info, unsigned cw, unsigned esc, char* dst, int lane) {
    decode_values<false>([&](int it, const u32x4& x, const u32x4& y) { st_wt(dst + it * 2048, x); st_wt(dst + it * 2048 + 1024, y); }, info, cw, esc, lane);
}
__device__ __forceinline__ void make_table(int n) {
    unsigned s[4], c = 0;
#pragma unroll
    for (int e = 0; e < 4; ++e) { const bool nz = (n >> e) & 1; s[e] = nz ? 0x0100u + c * 0x0202u : 0x0c0cu; c += nz; }
    *ldsp<u32x2>(LDS_TAB + n * 8) = (u32x2){s[0] | s[1] << 16, s[2] | s[3] << 16};
}
__device__ __forceinline__ void decode_out(unsigned info, unsigned cw, unsigned esc, char* outp, int lane) {
    const u32x4 mk = *ldsp<u32x4>(cw + lane * 16);
    decode_values<true>([&](int it, const u32x4& x, const u32x4& y) { *ldsp<u32x4>(cw + (it * 64 + lane) * 32) = x; *ldsp<u32x4>(cw + (it * 64 + lane) * 32 + 16) = y; }, info, cw, esc, lane);
    unsigned nl[16], nh[16], cl[16], st[8], base[16], run = cw;
#pragma unroll
    for (int k = 0; k < 8; ++k) {
        unsigned c[2];
#pragma unroll
        for (int j = 0; j < 2; ++j) {
            const int i = 2 * k + j;
            nl[i] = (((mk[i >> 2] & 0x0f0f0f0fu) << 3) >> (8 * (i & 3))) & 0xffu; nh[i] = (((mk[i >> 2] & 0xf0f0f0f0u) >> 1) >> (8 * (i & 3))) & 0xffu;
            cl[i] = __builtin_popcount(nl[i]); c[j] = cl[i] + __builtin_popcount(nh[i]);
        }
        const unsigned own = (c[0] | c[1] << 16) * 2u, inc = scan_add(own), t = last_lane(inc);
        st[k] = inc - own;
        base[2 * k] = run; run += t & 0xffffu; base[2 * k + 1] = run; run += t >> 16;
    }
    fence();
#pragma unroll
    for (int i = 0; i < 16; ++i) {
        const unsigned a0 = base[i] + ((i & 1) ? st[i >> 1] >> 16 : st[i >> 1] & 0xffffu), a1 = a0 + cl[i] * 2u;
        u32x4 o;
#pragma unroll
        for (int j = 0; j < 2; ++j) {
            const unsigned a = j ? a1 : a0, ad = a & ~3u;
            const unsigned d0 = *ldsp<unsigned>(ad), d1 = *ldsp<unsigned>(ad + 4u), d2 = *ldsp<unsigned>(ad + 8u);
            const u32x2 sel = *ldsp<u32x2>(LDS_TAB + (j ? nh[i] : nl[i]));
            const unsigned t0 = __builtin_amdgcn_alignbyte(d1, d0, a), t1 = __builtin_amdgcn_alignbyte(d2, d1, a);
            o[2 * j] = perm(t1, t0, sel.x); o[2 * j + 1] = perm(t1, t0, sel.y);
        }
        st_wt(outp + OUT_OFF(i), o);
    }
}

__device__ __forceinline__ void in_job(const Args& a, char* myws, int dest, int g, int kt, int j, int lane, unsigned seen, unsigned tail DBG_ARGS) {
    const int panel = g * 8 + ((j & 15) >> 1), half = j & 1, rank = a.rank;
    char* pq = a.ws[dest == rank ? (rank + 1 + lane) & 7 : dest];
    u32x4 v[16];
    if (j < 16) {
        const int sb = lane * 16, swz = sb ^ (((sb >> 9) & 1) << 5);
        const char* src = a.x + (size_t)(panel * 256 + half * 128 + (swz >> 6)) * (KS * 2) + kt * 128 + (swz & 63);
#pragma unroll
        for (int i = 0; i < 16; ++i) v[i] = *(const u32x4*)(src + (i >> 1) * 16 * (KS * 2) + (i & 1) * 64);
        __builtin_amdgcn_sched_barrier(0); wait_vm0();
    } else {
        const char* src = a.w + (size_t)(kt * 64 + (lane & 3) * 8) * (GN * 2) + (size_t)(dest * NB + panel * 256 + half * 128 + (lane >> 2) * 8) * 2;
        u32x4 r[16];
#pragma unroll
        for (int i = 0; i < 16; ++i) r[i] = *(const u32x4*)(src + (size_t)((i >> 3) * 32 + (i & 7)) * (GN * 2));
        __builtin_amdgcn_sched_barrier(0); wait_vm0();
#pragma unroll
        for (int kh = 0; kh < 2; ++kh)
#pragma unroll
            for (int i = 0; i < 8; ++i)
#pragma unroll
                for (int d = 0; d < 4; ++d) v[kh * 8 + i][d] = __builtin_amdgcn_perm(r[kh * 8 + 2 * d + 1][i >> 1], r[kh * 8 + 2 * d][i >> 1], (i & 1) ? 0x07060302u : 0x05040100u);
    }
    asm volatile("" : "+v"(pq));
    asm volatile("" : "+v"(seen));
    STAMPD(29);
    const unsigned call = seen + 1u;
    const int pos = g * 256 + kt * 32 + j;
    if (j < 16 || dest != rank) {
        char* dst = myws + (j < 16 ? OFF_XW : OFF_WW + (size_t)dest * (256 * 16384)) + (size_t)(g * 128 + kt * 16 + (j & 15)) * 16384 + lane * 16;
        GAS u64* const note = (GAS u64*)(pq + OFF_Q) + rank * NQ + pos;
        const unsigned info = encode_in(v, tail, dst, lane, [&]() {
            if ((pos & ~47) == 0) { wait_vm0(); if (lane < 7) __hip_atomic_store(note + NPOS, (u64)call << 32, __ATOMIC_RELAXED, __HIP_MEMORY_SCOPE_SYSTEM); STAMPD(30); } });
        if (info & INFO_RAW) {
            wait_vm0();
#pragma unroll
            for (int i = 0; i < 16; ++i) st_wt(dst + i * 1024, v[i]);
        }
        wait_vm0();
        if (lane < (j < 16 ? 7 : 1)) __hip_atomic_store(note, ((u64)call << 32) | info, __ATOMIC_RELAXED, __HIP_MEMORY_SCOPE_SYSTEM);
        STAMPD(31);
    }
    if (dest == rank) {
        char* dst = myws + img_off(g, j, kt) + lane * 16;
#pragma unroll
        for (int i = 0; i < 16; ++i) st_wt(dst + i * 1024, v[i]);
        wait_vm0();
        if (lane == 0) __hip_atomic_store((GAS unsigned*)(myws + flag_off(g, j, kt)), call, __ATOMIC_RELAXED, __HIP_MEMORY_SCOPE_SYSTEM);
    }
}

__device__ __forceinline__ bool expired(unsigned* tmo, unsigned call, u64 t0) {
    if (__builtin_amdgcn_readfirstlane(__hip_atomic_load(tmo, __ATOMIC_RELAXED, __HIP_MEMORY_SCOPE_AGENT)) == call || now() - t0 > DEADLINE) {
        __hip_atomic_store(tmo, call, __ATOMIC_RELAXED, __HIP_MEMORY_SCOPE_AGENT);
        return true;
    }
    return false;
}

__device__ __forceinline__ void tile_of(int ti, int xcd, int& pm, int& pn) { pm = (ti >> 4) * 8 + (xcd & 1) * 4 + (ti & 3); pn = ((ti >> 3) & 1) * 8 + (xcd >> 1) * 2 + ((ti >> 2) & 1); }
__device__ __forceinline__ size_t out_off(int src, int pm, int pn, int w, int lane) { return (size_t)(pm * 256 + (w >> 2) * 64 + (lane >> 3)) * (GN * 2) + (size_t)(src * NB + pn * 256 + (w & 3) * 64) * 2 + (lane & 7) * 16; }

__global__ __launch_bounds__(512, 2) void gemm_ar_kernel(const Args a) {
    extern __shared__ __attribute__((aligned(16))) unsigned char shm[];
    LAS unsigned char* lds = (LAS unsigned char*)shm;
    const int tid = threadIdx.x, wid = __builtin_amdgcn_readfirstlane(tid >> 6), lane = tid & 63;
    const int b = blockIdx.x, xcd = b & 7, idx = b >> 3, rank = a.rank;
    char* const myws = a.myws;
    char* const myout = a.myout;
    const u64 t0 = now();
    STAMP(0);
    unsigned* const ctr = (unsigned*)(myws + OFF_CTR) + b * 32 + wid;
    unsigned* const tmo = (unsigned*)(myws + OFF_TMO);
    const unsigned seen = __hip_atomic_load(ctr, __ATOMIC_RELAXED, __HIP_MEMORY_SCOPE_AGENT);
    const unsigned cw = LDS_CW + wid * C_WAVE, tl = LDS_TL + wid * C_TAIL;

    const int pk = (idx - NG + 7 * 8 + xcd) % 7, peer = (rank + 1 + pk) & 7, ord = xcd * 2 + (idx >= NG + 7);
    if (idx >= NG) { if (tid < 2) *ldsp<unsigned>(LDS_CTL + tid * 4) = tid ? 0u : 8u; if (tid < 16) make_table(tid); wait_lds(); bar(); }
    {
        int q, qe, delay;
        if (idx < 8) { q = wid; qe = 4; delay = (wid || !(idx & 2) ? DLY0 : 0) + wid * DLY1; }
        else if (idx < 10 || idx >= NG) { q = 0; qe = 0; delay = 0; }
        else { q = wid * 64 + (idx - 10) * 8 + xcd; qe = 2048; delay = DLY0 + (wid < 2 ? wid * DLY3 : (wid - 1) * DLY2); }
        for (int i = 0; i < delay; ++i) __builtin_amdgcn_s_sleep(8);
        for (; q < qe; q += 512) {
            int r;
            if (idx < 8) r = wid * 64 + (idx ^ 2) * 8 + xcd;
            else {
                const int kt = q < 896 ? q / 112 : 8 + (q - 896) / 144, rem = q < 896 ? q % 112 : (q - 896) % 144 - 32;
                r = kt * 32 + (rem < 0 ? rem + 32 : (rem % 7 + 1) * 512 + 16 + rem / 7);
            }
            in_job(a, myws, (rank + (r >> 9)) & 7, (r >> 8) & 1, (r >> 5) & 7, r & 31, lane, seen, tl DBG_PASS);
        }
    }
    const unsigned call = __builtin_amdgcn_readfirstlane(seen) + 1u;
    if (lane == 0) __hip_atomic_store(ctr, call, __ATOMIC_RELAXED, __HIP_MEMORY_SCOPE_AGENT);
    STAMP(1);

    if (idx >= NG) {
        const char* const pws = a.ws[peer];
        const u64* const queue = (const u64*)(myws + OFF_Q) + peer * NQ;
        unsigned* const tq = (unsigned*)(myws + OFF_TICK) + pk * 64;
        const unsigned tbase = (call - 1u) * NPOS - 128u;
        const unsigned ctl = LDS_CTL;
        unsigned pos = ((0x75634120u >> (4 * wid)) & 7u) * 16 + ord;
        bool bad = false;
        while (pos < (unsigned)NPOS) {
            const int g = pos >> 8, kt = (pos >> 5) & 7, j = pos & 31;
            const char* const src = pws + (pos >= NIN ? OFF_OW + (size_t)(pos - NIN) * 16384
                                           : (j < 16 ? OFF_XW : OFF_WW + (size_t)rank * (256 * 16384)) + (size_t)(g * 128 + kt * 16 + (j & 15)) * 16384) + lane * 16;
            unsigned info;
            int o = 0;
            for (unsigned at = (pos & ~47u) ? pos : pos + NPOS, spin = 1;; ++spin) {
                const u64 e = __hip_atomic_load(queue + at, __ATOMIC_RELAXED, __HIP_MEMORY_SCOPE_SYSTEM);
                info = __builtin_amdgcn_readfirstlane((unsigned)e);
                if (__builtin_amdgcn_readfirstlane((unsigned)(e >> 32)) >= call) {
                    if (at == pos) break;
                    STAMP1(2);
                    for (; o < 8192; o += 1024) { wait_vm<WIN - 1>(); __builtin_amdgcn_global_load_lds((const unsigned*)(src + o), ldsp<unsigned>(cw + o), 16, 0, 17); }
                    at = pos;
                    continue;
                }
                if ((spin & 15u) == 0 && expired(tmo, call, t0)) { bad = true; break; }
                if (at == pos) __builtin_amdgcn_s_sleep(POLL_SLEEP); else __builtin_amdgcn_s_sleep(POLL_FAST);
            }
            if (bad) break;
            LAP(24); STAMP1(2);
            if ((info & 1023u) > 512u || ((info >> 10) & 511u) - 1u >= (unsigned)NW_CAP) info = INFO_RAW;
            if (info & INFO_RAW) o = 0;
            const bool paced = pos < (unsigned)(NPOS - FREEN);
            const int wire = (info & INFO_RAW) ? 16384 : pos >= NIN ? wire_bytes<true>(info) : wire_bytes<false>(info);
            if (paced) {
                unsigned my = wid;
                if (lane == 0 && pos >= 128u) asm volatile("ds_add_rtn_u32 %0, %1, %2\n\ts_waitcnt lgkmcnt(0)" : "+v"(my) : "v"(ctl), "v"(1u) : "memory");
                my = __builtin_amdgcn_readfirstlane(my);
                for (;;) {
                    unsigned rel;
                    asm volatile("ds_read_b32 %0, %1 offset:4\n\ts_waitcnt lgkmcnt(0)" : "=v"(rel) : "v"(ctl) : "memory");
                    if ((int)(__builtin_amdgcn_readfirstlane(rel) + TOK - my) > 0) break;
                    if (now() - t0 > DEADLINE) break;
                    __builtin_amdgcn_s_sleep(1);
                }
            }
            LAP(25); STAMP1(7);
            for (; o < wire; o += 1024) {
                if (o + lane * 16 < wire) __builtin_amdgcn_global_load_lds((const unsigned*)(src + o), ldsp<unsigned>(cw + o), 16, 0, 17);
                if (pos >= (unsigned)(NPOS - RAWN)) wait_vm<RWIN - 1>(); else wait_vm<WIN - 1>();
            }
            wait_vm<REL>();
            if (lane == 0 && paced) asm volatile("ds_add_u32 %0, %1 offset:4" :: "v"(ctl), "v"(1u) : "memory");
            wait_vm0();
            LAP(26); STAMP1(3);
            int t = 0;
            char* dst;
            if (pos >= NIN) { int pm, pn; tile_of((pos - NIN) >> 6, ((pos - NIN) >> 3) & 7, pm, pn); dst = myout + out_off(peer, pm, pn, pos & 7, lane); }
            else { t = slot_of(peer, rank, kt); dst = myws + img_off(g, j, t) + lane * 16; }
            if (info & INFO_RAW) {
                u32x4 v[16];
#pragma unroll
                for (int i = 0; i < 16; ++i) v[i] = *ldsp<u32x4>(cw + i * 1024 + lane * 16);
#pragma unroll
                for (int i = 0; i < 16; ++i) st_wt(dst + (pos >= NIN ? OUT_OFF(i) : (size_t)i * 1024), v[i]);
            } else if (pos >= NIN) decode_out(info, cw, tl, dst, lane);
            else decode_in(info, cw, tl, dst, lane);
            LAP(27); STAMP1(4);
            unsigned nx = 0;
            if (lane == 0) nx = __hip_atomic_fetch_add(tq, 1u, __ATOMIC_RELAXED, __HIP_MEMORY_SCOPE_AGENT);
            wait_vm0();
            if (pos < NIN && lane == 0) __hip_atomic_store((unsigned*)(myws + flag_off(g, j, t)), call, __ATOMIC_RELAXED, __HIP_MEMORY_SCOPE_SYSTEM);
            LAP(28); STAMP(8 + (pos >> 8)); STAMP1(6);
            pos = __builtin_amdgcn_readfirstlane(nx) - tbase;
        }
        if (bad && lane == 0) *(GAS u16*)(myout + (size_t)rank * (NB * 2)) = 0x7fc0;
        STAMP(5);
        STAMP_FLUSH();
        return;
    }

    const int wr = wid >> 2, wc = wid & 3, fr = lane & 15, fq = lane >> 4;
    bool bad = false;
    const bool hm = idx < 4;
    unsigned voffA[2], voffB[2];
#pragma unroll
    for (int i = 0; i < 2; ++i) {
        int R, C; stage_rc(tid * 16 + i * 8192, R, C);
        voffA[i] = tid * 16 + (hm ? (idx & 1) * 8192 + i * 16384 : i * 8192);
        const int w_ = R >> 5, n_ = (R >> 4) & 1, q_ = (R & 15) >> 2, j_ = R & 3;
        voffB[i] = (w_ >> 1) * 16384 + (C >> 5) * 8192 + (4 * n_ + j_) * 1024 + ((w_ & 1) * 8 + q_) * 64 + ((C >> 3) & 3) * 16;
    }
    constexpr size_t kstep = 32768, hstepA = 16384, hstepB = 256;
    const unsigned ldsw = (unsigned)wid * 1024u;
    const int aoff = lds_byte(wr * 64 + fr, fq * 8), boff = lds_byte(wc * 32 + fr, fq * 8);
#define G_SA(b, h) (((b) * 2 + (h)) * HTB)
#define G_SB(b, h) ((4 + (b) * 2 + (h)) * HTB)
#define G_STAGE(bufoff, gbase, voff) do { _Pragma("unroll") for (int _i = 0; _i < 2; ++_i) \
        __builtin_amdgcn_global_load_lds((const unsigned*)((const char*)(gbase) + (voff)[_i]), (LAS unsigned*)(lds + (bufoff) + ldsw + _i * 8192), 16, 0, 17); } while (0)
#define G_LDA(dst, b, h) do { _Pragma("unroll") for (int m = 0; m < 4; ++m) _Pragma("unroll") for (int k = 0; k < 2; ++k) dst[m][k] = *(const LAS bf16x8*)(lds + G_SA(b, h) + aoff + m * 2048 + k * 1024); } while (0)
#define G_LDB(dst, b, h) do { _Pragma("unroll") for (int n = 0; n < 2; ++n) _Pragma("unroll") for (int k = 0; k < 2; ++k) dst[n][k] = *(const LAS bf16x8*)(lds + G_SB(b, h) + boff + n * 2048 + k * 1024); } while (0)
#define G_MMA(ai, bj, At, Bt) do { __builtin_amdgcn_s_setprio(1); _Pragma("unroll") for (int m = 0; m < 4; ++m) _Pragma("unroll") for (int n = 0; n < 2; ++n) _Pragma("unroll") for (int k = 0; k < 2; ++k) \
        acc[ai][bj][m][n] = __builtin_amdgcn_mfma_f32_16x16x32_bf16(Bt[n][k], At[m][k], acc[ai][bj][m][n], 0, 0, 0); __builtin_amdgcn_s_setprio(0); } while (0)
#define G_WAIT_V(n) asm volatile("s_waitcnt vmcnt(" #n ")" ::: "memory")
#define G_WAIT_L(n) asm volatile("s_waitcnt lgkmcnt(" #n ")" ::: "memory")
#define G_BAR __builtin_amdgcn_s_barrier()
#define G_SCHED __builtin_amdgcn_sched_barrier(0)
    for (int round = 0; round < 2; ++round) {
    const int ti = hm ? round * 8 + (idx >> 1) : round ? idx + 14 : idx < 10 ? idx - 2 : idx;
    int pm, pn; tile_of(ti, xcd, pm, pn);
    const char* const fX = myws + OFF_INF + (size_t)pm * 512 + lane * 8;
    const char* const fW = myws + OFF_INF + (size_t)(16 + pn) * 512 + lane * 8;
    auto poll = [&](int need) -> int {
        for (;;) {
            u32x2 fa, fb;
            asm volatile("global_load_dwordx2 %0, %2, off sc0 sc1\n\tglobal_load_dwordx2 %1, %3, off sc0 sc1\n\ts_waitcnt vmcnt(0)"
                         : "=&v"(fa), "=&v"(fb) : "v"(fX), "v"(fW) : "memory");
            const u64 miss = __builtin_amdgcn_ballot_w64(fa.x < call || fa.y < call || fb.x < call || fb.y < call);
            const int upto = miss ? __builtin_ctzll(miss) : 64;
            if (upto >= need) return upto;
            if (expired(tmo, call, t0)) { bad = true; return 64; }
            __builtin_amdgcn_s_sleep(4);
        }
    };
    f32x4 acc[2][2][4][2];
#pragma unroll
    for (int x = 0; x < 2; ++x)
#pragma unroll
        for (int y = 0; y < 2; ++y)
#pragma unroll
            for (int m = 0; m < 4; ++m)
#pragma unroll
                for (int n = 0; n < 2; ++n) acc[x][y][m][n] = (f32x4){0.f, 0.f, 0.f, 0.f};
    bf16x8 At[4][2], B0[2][2], B1[2][2];
    const char* const cA = myws + OFF_XI + (size_t)pm * 64 * kstep;
    const char* const cB = myws + OFF_WI + (size_t)pn * 64 * kstep;
    int ready = poll(2);
    STAMP1(2);
    wait_vm0();
    if (hm) {
        G_STAGE(0, cA, voffA); G_STAGE(HTB, cB, voffB); G_STAGE(2 * HTB, cB + hstepB, voffB);
        G_STAGE(3 * HTB, cA + kstep, voffA); G_STAGE(4 * HTB, cB + kstep, voffB); G_STAGE(5 * HTB, cB + kstep + hstepB, voffB);
        if (wr == 1) G_BAR;
        G_WAIT_V(6); G_BAR; G_BAR;
        unsigned cur = 0, nxt = 6 * HTB;
        for (int t = 0; t < 64; ++t) {
            if (t < 62 && ready < t + 3) ready = poll(t + 3);
            const size_t k2 = t < 62 ? (size_t)(t + 2) * kstep : 0;
            LAS unsigned char* const la = lds + cur + aoff; LAS unsigned char* const lb = lds + cur + boff;
#pragma unroll
            for (int n = 0; n < 2; ++n)
#pragma unroll
                for (int k = 0; k < 2; ++k) B0[n][k] = *(const LAS bf16x8*)(lb + HTB + n * 2048 + k * 1024);
#pragma unroll
            for (int m = 0; m < 4; ++m)
#pragma unroll
                for (int k = 0; k < 2; ++k) At[m][k] = *(const LAS bf16x8*)(la + m * 2048 + k * 1024);
            G_STAGE(nxt, cA + k2, voffA); G_STAGE(nxt + HTB, cB + k2, voffB);
            G_WAIT_V(10); G_BAR; G_WAIT_L(0); G_MMA(0, 0, At, B0); G_BAR; G_SCHED;
#pragma unroll
            for (int n = 0; n < 2; ++n)
#pragma unroll
                for (int k = 0; k < 2; ++k) B1[n][k] = *(const LAS bf16x8*)(lb + 2 * HTB + n * 2048 + k * 1024);
            G_STAGE(nxt + 2 * HTB, cB + k2 + hstepB, voffB);
            G_WAIT_V(8); G_BAR; G_WAIT_L(0); G_MMA(0, 1, At, B1); G_BAR; G_SCHED;
            nxt = cur; cur = cur == 6 * HTB ? 0 : cur + 3 * HTB;
        }
    } else {
    G_STAGE(G_SB(0, 0), cB, voffB); G_STAGE(G_SA(0, 0), cA, voffA); G_STAGE(G_SB(0, 1), cB + hstepB, voffB); G_STAGE(G_SA(0, 1), cA + hstepA, voffA);
    if (wr == 1) G_BAR;
    G_WAIT_V(4); G_BAR;
    G_STAGE(G_SB(1, 0), cB + kstep, voffB); G_STAGE(G_SA(1, 0), cA + kstep, voffA); G_STAGE(G_SB(1, 1), cB + hstepB + kstep, voffB);
    G_WAIT_V(6); G_BAR;
    for (int t = 0; t < 64; t += 2) {
        const bool last = (t == 62);
        if (!last && ready < t + 4) ready = poll(t + 4);
        const char* a1 = cA + (size_t)(t + 1) * kstep;
        const char* a2 = last ? cA : cA + (size_t)(t + 2) * kstep; const char* b2 = last ? cB : cB + (size_t)(t + 2) * kstep;
        const char* a3 = a2 + kstep; const char* b3 = b2 + kstep;
        G_LDB(B0, 0, 0); G_SCHED; G_LDA(At, 0, 0); G_STAGE(G_SA(1, 1), a1 + hstepA, voffA);
        G_WAIT_L(8); G_BAR; G_WAIT_L(0); G_MMA(0, 0, At, B0); G_BAR; G_SCHED;
        G_LDB(B1, 0, 1); G_STAGE(G_SB(0, 0), b2, voffB);
        G_BAR; G_WAIT_L(0); G_MMA(0, 1, At, B1); G_BAR;
        G_LDA(At, 0, 1); G_STAGE(G_SA(0, 0), a2, voffA);
        G_BAR; G_WAIT_L(0); G_MMA(1, 0, At, B0); G_BAR; G_SCHED;
        G_STAGE(G_SB(0, 1), b2 + hstepB, voffB);
        G_WAIT_V(6); G_BAR; G_MMA(1, 1, At, B1); G_BAR;
        G_LDB(B0, 1, 0); G_SCHED; G_LDA(At, 1, 0); G_STAGE(G_SA(0, 1), a2 + hstepA, voffA);
        G_WAIT_L(8); G_BAR; G_WAIT_L(0); G_MMA(0, 0, At, B0); G_BAR; G_SCHED;
        G_LDB(B1, 1, 1); G_STAGE(G_SB(1, 0), b3, voffB);
        G_BAR; G_WAIT_L(0); G_MMA(0, 1, At, B1); G_BAR;
        G_LDA(At, 1, 1); G_STAGE(G_SA(1, 0), a3, voffA);
        G_BAR; G_WAIT_L(0); G_MMA(1, 0, At, B0); G_BAR; G_SCHED;
        G_STAGE(G_SB(1, 1), b3 + hstepB, voffB);
        G_WAIT_V(6); G_BAR; G_MMA(1, 1, At, B1); G_BAR;
    }
    }
    G_WAIT_V(0);
    if (wr == 0) G_BAR;
    G_BAR;
    STAMP(round ? 6 : 3);

    int ln = lane;
    asm volatile("" : "+v"(ln) :: "memory");
    char* pq = a.ws[(rank + 1 + (ln >> 3)) & 7];
    if (bad) acc[0][0][0][0][0] = __builtin_nanf("");
    const unsigned cx = hm ? LDS_CW + ((idx & 1) * 4 + wc) * C_WAVE + wr * 8192 : cw;
#pragma unroll
    for (int ai = 0; ai < 2; ++ai) if (ai == 0 || !hm)
#pragma unroll
        for (int m = 0; m < 4; ++m)
#pragma unroll
            for (int bj = 0; bj < 2; ++bj) {
                const f32x4 v0 = acc[ai][bj][m][0], v1 = acc[ai][bj][m][1];
                u32x4 w; w[0] = relu_pk(cvt_pk_bf16(v0[0], v0[1])); w[1] = relu_pk(cvt_pk_bf16(v0[2], v0[3]));
                w[2] = relu_pk(cvt_pk_bf16(v1[0], v1[1])); w[3] = relu_pk(cvt_pk_bf16(v1[2], v1[3]));
                *ldsp<u32x4>(cx + (ai * 64 + m * 16 + (ln & 15)) * 128 + (((bj * 4 + (ln >> 4)) ^ (ln & 7)) << 4)) = w;
            }
    asm volatile("" ::: "memory");
    if (hm) { wait_lds(); bar(); }
    if (!hm || wr == (idx & 1)) {
    u32x4 W[16];
#pragma unroll
    for (int i = 0; i < 16; ++i) W[i] = *ldsp<u32x4>(cw + i * 1024 + (ln >> 3) * 128 + (((ln & 7) ^ (ln >> 3)) << 4));
    wait_lds(); wait_vm0();
    asm volatile("" : "+v"(pq));
    char* const outp = myout + out_off(rank, pm, pn, wid, ln);
#pragma unroll
    for (int i = 0; i < 16; ++i) st_wt(outp + OUT_OFF(i), W[i]);
    char* const slot = myws + OFF_OW + (size_t)((ti * 8 + xcd) * 8 + wid) * 16384 + ln * 16;
    const unsigned info = (ti * 8 + xcd) * 8 + wid >= NPOS - NIN - RAWN ? INFO_RAW : encode_out(W, cw, tl, slot, ln);
    if (info & INFO_RAW) {
#pragma unroll
        for (int i = 0; i < 16; ++i) st_wt(slot + i * 1024, W[i]);
    }
    if (ln == 0) *(LAS unsigned*)(lds + LDS_CTL + wid * 4) = info;
    }
    wait_lds();
    wait_vm0();
    bar();
    if (wid == 0 && ln < 56 && (!hm || ((ln >> 2) & 1) == (idx & 1))) {
        const unsigned inf = *(const LAS unsigned*)(lds + LDS_CTL + (ln & 7) * 4);
        __hip_atomic_store((GAS u64*)(pq + OFF_Q) + rank * NQ + NIN + (ti * 8 + xcd) * 8 + (ln & 7), ((u64)call << 32) | inf, __ATOMIC_RELAXED, __HIP_MEMORY_SCOPE_SYSTEM);
    }
    STAMP(round ? 7 : 4);
    }
    STAMP(5);
    STAMP_FLUSH();
}

extern "C" void kernel_launch(int rank, int nranks, ncclComm_t comm,
                              hipStream_t stream,
                              void* const* d_in, const int* in_sizes, int n_in,
                              void* d_out, int out_size,
                              const int* in_dtypes, int out_dtype,
                              void* const* d_out_all,
                              void* const* d_ws_all, size_t ws_bytes) {
    if (nranks != NR || (unsigned)rank >= (unsigned)NR || ws_bytes < WS_NEED || n_in < 2) { fprintf(stderr, "unsupported configuration\n"); return; }
    static bool attr[NR] = {};
    if (!attr[rank]) { HIP_CHECK(hipFuncSetAttribute((const void*)gemm_ar_kernel, hipFuncAttributeMaxDynamicSharedMemorySize, LDS_BYTES)); attr[rank] = true; }
    Args a;
    a.x = (const char*)d_in[0]; a.w = (const char*)d_in[1]; a.rank = rank;
    for (int p = 0; p < NR; ++p) a.ws[p] = (char*)d_ws_all[p];
    a.myws = a.ws[rank]; a.myout = (char*)d_out;
    hipLaunchKernelGGL(gemm_ar_kernel, dim3(256), dim3(512), LDS_BYTES, stream, a);
    HIP_CHECK(hipGetLastError());
}
```

```cpp
#include <hip/hip_runtime.h>
#include <rccl/rccl.h>
#include <cstdio>

#define HIP_CHECK(expr) do { hipError_t _e = (expr); if (_e != hipSuccess) { \
    fprintf(stderr, "HIP error %s at %s:%d\n", hipGetErrorString(_e), __FILE__, __LINE__); \
    return; } } while (0)

#define LAS __attribute__((address_space(3)))
#define GAS __attribute__((address_space(1)))
typedef unsigned short u16;
typedef unsigned long long u64;
typedef short bf16x8 __attribute__((ext_vector_type(8)));
typedef float f32x4 __attribute__((ext_vector_type(4)));
typedef unsigned u32x4 __attribute__((ext_vector_type(4)));
typedef unsigned u32x2 __attribute__((ext_vector_type(2)));

constexpr int NR = 8;
constexpr int GM = 4096, GN = 32768, KS = 512, NB = 4096;
constexpr int HTB = 16384;
constexpr int NG = 18;
constexpr int NIN = 512, NPOS = 2560;
constexpr int NQ = NPOS + 64;
constexpr unsigned LDS_TAB = 64, LDS_CW = 256, C_WAVE = 16384 + 128, LDS_TL = LDS_CW + 8 * C_WAVE, C_TAIL = 3584, ESC_CAP = 3560;
constexpr unsigned LDS_CTL = LDS_TL + 8 * C_TAIL + 64;
constexpr int LDS_BYTES = LDS_CTL + 64;
static_assert(LDS_TL >= 8 * HTB && LDS_CTL >= 9 * HTB && LDS_BYTES <= 163840, "LDS layout");
constexpr size_t OFF_CTR = 0;
constexpr size_t OFF_TMO = 32768;
constexpr size_t OFF_TICK = 32768 + 256;
constexpr size_t OFF_STAMP = 65536;
constexpr size_t OFF_INF = 512u << 10;
constexpr size_t OFF_Q = 1u << 20;
constexpr size_t OFF_XI = 4u << 20;
constexpr size_t OFF_WI = 36u << 20;
constexpr size_t OFF_XW = 68u << 20;
constexpr size_t OFF_WW = 72u << 20;
constexpr size_t OFF_OW = 104u << 20;
constexpr size_t WS_NEED = 136u << 20;
constexpr u64 DEADLINE = 50000000ull;

#define TOK 2
#define WIN 2
#define REL (WIN - 1)
#define DLY1 3
#define DLY2 5
#define POLL_SLEEP 6
#define DLY0 4
#define EARLY_AT 3
#define EWIN 2
#define EWN 4
#define DLY3 0
#define POLL_FAST 1
#define WIRE_ALIGN 64
struct Args { const char* x; const char* w; char* myws; char* myout; int rank; char* ws[NR]; };

__device__ __forceinline__ int lds_byte(int r, int c) { const int st = (r >> 4) * 2 + (c >> 5), rr = r & 15, cc = c & 31, ob = rr * 64 + cc * 2; return st * 1024 + (ob ^ (((ob >> 9) & 1) << 5)); }
__device__ __forceinline__ void stage_rc(int b, int& R, int& C) { const int st = b / 1024, sb = b % 1024, swz = sb ^ (((sb >> 9) & 1) << 5); R = (st >> 1) * 16 + swz / 64; C = (st & 1) * 32 + (swz % 64) / 2; }
__device__ __forceinline__ unsigned cvt_pk_bf16(float lo, float hi) { unsigned r; asm volatile("v_cvt_pk_bf16_f32 %0, %1, %2" : "=v"(r) : "v"(lo), "v"(hi)); return r; }
__device__ __forceinline__ unsigned relu_pk(unsigned v) { unsigned r; asm("v_pk_max_i16 %0, %1, 0" : "=v"(r) : "v"(v)); return r; }
__device__ __forceinline__ void st_wt(char* p, u32x4 v) { asm volatile("global_store_dwordx4 %0, %1, off sc0 sc1\n\ts_nop 1" :: "v"(p), "v"(v) : "memory"); }
__device__ __forceinline__ void wait_vm0() { __builtin_amdgcn_s_waitcnt(0x0F70); asm volatile("s_waitcnt vmcnt(0)" ::: "memory"); }
template <int N> __device__ __forceinline__ void wait_vm() { asm volatile("s_waitcnt vmcnt(%0)" :: "n"(N) : "memory"); }
__device__ __forceinline__ void bar() { asm volatile("" ::: "memory"); __builtin_amdgcn_s_barrier(); asm volatile("" ::: "memory"); }
__device__ __forceinline__ u64 now() { return __builtin_amdgcn_s_memrealtime(); }

#define STAMP(i) do { } while (0)
#define STAMP1(i) do { } while (0)
#define STAMPD(i) do { } while (0)
#define DBG_ARGS
#define DBG_PASS
#define LAP(i) do { } while (0)
#define STAMP_FLUSH() do { } while (0)

__device__ __forceinline__ int slot_of(int s, int p, int kt) { return s == p ? kt : 8 + 7 * kt + ((s - p - 1) & 7); }
__device__ __forceinline__ size_t img_off(int g, int j, int t) { return (j < 16 ? OFF_XI : OFF_WI) + (size_t)((g * 8 + ((j & 15) >> 1)) * 64 + t) * 32768 + (j & 1) * 16384; }
__device__ __forceinline__ size_t flag_off(int g, int j, int t) { return OFF_INF + (size_t)((((j >> 4) * 16 + g * 8 + ((j & 15) >> 1)) * 64 + t) * 2 + (j & 1)) * 4; }

#define OUT_OFF(i) ((size_t)(((i) >> 3) * 128 + ((i) & 7) * 8) * (GN * 2))
constexpr unsigned INFO_RAW = 1u << 31;
constexpr int NW_CAP = 384;
template <bool OUT> __device__ __forceinline__ int cls_off(int nch) { return OUT ? 1024 + nch * 16 : 8192; }
template <bool OUT> __device__ __forceinline__ int cls_bytes(int nch) { return OUT ? (nch * 2 + 3) & ~3 : 2048; }
template <bool OUT> __device__ __forceinline__ int wire_bytes(unsigned info) {
    const int nch = info & 1023; return (cls_off<OUT>(nch) + cls_bytes<OUT>(nch) + (int)((info >> 10) & 511) * 4 + (WIRE_ALIGN - 1)) & ~(WIRE_ALIGN - 1); }
__device__ __forceinline__ unsigned pk_min_u16(unsigned x, unsigned y) { unsigned r; asm("v_pk_min_u16 %0, %1, %2" : "=v"(r) : "v"(x), "s"(y)); return r; }
__device__ __forceinline__ unsigned pk_rsub(unsigned s, unsigned y) { unsigned r; asm("v_pk_sub_u16 %0, %1, %2" : "=v"(r) : "s"(s), "v"(y)); return r; }
__device__ __forceinline__ unsigned sum_u16(unsigned x, unsigned acc) { unsigned r; asm("v_sad_u16 %0, %1, 0, %2" : "=v"(r) : "v"(x), "v"(acc)); return r; }
template <int S> __device__ __forceinline__ unsigned lshl_add(unsigned x, unsigned y) { unsigned r; asm("v_lshl_add_u32 %0, %1, %2, %3" : "=v"(r) : "v"(x), "n"(S), "v"(y)); return r; }
template <int S> __device__ __forceinline__ unsigned lshl_or(unsigned x, unsigned y) { unsigned r; asm("v_lshl_or_b32 %0, %1, %2, %3" : "=v"(r) : "v"(x), "n"(S), "v"(y)); return r; }
__device__ __forceinline__ unsigned perm(unsigned hi, unsigned lo, unsigned sel) { return __builtin_amdgcn_perm(hi, lo, sel); }
__device__ __forceinline__ void wait_lds() { asm volatile("s_waitcnt lgkmcnt(0)" ::: "memory"); }
__device__ __forceinline__ void fence() { asm volatile("" ::: "memory"); }
template <int C, int RM> __device__ __forceinline__ unsigned dpp0(unsigned x) { return (unsigned)__builtin_amdgcn_update_dpp(0, (int)x, C, RM, 0xf, false); }
__device__ __forceinline__ unsigned scan_add(unsigned x) {
    x += dpp0<0x111, 0xf>(x); x += dpp0<0x112, 0xf>(x); x += dpp0<0x114, 0xf>(x); x += dpp0<0x118, 0xf>(x); x += dpp0<0x142, 0xa>(x); x += dpp0<0x143, 0xc>(x); return x; }
__device__ __forceinline__ unsigned scan_max(unsigned x) {
    x = max(x, dpp0<0x111, 0xf>(x)); x = max(x, dpp0<0x112, 0xf>(x)); x = max(x, dpp0<0x114, 0xf>(x)); x = max(x, dpp0<0x118, 0xf>(x)); x = max(x, dpp0<0x142, 0xa>(x)); x = max(x, dpp0<0x143, 0xc>(x)); return x; }
__device__ __forceinline__ unsigned last_lane(unsigned x) { return (unsigned)__builtin_amdgcn_readlane((int)x, 63); }
template <class T> __device__ __forceinline__ LAS T* ldsp(unsigned a) { return (LAS T*)(size_t)a; }

template <bool OUT, class GET, class MID>
__device__ __forceinline__ unsigned encode_values(GET get, MID mid, int nch, unsigned T, const u32x4& mk, unsigned tail, char* dst, int lane) {
    const unsigned T2 = T * 0x10001u;
    u32x4 R0[8], R1[8];
    if (!OUT) {
#pragma unroll
        for (int it = 0; it < 8; ++it) {
            u32x4 x, y, lo; get(it, x, y);
#pragma unroll
            for (int q = 0; q < 4; ++q) {
                const unsigned w0 = q < 2 ? x[2 * q] : y[2 * q - 4], w1 = q < 2 ? x[2 * q + 1] : y[2 * q - 3];
                R0[it][q] = pk_rsub(T2, __builtin_amdgcn_alignbit(w0, w0, 31)); R1[it][q] = pk_rsub(T2, __builtin_amdgcn_alignbit(w1, w1, 31));
                lo[q] = perm(R1[it][q], R0[it][q], 0x06040200u);
            }
            st_wt(dst + it * 1024, lo);
        }
    }
#pragma unroll
    for (int i = 0; i < 4; ++i) if (i < 3 || lane < 32) *ldsp<u32x4>(tail + i * 1024 + lane * 16) = (u32x4){0u, 0u, 0u, 0u};
    fence();
    u32x4 Lb[8], Ln[8]; unsigned tot = 0, direct = 0;
#pragma unroll
    for (int it = 0; it < 8; ++it) if (!OUT || it * 64 < nch) {
        u32x4 x, y; if (OUT) get(it, x, y);
        if (it == EARLY_AT) mid();
        unsigned h = 0, h0 = 0;
#pragma unroll
        for (int q = 0; q < 4; ++q) {
            const unsigned w0 = q < 2 ? x[2 * q] : y[2 * q - 4], w1 = q < 2 ? x[2 * q + 1] : y[2 * q - 3];
            const unsigned r0 = OUT ? pk_rsub(T2, w0 << 1) : R0[it][q], r1 = OUT ? pk_rsub(T2, w1 << 1) : R1[it][q];
            const unsigned ec = perm(pk_min_u16(r1, 0x07ff07ffu), pk_min_u16(r0, 0x07ff07ffu), 0x07050301u);
            Ln[it][q] = perm(r1, r0, 0x07050301u) + perm(0x01000000u, 0x00010207u, ec) - 0x03030303u;
            tot = __builtin_amdgcn_sad_u8(Ln[it][q], 0u, tot);
            if (OUT) { const unsigned b0 = perm(0u, 0x01000100u, ec); Lb[it][q] = perm(r1, r0, 0x06040200u) | b0; h0 |= b0 << q; h |= perm(0u, 0x01010000u, ec) << q; }
            else h |= perm(0u, 0x03020100u, ec) << (2 * q);
        }
        if (OUT) { direct += __builtin_popcount(h | h0); if (it * 64 + lane < nch) *ldsp<u16>(tail + (it * 64 + lane) * 2) = (u16)(h | (h >> 12)); }
        else { direct += __builtin_popcount((h | (h >> 1)) & 0x55555555u); *ldsp<unsigned>(tail + (it * 64 + lane) * 4) = h; }
    }
    const unsigned inc = scan_add(tot), nw = (last_lane(inc) + 33u) >> 5;
    const unsigned nesc = (OUT ? (unsigned)((nch + 63) >> 6) * 1024u : 8192u) - last_lane(scan_add(direct));
    if (nw > (unsigned)NW_CAP || nesc > (unsigned)ESC_CAP) return INFO_RAW;
    if (OUT) st_wt(dst, mk);
    unsigned pos = (tail + cls_bytes<OUT>(nch)) * 8u + 1u + inc - tot;
#pragma unroll
    for (int it = 0; it < 8; ++it) if (!OUT || it * 64 < nch) {
        if (OUT && it * 64 + lane < nch) st_wt(dst + 1024 + it * 1024, Lb[it]);
#pragma unroll
        for (int e = 0; e < 16; ++e) {
            pos += (Ln[it][e >> 2] >> (8 * (e & 3))) & 0xffu;
            asm volatile("ds_or_b32 %0, %1" :: "v"((pos >> 3) & ~3u), "v"(0x80000000u >> (pos & 31u)) : "memory");
        }
    }
    const unsigned info = (T >> 4) << 19 | nw << 10 | (OUT ? (unsigned)nch : 0u);
    const int rest = wire_bytes<OUT>(info) - cls_off<OUT>(nch);
    wait_lds();
#pragma unroll
    for (int i = 0; i < 4; ++i) if (i * 1024 < rest) { if (i * 1024 + lane * 16 < rest) st_wt(dst + cls_off<OUT>(nch) + i * 1024, *ldsp<u32x4>(tail + i * 1024 + lane * 16)); }
    return info;
}
__device__ __forceinline__ unsigned top_of(unsigned sum2, unsigned n) {
    return ((unsigned)__builtin_amdgcn_readfirstlane((int)((float)sum2 * __builtin_amdgcn_rcpf((float)n))) + 2u * (276u + 128u) + 8u) & 0xfff0u; }

template <class MID> __device__ __forceinline__ unsigned encode_in(const u32x4 (&v)[16], unsigned tail, char* dst, int lane, MID mid) {
    unsigned s = 0;
#pragma unroll
    for (int i = 0; i < 16; ++i) s = sum_u16(__builtin_amdgcn_alignbit(v[i][i & 3], v[i][i & 3], 31), s);
    return encode_values<false>([&](int it, u32x4& x, u32x4& y) { x = v[2 * it]; y = v[2 * it + 1]; }, mid, 512, top_of(last_lane(scan_add(s)), 2048u), v[0], tail, dst, lane);
}

__device__ __forceinline__ unsigned encode_out(const u32x4 (&W)[16], unsigned cw, unsigned tail, char* dst, int lane) {
    u32x4 mk; unsigned s = 0;
#pragma unroll
    for (int j = 0; j < 4; ++j) {
        unsigned t[2] = {0u, 0u};
#pragma unroll
        for (int d = 15; d >= 0; --d) { const unsigned w = W[j * 4 + (d >> 2)][d & 3]; t[d >> 3] = lshl_or<2>(t[d >> 3], pk_min_u16(w, 0x00010001u)); if (j == 0) s = sum_u16(w, s); }
        mk[j] = perm(t[1], t[0], 0x05040100u) | (perm(t[1], t[0], 0x07060302u) << 1);
    }
    unsigned end[8], run = cw;
    unsigned base[16];
#pragma unroll
    for (int k = 0; k < 8; ++k) {
        const unsigned m = mk[k >> 1] >> (16 * (k & 1));
        end[k] = scan_add(((unsigned)__builtin_popcount(m & 0xffu) | (unsigned)__builtin_popcount(m & 0xff00u) << 16) * 2u);
    }
#pragma unroll
    for (int k = 0; k < 8; ++k) { const unsigned t = last_lane(end[k]); base[2 * k] = run; run += t & 0xffffu; base[2 * k + 1] = run; run += t >> 16; }
    const unsigned n = (run - cw) >> 1;
    const int nch = (int)(n + 15u) >> 4;
    const unsigned T = top_of(4u * last_lane(scan_add(s)), max(base[4] - cw, 1u));
    {
        const unsigned pad = (((T - 0x200u) >> 1) & 0x7fffu) * 0x10001u, at = cw + (min(n, 8191u) >> 10) * 2048u + lane * 16;
        *ldsp<u32x4>(at) = (u32x4){pad, pad, pad, pad}; *ldsp<u32x4>(at + 1024) = (u32x4){pad, pad, pad, pad};
    }
    fence();
#pragma unroll
    for (int i = 0; i < 16; ++i) {
        unsigned a = base[i] + ((i & 1) ? end[i >> 1] >> 16 : end[i >> 1] & 0xffffu);
#pragma unroll
        for (int e = 7; e >= 0; --e) {
            const unsigned nz = (unsigned)__builtin_amdgcn_sbfe((int)mk[i >> 2], (i & 3) * 8 + e, 1);
            a = lshl_add<1>(nz, a);
            if (e & 1) asm volatile("ds_write_b16_d16_hi %0, %1" :: "v"(a & nz), "v"(W[i][e >> 1]) : "memory");
            else asm volatile("ds_write_b16 %0, %1" :: "v"(a & nz), "v"(W[i][e >> 1]) : "memory");
        }
    }
    wait_lds();
    return encode_values<true>([&](int it, u32x4& x, u32x4& y) { x = *ldsp<u32x4>(cw + (it * 64 + lane) * 32); y = *ldsp<u32x4>(cw + (it * 64 + lane) * 32 + 16); },
                               []() {}, nch, T, mk, tail, dst, lane);
}

template <bool OUT, class PUT>
__device__ __forceinline__ void decode_values(PUT put, unsigned info, unsigned cw, unsigned esc, int lane) {
    const int nch = OUT ? (int)(info & 1023u) : 512;
    const unsigned T2 = ((info >> 19) & 0xfffu) * 0x100010u;
    u32x4 L[8]; unsigned h[8];
#pragma unroll
    for (int it = 0; it < 8; ++it) if (!OUT || it * 64 < nch) {
        L[it] = *ldsp<u32x4>(cw + (OUT ? 1024 : 0) + (it * 64 + lane) * 16);
        h[it] = OUT ? *ldsp<u16>(cw + cls_off<OUT>(nch) + (it * 64 + lane) * 2) : *ldsp<unsigned>(cw + cls_off<OUT>(nch) + (it * 64 + lane) * 4);
    }
    {
        const unsigned str = cw + cls_off<OUT>(nch) + cls_bytes<OUT>(nch), nw = (info >> 10) & 511u, bits = nw * 32u - 2u;
        const unsigned lg = nw > 248u ? 9u : nw > 124u ? 8u : 7u, S = (bits + (1u << lg) - 1u) >> lg, msk = ((1u << S) - 1u) << (31u - S);
        unsigned at = esc + 4u, before = 1u;
        for (unsigned s0 = 0; s0 < (1u << lg); s0 += 128u) {
            unsigned x[2], a[2], l[2];
#pragma unroll
            for (int k = 0; k < 2; ++k) {
                const unsigned o = (s0 + lane * 2u + k) * S, i = min(o >> 5, nw - 1u);
                const unsigned d0 = *ldsp<unsigned>(str + i * 4u), d1 = *ldsp<unsigned>(str + i * 4u + 4u);
                a[k] = o + 1u;
                x[k] = __builtin_amdgcn_alignbit(d0, i + 1u < nw ? d1 : 0u, ~o) & msk;
                if (o >= bits) x[k] = 0u;
                l[k] = x[k] ? a[k] + 31u - (unsigned)__builtin_ctz(x[k]) : 0u;
            }
            const unsigned c0 = __builtin_popcount(x[0]), c = c0 + __builtin_popcount(x[1]), ci = scan_add(c), li = scan_max(max(l[0], l[1]));
            const unsigned prev = max(before, dpp0<0x138, 0xf>(li));
            unsigned p = at + ci - c;
#pragma unroll
            for (int k = 0; k < 2; ++k) {
                unsigned w = x[k], carry = a[k] - (k ? max(prev, l[0]) : prev);
                while (w) {
                    const unsigned z = __builtin_clz(w), len = z + carry;
                    w = (w << z) + 0x80000000u; carry = 0u;
                    *ldsp<unsigned char>(p) = (unsigned char)(len + perm(0x020202fcu, 0x03030300u, min(len, 7u)));
                    ++p;
                }
            }
            at = min(at + last_lane(ci), esc + 4u + ESC_CAP); before = max(before, last_lane(li));
        }
    }
    u32x4 C[8], M[8]; unsigned cnt = 0;
#pragma unroll
    for (int it = 0; it < 8; ++it) if (!OUT || it * 64 < nch) {
        if (OUT) { h[it] = (h[it] | (h[it] << 12)) << 1; if (it * 64 + lane >= nch) h[it] = ~0u; }
#pragma unroll
        for (int q = 0; q < 4; ++q) {
            C[it][q] = OUT ? (L[it][q] & 0x01010101u) | ((h[it] >> q) & 0x02020202u) : (h[it] >> (2 * q)) & 0x03030303u;
            M[it][q] = perm(0u, 0x000000ffu, C[it][q]);
            cnt += __builtin_popcount(M[it][q]);
        }
    }
    cnt >>= 3;
    unsigned p = esc + 3u + scan_add(cnt) - cnt;
    fence();
#pragma unroll
    for (int it = 0; it < 8; ++it) if (!OUT || it * 64 < nch) {
        u32x4 x, y;
#pragma unroll
        for (int q = 0; q < 4; ++q) {
            const unsigned f = (M[it][q] & 0x01010101u) + (p & 3u), t = lshl_add<8>(f, f), sel = lshl_add<16>(t, t), ad = p & ~3u;
            const unsigned d0 = *ldsp<unsigned>(ad), d1 = *ldsp<unsigned>(ad + 4u);
            p = ad + (sel >> 24);
            const unsigned E = (perm(d1, d0, sel) & M[it][q]) | C[it][q], lo = OUT ? L[it][q] & 0xfefefefeu : L[it][q];
            const unsigned q0 = pk_rsub(T2, perm(E, lo, 0x05010400u)), q1 = pk_rsub(T2, perm(E, lo, 0x07030602u));
            const unsigned w0 = __builtin_amdgcn_alignbit(q0, q0, 1), w1 = __builtin_amdgcn_alignbit(q1, q1, 1);
            if (q < 2) { x[2 * q] = w0; x[2 * q + 1] = w1; } else { y[2 * q - 4] = w0; y[2 * q - 3] = w1; }
        }
        put(it, x, y);
    }
}

__device__ __forceinline__ void decode_in(unsigned info, unsigned cw, unsigned esc, char* dst, int lane) {
    decode_values<false>([&](int it, const u32x4& x, const u32x4& y) { st_wt(dst + it * 2048, x); st_wt(dst + it * 2048 + 1024, y); }, info, cw, esc, lane);
}
__device__ __forceinline__ void make_table(int n) {
    unsigned s[4], c = 0;
#pragma unroll
    for (int e = 0; e < 4; ++e) { const bool nz = (n >> e) & 1; s[e] = nz ? 0x0100u + c * 0x0202u : 0x0c0cu; c += nz; }
    *ldsp<u32x2>(LDS_TAB + n * 8) = (u32x2){s[0] | s[1] << 16, s[2] | s[3] << 16};
}
__device__ __forceinline__ void decode_out(unsigned info, unsigned cw, unsigned esc, char* outp, int lane) {
    const u32x4 mk = *ldsp<u32x4>(cw + lane * 16);
    decode_values<true>([&](int it, const u32x4& x, const u32x4& y) { *ldsp<u32x4>(cw + (it * 64 + lane) * 32) = x; *ldsp<u32x4>(cw + (it * 64 + lane) * 32 + 16) = y; }, info, cw, esc, lane);
    unsigned nl[16], nh[16], cl[16], st[8], base[16], run = cw;
#pragma unroll
    for (int k = 0; k < 8; ++k) {
        unsigned c[2];
#pragma unroll
        for (int j = 0; j < 2; ++j) {
            const int i = 2 * k + j;
            nl[i] = (((mk[i >> 2] & 0x0f0f0f0fu) << 3) >> (8 * (i & 3))) & 0xffu; nh[i] = (((mk[i >> 2] & 0xf0f0f0f0u) >> 1) >> (8 * (i & 3))) & 0xffu;
            cl[i] = __builtin_popcount(nl[i]); c[j] = cl[i] + __builtin_popcount(nh[i]);
        }
        const unsigned own = (c[0] | c[1] << 16) * 2u, inc = scan_add(own), t = last_lane(inc);
        st[k] = inc - own;
        base[2 * k] = run; run += t & 0xffffu; base[2 * k + 1] = run; run += t >> 16;
    }
    fence();
#pragma unroll
    for (int i = 0; i < 16; ++i) {
        const unsigned a0 = base[i] + ((i & 1) ? st[i >> 1] >> 16 : st[i >> 1] & 0xffffu), a1 = a0 + cl[i] * 2u;
        u32x4 o;
#pragma unroll
        for (int j = 0; j < 2; ++j) {
            const unsigned a = j ? a1 : a0, ad = a & ~3u;
            const unsigned d0 = *ldsp<unsigned>(ad), d1 = *ldsp<unsigned>(ad + 4u), d2 = *ldsp<unsigned>(ad + 8u);
            const u32x2 sel = *ldsp<u32x2>(LDS_TAB + (j ? nh[i] : nl[i]));
            const unsigned t0 = __builtin_amdgcn_alignbyte(d1, d0, a), t1 = __builtin_amdgcn_alignbyte(d2, d1, a);
            o[2 * j] = perm(t1, t0, sel.x); o[2 * j + 1] = perm(t1, t0, sel.y);
        }
        st_wt(outp + OUT_OFF(i), o);
    }
}

__device__ __forceinline__ void in_job(const Args& a, char* myws, int dest, int g, int kt, int j, int lane, unsigned seen, unsigned tail DBG_ARGS) {
    const int panel = g * 8 + ((j & 15) >> 1), half = j & 1, rank = a.rank;
    char* pq = a.ws[dest == rank ? (rank + 1 + lane) & 7 : dest];
    u32x4 v[16];
    if (j < 16) {
        const int sb = lane * 16, swz = sb ^ (((sb >> 9) & 1) << 5);
        const char* src = a.x + (size_t)(panel * 256 + half * 128 + (swz >> 6)) * (KS * 2) + kt * 128 + (swz & 63);
#pragma unroll
        for (int i = 0; i < 16; ++i) v[i] = *(const u32x4*)(src + (i >> 1) * 16 * (KS * 2) + (i & 1) * 64);
        __builtin_amdgcn_sched_barrier(0); wait_vm0();
    } else {
        const char* src = a.w + (size_t)(kt * 64 + (lane & 3) * 8) * (GN * 2) + (size_t)(dest * NB + panel * 256 + half * 128 + (lane >> 2) * 8) * 2;
        u32x4 r[16];
#pragma unroll
        for (int i = 0; i < 16; ++i) r[i] = *(const u32x4*)(src + (size_t)((i >> 3) * 32 + (i & 7)) * (GN * 2));
        __builtin_amdgcn_sched_barrier(0); wait_vm0();
#pragma unroll
        for (int kh = 0; kh < 2; ++kh)
#pragma unroll
            for (int i = 0; i < 8; ++i)
#pragma unroll
                for (int d = 0; d < 4; ++d) v[kh * 8 + i][d] = __builtin_amdgcn_perm(r[kh * 8 + 2 * d + 1][i >> 1], r[kh * 8 + 2 * d][i >> 1], (i & 1) ? 0x07060302u : 0x05040100u);
    }
    asm volatile("" : "+v"(pq));
    asm volatile("" : "+v"(seen));
    STAMPD(29);
    const unsigned call = seen + 1u;
    const int pos = g * 256 + kt * 32 + j;
    if (j < 16 || dest != rank) {
        char* dst = myws + (j < 16 ? OFF_XW : OFF_WW + (size_t)dest * (256 * 16384)) + (size_t)(g * 128 + kt * 16 + (j & 15)) * 16384 + lane * 16;
        GAS u64* const note = (GAS u64*)(pq + OFF_Q) + rank * NQ + pos;
        const unsigned info = encode_in(v, tail, dst, lane, [&]() {
            if (pos < 64) { wait_vm0(); if (lane < (j < 16 ? 7 : 1)) __hip_atomic_store(note + NPOS, (u64)call << 32, __ATOMIC_RELAXED, __HIP_MEMORY_SCOPE_SYSTEM); STAMPD(30); } });
        if (info & INFO_RAW) {
            wait_vm0();
#pragma unroll
            for (int i = 0; i < 16; ++i) st_wt(dst + i * 1024, v[i]);
        }
        wait_vm0();
        if (lane < (j < 16 ? 7 : 1)) __hip_atomic_store(note, ((u64)call << 32) | info, __ATOMIC_RELAXED, __HIP_MEMORY_SCOPE_SYSTEM);
        STAMPD(31);
    }
    if (dest == rank) {
        char* dst = myws + img_off(g, j, kt) + lane * 16;
#pragma unroll
        for (int i = 0; i < 16; ++i) st_wt(dst + i * 1024, v[i]);
        wait_vm0();
        if (lane == 0) __hip_atomic_store((GAS unsigned*)(myws + flag_off(g, j, kt)), call, __ATOMIC_RELAXED, __HIP_MEMORY_SCOPE_SYSTEM);
    }
}

__device__ __forceinline__ bool expired(unsigned* tmo, unsigned call, u64 t0) {
    if (__builtin_amdgcn_readfirstlane(__hip_atomic_load(tmo, __ATOMIC_RELAXED, __HIP_MEMORY_SCOPE_AGENT)) == call || now() - t0 > DEADLINE) {
        __hip_atomic_store(tmo, call, __ATOMIC_RELAXED, __HIP_MEMORY_SCOPE_AGENT);
        return true;
    }
    return false;
}

__device__ __forceinline__ void tile_of(int ti, int xcd, int& pm, int& pn) { pm = (ti >> 4) * 8 + (xcd & 1) * 4 + (ti & 3); pn = ((ti >> 3) & 1) * 8 + (xcd >> 1) * 2 + ((ti >> 2) & 1); }
__device__ __forceinline__ size_t out_off(int src, int pm, int pn, int w, int lane) { return (size_t)(pm * 256 + (w >> 2) * 64 + (lane >> 3)) * (GN * 2) + (size_t)(src * NB + pn * 256 + (w & 3) * 64) * 2 + (lane & 7) * 16; }

__global__ __launch_bounds__(512, 2) void gemm_ar_kernel(const Args a) {
    extern __shared__ __attribute__((aligned(16))) unsigned char shm[];
    LAS unsigned char* lds = (LAS unsigned char*)shm;
    const int tid = threadIdx.x, wid = __builtin_amdgcn_readfirstlane(tid >> 6), lane = tid & 63;
    const int b = blockIdx.x, xcd = b & 7, idx = b >> 3, rank = a.rank;
    char* const myws = a.myws;
    char* const myout = a.myout;
    const u64 t0 = now();
    STAMP(0);
    unsigned* const ctr = (unsigned*)(myws + OFF_CTR) + b * 32 + wid;
    unsigned* const tmo = (unsigned*)(myws + OFF_TMO);
    const unsigned seen = __hip_atomic_load(ctr, __ATOMIC_RELAXED, __HIP_MEMORY_SCOPE_AGENT);
    const unsigned cw = LDS_CW + wid * C_WAVE, tl = LDS_TL + wid * C_TAIL;

    const int pk = (idx - NG + 7 * 8 + xcd) % 7, peer = (rank + 1 + pk) & 7, ord = xcd * 2 + (idx >= NG + 7);
    if (idx >= NG) { if (tid < 2) *ldsp<unsigned>(LDS_CTL + tid * 4) = tid ? 0u : 8u; if (tid < 16) make_table(tid); wait_lds(); bar(); }
    {
        int q, qe, delay;
        if (idx < 8) { q = wid; qe = 4; delay = (wid || (idx & 6) != 2 ? DLY0 : 0) + wid * DLY1; }
        else if (idx < 10 || idx >= NG) { q = 0; qe = 0; delay = 0; }
        else { q = wid * 64 + (idx - 10) * 8 + xcd; qe = 2048; delay = q < 112 ? DLY3 : DLY0 + (q < 224 ? 0 : (wid - 2) * DLY2); }
        for (int i = 0; i < delay; ++i) __builtin_amdgcn_s_sleep(8);
        for (; q < qe; q += 512) {
            int r;
            if (idx < 8) r = wid * 64 + (idx ^ 2) * 8 + xcd;
            else {
                const int kt = q < 896 ? q / 112 : 8 + (q - 896) / 144, rem = q < 896 ? q % 112 : (q - 896) % 144 - 32;
                r = kt * 32 + (rem < 0 ? rem + 32 : (rem % 7 + 1) * 512 + 16 + rem / 7);
            }
            in_job(a, myws, (rank + (r >> 9)) & 7, (r >> 8) & 1, (r >> 5) & 7, r & 31, lane, seen, tl DBG_PASS);
        }
    }
    const unsigned call = __builtin_amdgcn_readfirstlane(seen) + 1u;
    if (lane == 0) __hip_atomic_store(ctr, call, __ATOMIC_RELAXED, __HIP_MEMORY_SCOPE_AGENT);
    STAMP(1);

    if (idx >= NG) {
        const char* const pws = a.ws[peer];
        const u64* const queue = (const u64*)(myws + OFF_Q) + peer * NQ;
        unsigned* const tq = (unsigned*)(myws + OFF_TICK) + pk * 64;
        const unsigned tbase = (call - 1u) * NPOS - 128u;
        const unsigned ctl = LDS_CTL;
        unsigned pos = wid * 16 + ord;
        bool bad = false;
        while (pos < (unsigned)NPOS) {
            const int g = pos >> 8, kt = (pos >> 5) & 7, j = pos & 31;
            const char* const src = pws + (pos >= NIN ? OFF_OW + (size_t)(pos - NIN) * 16384
                                           : (j < 16 ? OFF_XW : OFF_WW + (size_t)rank * (256 * 16384)) + (size_t)(g * 128 + kt * 16 + (j & 15)) * 16384) + lane * 16;
            auto token = [&]() {
                unsigned my = wid;
                if (lane == 0 && pos >= 128u) asm volatile("ds_add_rtn_u32 %0, %1, %2\n\ts_waitcnt lgkmcnt(0)" : "+v"(my) : "v"(ctl), "v"(1u) : "memory");
                my = __builtin_amdgcn_readfirstlane(my);
                for (;;) {
                    unsigned rel;
                    asm volatile("ds_read_b32 %0, %1 offset:4\n\ts_waitcnt lgkmcnt(0)" : "=v"(rel) : "v"(ctl) : "memory");
                    if ((int)(__builtin_amdgcn_readfirstlane(rel) + TOK - my) > 0) break;
                    if (now() - t0 > DEADLINE) break;
                    __builtin_amdgcn_s_sleep(1);
                }
            };
            unsigned info;
            int o = 0;
            for (unsigned at = pos < 64u ? pos + NPOS : pos, spin = 1;; ++spin) {
                const u64 e = __hip_atomic_load(queue + at, __ATOMIC_RELAXED, __HIP_MEMORY_SCOPE_SYSTEM);
                info = __builtin_amdgcn_readfirstlane((unsigned)e);
                if (__builtin_amdgcn_readfirstlane((unsigned)(e >> 32)) >= call) {
                    if (at == pos) break;
                    STAMP1(2);
                    token();
                    for (; o < 8192; o += 1024) { if (pos < 16u && o < EWN * 1024) wait_vm<EWIN - 1>(); else wait_vm<WIN - 1>(); __builtin_amdgcn_global_load_lds((const unsigned*)(src + o), ldsp<unsigned>(cw + o), 16, 0, 17); }
                    at = pos;
                    continue;
                }
                if ((spin & 15u) == 0 && expired(tmo, call, t0)) { bad = true; break; }
                if (at == pos) __builtin_amdgcn_s_sleep(POLL_SLEEP); else __builtin_amdgcn_s_sleep(POLL_FAST);
            }
            if (bad) break;
            LAP(24); STAMP1(2);
            if ((info & 1023u) > 512u || ((info >> 10) & 511u) - 1u >= (unsigned)NW_CAP) info = INFO_RAW;
            if (info & INFO_RAW) o = 0;
            const int wire = (info & INFO_RAW) ? 16384 : pos >= NIN ? wire_bytes<true>(info) : wire_bytes<false>(info);
            if (pos >= 64u) token();
            LAP(25); STAMP1(7);
            for (; o < wire; o += 1024) {
                if (o + lane * 16 < wire) __builtin_amdgcn_global_load_lds((const unsigned*)(src + o), ldsp<unsigned>(cw + o), 16, 0, 17);
                wait_vm<WIN - 1>();
            }
            wait_vm<REL>();
            if (lane == 0) asm volatile("ds_add_u32 %0, %1 offset:4" :: "v"(ctl), "v"(1u) : "memory");
            wait_vm0();
            LAP(26); STAMP1(3);
            int t = 0;
            char* dst;
            if (pos >= NIN) { int pm, pn; tile_of((pos - NIN) >> 6, ((pos - NIN) >> 3) & 7, pm, pn); dst = myout + out_off(peer, pm, pn, pos & 7, lane); }
            else { t = slot_of(peer, rank, kt); dst = myws + img_off(g, j, t) + lane * 16; }
            if (info & INFO_RAW) {
                u32x4 v[16];
#pragma unroll
                for (int i = 0; i < 16; ++i) v[i] = *ldsp<u32x4>(cw + i * 1024 + lane * 16);
#pragma unroll
                for (int i = 0; i < 16; ++i) st_wt(dst + (pos >= NIN ? OUT_OFF(i) : (size_t)i * 1024), v[i]);
            } else if (pos >= NIN) decode_out(info, cw, tl, dst, lane);
            else decode_in(info, cw, tl, dst, lane);
            LAP(27); STAMP1(4);
            unsigned nx = 0;
            if (lane == 0) nx = __hip_atomic_fetch_add(tq, 1u, __ATOMIC_RELAXED, __HIP_MEMORY_SCOPE_AGENT);
            wait_vm0();
            if (pos < NIN && lane == 0) __hip_atomic_store((unsigned*)(myws + flag_off(g, j, t)), call, __ATOMIC_RELAXED, __HIP_MEMORY_SCOPE_SYSTEM);
            LAP(28); STAMP(8 + (pos >> 8)); STAMP1(6);
            pos = __builtin_amdgcn_readfirstlane(nx) - tbase;
        }
        if (bad && lane == 0) *(GAS u16*)(myout + (size_t)rank * (NB * 2)) = 0x7fc0;
        STAMP(5);
        STAMP_FLUSH();
        return;
    }

    const int wr = wid >> 2, wc = wid & 3, fr = lane & 15, fq = lane >> 4;
    bool bad = false;
    const bool hm = idx < 4;
    unsigned voffA[2], voffB[2];
#pragma unroll
    for (int i = 0; i < 2; ++i) {
        int R, C; stage_rc(tid * 16 + i * 8192, R, C);
        voffA[i] = tid * 16 + (hm ? (idx & 1) * 8192 + i * 16384 : i * 8192);
        const int w_ = R >> 5, n_ = (R >> 4) & 1, q_ = (R & 15) >> 2, j_ = R & 3;
        voffB[i] = (w_ >> 1) * 16384 + (C >> 5) * 8192 + (4 * n_ + j_) * 1024 + ((w_ & 1) * 8 + q_) * 64 + ((C >> 3) & 3) * 16;
    }
    constexpr size_t kstep = 32768, hstepA = 16384, hstepB = 256;
    const unsigned ldsw = (unsigned)wid * 1024u;
    const int aoff = lds_byte(wr * 64 + fr, fq * 8), boff = lds_byte(wc * 32 + fr, fq * 8);
#define G_SA(b, h) (((b) * 2 + (h)) * HTB)
#define G_SB(b, h) ((4 + (b) * 2 + (h)) * HTB)
#define G_STAGE(bufoff, gbase, voff) do { _Pragma("unroll") for (int _i = 0; _i < 2; ++_i) \
        __builtin_amdgcn_global_load_lds((const unsigned*)((const char*)(gbase) + (voff)[_i]), (LAS unsigned*)(lds + (bufoff) + ldsw + _i * 8192), 16, 0, 17); } while (0)
#define G_LDA(dst, b, h) do { _Pragma("unroll") for (int m = 0; m < 4; ++m) _Pragma("unroll") for (int k = 0; k < 2; ++k) dst[m][k] = *(const LAS bf16x8*)(lds + G_SA(b, h) + aoff + m * 2048 + k * 1024); } while (0)
#define G_LDB(dst, b, h) do { _Pragma("unroll") for (int n = 0; n < 2; ++n) _Pragma("unroll") for (int k = 0; k < 2; ++k) dst[n][k] = *(const LAS bf16x8*)(lds + G_SB(b, h) + boff + n * 2048 + k * 1024); } while (0)
#define G_MMA(ai, bj, At, Bt) do { __builtin_amdgcn_s_setprio(1); _Pragma("unroll") for (int m = 0; m < 4; ++m) _Pragma("unroll") for (int n = 0; n < 2; ++n) _Pragma("unroll") for (int k = 0; k < 2; ++k) \
        acc[ai][bj][m][n] = __builtin_amdgcn_mfma_f32_16x16x32_bf16(Bt[n][k], At[m][k], acc[ai][bj][m][n], 0, 0, 0); __builtin_amdgcn_s_setprio(0); } while (0)
#define G_WAIT_V(n) asm volatile("s_waitcnt vmcnt(" #n ")" ::: "memory")
#define G_WAIT_L(n) asm volatile("s_waitcnt lgkmcnt(" #n ")" ::: "memory")
#define G_BAR __builtin_amdgcn_s_barrier()
#define G_SCHED __builtin_amdgcn_sched_barrier(0)
    for (int round = 0; round < 2; ++round) {
    const int ti = hm ? round * 8 + (idx >> 1) : round ? idx + 14 : idx < 10 ? idx - 2 : idx;
    int pm, pn; tile_of(ti, xcd, pm, pn);
    const char* const fX = myws + OFF_INF + (size_t)pm * 512 + lane * 8;
    const char* const fW = myws + OFF_INF + (size_t)(16 + pn) * 512 + lane * 8;
    auto poll = [&](int need) -> int {
        for (;;) {
            u32x2 fa, fb;
            asm volatile("global_load_dwordx2 %0, %2, off sc0 sc1\n\tglobal_load_dwordx2 %1, %3, off sc0 sc1\n\ts_waitcnt vmcnt(0)"
                         : "=&v"(fa), "=&v"(fb) : "v"(fX), "v"(fW) : "memory");
            const u64 miss = __builtin_amdgcn_ballot_w64(fa.x < call || fa.y < call || fb.x < call || fb.y < call);
            const int upto = miss ? __builtin_ctzll(miss) : 64;
            if (upto >= need) return upto;
            if (expired(tmo, call, t0)) { bad = true; return 64; }
            __builtin_amdgcn_s_sleep(4);
        }
    };
    f32x4 acc[2][2][4][2];
#pragma unroll
    for (int x = 0; x < 2; ++x)
#pragma unroll
        for (int y = 0; y < 2; ++y)
#pragma unroll
            for (int m = 0; m < 4; ++m)
#pragma unroll
                for (int n = 0; n < 2; ++n) acc[x][y][m][n] = (f32x4){0.f, 0.f, 0.f, 0.f};
    bf16x8 At[4][2], B0[2][2], B1[2][2];
    const char* const cA = myws + OFF_XI + (size_t)pm * 64 * kstep;
    const char* const cB = myws + OFF_WI + (size_t)pn * 64 * kstep;
    int ready = poll(2);
    STAMP1(2);
    wait_vm0();
    if (hm) {
        G_STAGE(0, cA, voffA); G_STAGE(HTB, cB, voffB); G_STAGE(2 * HTB, cB + hstepB, voffB);
        G_STAGE(3 * HTB, cA + kstep, voffA); G_STAGE(4 * HTB, cB + kstep, voffB); G_STAGE(5 * HTB, cB + kstep + hstepB, voffB);
        if (wr == 1) G_BAR;
        G_WAIT_V(6); G_BAR; G_BAR;
        unsigned cur = 0, nxt = 6 * HTB;
        for (int t = 0; t < 64; ++t) {
            if (t < 62 && ready < t + 3) ready = poll(t + 3);
            const size_t k2 = t < 62 ? (size_t)(t + 2) * kstep : 0;
            LAS unsigned char* const la = lds + cur + aoff; LAS unsigned char* const lb = lds + cur + boff;
#pragma unroll
            for (int n = 0; n < 2; ++n)
#pragma unroll
                for (int k = 0; k < 2; ++k) B0[n][k] = *(const LAS bf16x8*)(lb + HTB + n * 2048 + k * 1024);
#pragma unroll
            for (int m = 0; m < 4; ++m)
#pragma unroll
                for (int k = 0; k < 2; ++k) At[m][k] = *(const LAS bf16x8*)(la + m * 2048 + k * 1024);
            G_STAGE(nxt, cA + k2, voffA); G_STAGE(nxt + HTB, cB + k2, voffB);
            G_WAIT_V(10); G_BAR; G_WAIT_L(0); G_MMA(0, 0, At, B0); G_BAR; G_SCHED;
#pragma unroll
            for (int n = 0; n < 2; ++n)
#pragma unroll
                for (int k = 0; k < 2; ++k) B1[n][k] = *(const LAS bf16x8*)(lb + 2 * HTB + n * 2048 + k * 1024);
            G_STAGE(nxt + 2 * HTB, cB + k2 + hstepB, voffB);
            G_WAIT_V(8); G_BAR; G_WAIT_L(0); G_MMA(0, 1, At, B1); G_BAR; G_SCHED;
            nxt = cur; cur = cur == 6 * HTB ? 0 : cur + 3 * HTB;
        }
    } else {
    G_STAGE(G_SB(0, 0), cB, voffB); G_STAGE(G_SA(0, 0), cA, voffA); G_STAGE(G_SB(0, 1), cB + hstepB, voffB); G_STAGE(G_SA(0, 1), cA + hstepA, voffA);
    if (wr == 1) G_BAR;
    G_WAIT_V(4); G_BAR;
    G_STAGE(G_SB(1, 0), cB + kstep, voffB); G_STAGE(G_SA(1, 0), cA + kstep, voffA); G_STAGE(G_SB(1, 1), cB + hstepB + kstep, voffB);
    G_WAIT_V(6); G_BAR;
    for (int t = 0; t < 64; t += 2) {
        const bool last = (t == 62);
        if (!last && ready < t + 4) ready = poll(t + 4);
        const char* a1 = cA + (size_t)(t + 1) * kstep;
        const char* a2 = last ? cA : cA + (size_t)(t + 2) * kstep; const char* b2 = last ? cB : cB + (size_t)(t + 2) * kstep;
        const char* a3 = a2 + kstep; const char* b3 = b2 + kstep;
        G_LDB(B0, 0, 0); G_SCHED; G_LDA(At, 0, 0); G_STAGE(G_SA(1, 1), a1 + hstepA, voffA);
        G_WAIT_L(8); G_BAR; G_WAIT_L(0); G_MMA(0, 0, At, B0); G_BAR; G_SCHED;
        G_LDB(B1, 0, 1); G_STAGE(G_SB(0, 0), b2, voffB);
        G_BAR; G_WAIT_L(0); G_MMA(0, 1, At, B1); G_BAR;
        G_LDA(At, 0, 1); G_STAGE(G_SA(0, 0), a2, voffA);
        G_BAR; G_WAIT_L(0); G_MMA(1, 0, At, B0); G_BAR; G_SCHED;
        G_STAGE(G_SB(0, 1), b2 + hstepB, voffB);
        G_WAIT_V(6); G_BAR; G_MMA(1, 1, At, B1); G_BAR;
        G_LDB(B0, 1, 0); G_SCHED; G_LDA(At, 1, 0); G_STAGE(G_SA(0, 1), a2 + hstepA, voffA);
        G_WAIT_L(8); G_BAR; G_WAIT_L(0); G_MMA(0, 0, At, B0); G_BAR; G_SCHED;
        G_LDB(B1, 1, 1); G_STAGE(G_SB(1, 0), b3, voffB);
        G_BAR; G_WAIT_L(0); G_MMA(0, 1, At, B1); G_BAR;
        G_LDA(At, 1, 1); G_STAGE(G_SA(1, 0), a3, voffA);
        G_BAR; G_WAIT_L(0); G_MMA(1, 0, At, B0); G_BAR; G_SCHED;
        G_STAGE(G_SB(1, 1), b3 + hstepB, voffB);
        G_WAIT_V(6); G_BAR; G_MMA(1, 1, At, B1); G_BAR;
    }
    }
    G_WAIT_V(0);
    if (wr == 0) G_BAR;
    G_BAR;
    STAMP(round ? 6 : 3);

    int ln = lane;
    asm volatile("" : "+v"(ln) :: "memory");
    char* pq = a.ws[(rank + 1 + (ln >> 3)) & 7];
    if (bad) acc[0][0][0][0][0] = __builtin_nanf("");
    const unsigned cx = hm ? LDS_CW + ((idx & 1) * 4 + wc) * C_WAVE + wr * 8192 : cw;
#pragma unroll
    for (int ai = 0; ai < 2; ++ai) if (ai == 0 || !hm)
#pragma unroll
        for (int m = 0; m < 4; ++m)
#pragma unroll
            for (int bj = 0; bj < 2; ++bj) {
                const f32x4 v0 = acc[ai][bj][m][0], v1 = acc[ai][bj][m][1];
                u32x4 w; w[0] = relu_pk(cvt_pk_bf16(v0[0], v0[1])); w[1] = relu_pk(cvt_pk_bf16(v0[2], v0[3]));
                w[2] = relu_pk(cvt_pk_bf16(v1[0], v1[1])); w[3] = relu_pk(cvt_pk_bf16(v1[2], v1[3]));
                *ldsp<u32x4>(cx + (ai * 64 + m * 16 + (ln & 15)) * 128 + (((bj * 4 + (ln >> 4)) ^ (ln & 7)) << 4)) = w;
            }
    asm volatile("" ::: "memory");
    if (hm) { wait_lds(); bar(); }
    if (!hm || wr == (idx & 1)) {
    u32x4 W[16];
#pragma unroll
    for (int i = 0; i < 16; ++i) W[i] = *ldsp<u32x4>(cw + i * 1024 + (ln >> 3) * 128 + (((ln & 7) ^ (ln >> 3)) << 4));
    wait_lds(); wait_vm0();
    asm volatile("" : "+v"(pq));
    char* const outp = myout + out_off(rank, pm, pn, wid, ln);
#pragma unroll
    for (int i = 0; i < 16; ++i) st_wt(outp + OUT_OFF(i), W[i]);
    char* const slot = myws + OFF_OW + (size_t)((ti * 8 + xcd) * 8 + wid) * 16384 + ln * 16;
    const unsigned info = encode_out(W, cw, tl, slot, ln);
    if (info & INFO_RAW) {
#pragma unroll
        for (int i = 0; i < 16; ++i) st_wt(slot + i * 1024, W[i]);
    }
    if (ln == 0) *(LAS unsigned*)(lds + LDS_CTL + wid * 4) = info;
    }
    wait_lds();
    wait_vm0();
    bar();
    if (wid == 0 && ln < 56 && (!hm || ((ln >> 2) & 1) == (idx & 1))) {
        const unsigned inf = *(const LAS unsigned*)(lds + LDS_CTL + (ln & 7) * 4);
        __hip_atomic_store((GAS u64*)(pq + OFF_Q) + rank * NQ + NIN + (ti * 8 + xcd) * 8 + (ln & 7), ((u64)call << 32) | inf, __ATOMIC_RELAXED, __HIP_MEMORY_SCOPE_SYSTEM);
    }
    STAMP(round ? 7 : 4);
    }
    STAMP(5);
    STAMP_FLUSH();
}

extern "C" void kernel_launch(int rank, int nranks, ncclComm_t comm,
                              hipStream_t stream,
                              void* const* d_in, const int* in_sizes, int n_in,
                              void* d_out, int out_size,
                              const int* in_dtypes, int out_dtype,
                              void* const* d_out_all,
                              void* const* d_ws_all, size_t ws_bytes) {
    if (nranks != NR || (unsigned)rank >= (unsigned)NR || ws_bytes < WS_NEED || n_in < 2) { fprintf(stderr, "unsupported configuration\n"); return; }
    static bool attr[NR] = {};
    if (!attr[rank]) { HIP_CHECK(hipFuncSetAttribute((const void*)gemm_ar_kernel, hipFuncAttributeMaxDynamicSharedMemorySize, LDS_BYTES)); attr[rank] = true; }
    Args a;
    a.x = (const char*)d_in[0]; a.w = (const char*)d_in[1]; a.rank = rank;
    for (int p = 0; p < NR; ++p) a.ws[p] = (char*)d_ws_all[p];
    a.myws = a.ws[rank]; a.myout = (char*)d_out;
    hipLaunchKernelGGL(gemm_ar_kernel, dim3(256), dim3(512), LDS_BYTES, stream, a);
    HIP_CHECK(hipGetLastError());
}
```

```cpp
#include <hip/hip_runtime.h>
#include <rccl/rccl.h>
#include <cstdio>

#define HIP_CHECK(expr) do { hipError_t _e = (expr); if (_e != hipSuccess) { \
    fprintf(stderr, "HIP error %s at %s:%d\n", hipGetErrorString(_e), __FILE__, __LINE__); \
    return; } } while (0)

#define LAS __attribute__((address_space(3)))
#define GAS __attribute__((address_space(1)))
typedef unsigned short u16;
typedef unsigned long long u64;
typedef short bf16x8 __attribute__((ext_vector_type(8)));
typedef float f32x4 __attribute__((ext_vector_type(4)));
typedef unsigned u32x4 __attribute__((ext_vector_type(4)));
typedef unsigned u32x2 __attribute__((ext_vector_type(2)));

constexpr int NR = 8;
constexpr int GM = 4096, GN = 32768, KS = 512, NB = 4096;
constexpr int HTB = 16384;
constexpr int NG = 18;
constexpr int NIN = 512, NPOS = 2560;
constexpr int NQ = NPOS + 64;
constexpr unsigned LDS_TAB = 64, LDS_CW = 256, C_WAVE = 16384 + 128, LDS_TL = LDS_CW + 8 * C_WAVE, C_TAIL = 3584, ESC_CAP = 3560;
constexpr unsigned LDS_CTL = LDS_TL + 8 * C_TAIL + 64;
constexpr int LDS_BYTES = LDS_CTL + 64;
static_assert(LDS_TL >= 8 * HTB && LDS_CTL >= 9 * HTB && LDS_BYTES <= 163840, "LDS layout");
constexpr size_t OFF_CTR = 0;
constexpr size_t OFF_TMO = 32768;
constexpr size_t OFF_TICK = 32768 + 256;
constexpr size_t OFF_STAMP = 65536;
constexpr size_t OFF_INF = 512u << 10;
constexpr size_t OFF_Q = 1u << 20;
constexpr size_t OFF_XI = 4u << 20;
constexpr size_t OFF_WI = 36u << 20;
constexpr size_t OFF_XW = 68u << 20;
constexpr size_t OFF_WW = 72u << 20;
constexpr size_t OFF_OW = 104u << 20;
constexpr size_t WS_NEED = 136u << 20;
constexpr u64 DEADLINE = 50000000ull;

#define TOK 2
#define WIN 2
#define REL (WIN - 1)
#define DLY1 3
#define DLY2 5
#define POLL_SLEEP 6
#define DLY0 4
#define EARLY_AT 3
#define ORD 0
#define PEEK 1
#define EARLY(pos) (((pos) & (ORD ? ~63 : ~47)) == 0)
#define AHEAD 2
#define DLY3 1
#define POLL_FAST 1
#define WIRE_ALIGN 64
struct Args { const char* x; const char* w; char* myws; char* myout; int rank; char* ws[NR]; };

__device__ __forceinline__ int lds_byte(int r, int c) { const int st = (r >> 4) * 2 + (c >> 5), rr = r & 15, cc = c & 31, ob = rr * 64 + cc * 2; return st * 1024 + (ob ^ (((ob >> 9) & 1) << 5)); }
__device__ __forceinline__ void stage_rc(int b, int& R, int& C) { const int st = b / 1024, sb = b % 1024, swz = sb ^ (((sb >> 9) & 1) << 5); R = (st >> 1) * 16 + swz / 64; C = (st & 1) * 32 + (swz % 64) / 2; }
__device__ __forceinline__ unsigned cvt_pk_bf16(float lo, float hi) { unsigned r; asm volatile("v_cvt_pk_bf16_f32 %0, %1, %2" : "=v"(r) : "v"(lo), "v"(hi)); return r; }
__device__ __forceinline__ unsigned relu_pk(unsigned v) { unsigned r; asm("v_pk_max_i16 %0, %1, 0" : "=v"(r) : "v"(v)); return r; }
__device__ __forceinline__ void st_wt(char* p, u32x4 v) { asm volatile("global_store_dwordx4 %0, %1, off sc0 sc1\n\ts_nop 1" :: "v"(p), "v"(v) : "memory"); }
__device__ __forceinline__ void wait_vm0() { __builtin_amdgcn_s_waitcnt(0x0F70); asm volatile("s_waitcnt vmcnt(0)" ::: "memory"); }
template <int N> __device__ __forceinline__ void wait_vm() { asm volatile("s_waitcnt vmcnt(%0)" :: "n"(N) : "memory"); }
__device__ __forceinline__ void bar() { asm volatile("" ::: "memory"); __builtin_amdgcn_s_barrier(); asm volatile("" ::: "memory"); }
__device__ __forceinline__ u64 now() { return __builtin_amdgcn_s_memrealtime(); }

#define STAMP(i) do { } while (0)
#define STAMP1(i) do { } while (0)
#define STAMPD(i) do { } while (0)
#define SSTAMP(i) do { } while (0)
#define DBG_ARGS
#define DBG_PASS
#define LAP(i) do { } while (0)
#define STAMP_FLUSH() do { } while (0)

__device__ __forceinline__ int slot_of(int s, int p, int kt) { return s == p ? kt : 8 + 7 * kt + ((s - p - 1) & 7); }
__device__ __forceinline__ size_t img_off(int g, int j, int t) { return (j < 16 ? OFF_XI : OFF_WI) + (size_t)((g * 8 + ((j & 15) >> 1)) * 64 + t) * 32768 + (j & 1) * 16384; }
__device__ __forceinline__ size_t flag_off(int g, int j, int t) { return OFF_INF + (size_t)((((j >> 4) * 16 + g * 8 + ((j & 15) >> 1)) * 64 + t) * 2 + (j & 1)) * 4; }

#define OUT_OFF(i) ((size_t)(((i) >> 3) * 128 + ((i) & 7) * 8) * (GN * 2))
constexpr unsigned INFO_RAW = 1u << 31;
constexpr int NW_CAP = 384;
template <bool OUT> __device__ __forceinline__ int cls_off(int nch) { return OUT ? 1024 + nch * 16 : 8192; }
template <bool OUT> __device__ __forceinline__ int cls_bytes(int nch) { return OUT ? (nch * 2 + 3) & ~3 : 2048; }
template <bool OUT> __device__ __forceinline__ int wire_bytes(unsigned info) {
    const int nch = info & 1023; return (cls_off<OUT>(nch) + cls_bytes<OUT>(nch) + (int)((info >> 10) & 511) * 4 + (WIRE_ALIGN - 1)) & ~(WIRE_ALIGN - 1); }
__device__ __forceinline__ unsigned pk_min_u16(unsigned x, unsigned y) { unsigned r; asm("v_pk_min_u16 %0, %1, %2" : "=v"(r) : "v"(x), "s"(y)); return r; }
__device__ __forceinline__ unsigned pk_rsub(unsigned s, unsigned y) { unsigned r; asm("v_pk_sub_u16 %0, %1, %2" : "=v"(r) : "s"(s), "v"(y)); return r; }
__device__ __forceinline__ unsigned pk_half(unsigned x) { unsigned r; asm("v_pk_lshrrev_b16 %0, 1, %1 op_sel_hi:[0,1]" : "=v"(r) : "v"(x)); return r; }
__device__ __forceinline__ unsigned sum_u16(unsigned x, unsigned acc) { unsigned r; asm("v_sad_u16 %0, %1, 0, %2" : "=v"(r) : "v"(x), "v"(acc)); return r; }
template <int S> __device__ __forceinline__ unsigned lshl_add(unsigned x, unsigned y) { unsigned r; asm("v_lshl_add_u32 %0, %1, %2, %3" : "=v"(r) : "v"(x), "n"(S), "v"(y)); return r; }
template <int S> __device__ __forceinline__ unsigned lshl_or(unsigned x, unsigned y) { unsigned r; asm("v_lshl_or_b32 %0, %1, %2, %3" : "=v"(r) : "v"(x), "n"(S), "v"(y)); return r; }
__device__ __forceinline__ unsigned perm(unsigned hi, unsigned lo, unsigned sel) { return __builtin_amdgcn_perm(hi, lo, sel); }
__device__ __forceinline__ void wait_lds() { asm volatile("s_waitcnt lgkmcnt(0)" ::: "memory"); }
__device__ __forceinline__ void fence() { asm volatile("" ::: "memory"); }
template <int C, int RM> __device__ __forceinline__ unsigned dpp0(unsigned x) { return (unsigned)__builtin_amdgcn_update_dpp(0, (int)x, C, RM, 0xf, false); }
__device__ __forceinline__ unsigned scan_add(unsigned x) {
    x += dpp0<0x111, 0xf>(x); x += dpp0<0x112, 0xf>(x); x += dpp0<0x114, 0xf>(x); x += dpp0<0x118, 0xf>(x); x += dpp0<0x142, 0xa>(x); x += dpp0<0x143, 0xc>(x); return x; }
__device__ __forceinline__ unsigned scan_max(unsigned x) {
    x = max(x, dpp0<0x111, 0xf>(x)); x = max(x, dpp0<0x112, 0xf>(x)); x = max(x, dpp0<0x114, 0xf>(x)); x = max(x, dpp0<0x118, 0xf>(x)); x = max(x, dpp0<0x142, 0xa>(x)); x = max(x, dpp0<0x143, 0xc>(x)); return x; }
__device__ __forceinline__ unsigned last_lane(unsigned x) { return (unsigned)__builtin_amdgcn_readlane((int)x, 63); }
template <class T> __device__ __forceinline__ LAS T* ldsp(unsigned a) { return (LAS T*)(size_t)a; }

template <bool OUT, class GET, class MID>
__device__ __forceinline__ unsigned encode_values(GET get, MID mid, int nch, unsigned T, const u32x4& mk, unsigned tail, char* dst, int lane) {
    const unsigned T2 = T * 0x10001u;
    u32x4 R0[8], R1[8];
    if (!OUT) {
#pragma unroll
        for (int it = 0; it < 8; ++it) {
            u32x4 x, y, lo; get(it, x, y);
#pragma unroll
            for (int q = 0; q < 4; ++q) {
                const unsigned w0 = q < 2 ? x[2 * q] : y[2 * q - 4], w1 = q < 2 ? x[2 * q + 1] : y[2 * q - 3];
                R0[it][q] = pk_rsub(T2, __builtin_amdgcn_alignbit(w0, w0, 31)); R1[it][q] = pk_rsub(T2, __builtin_amdgcn_alignbit(w1, w1, 31));
                lo[q] = perm(R1[it][q], R0[it][q], 0x06040200u);
            }
            st_wt(dst + it * 1024, lo);
        }
    }
#pragma unroll
    for (int i = 0; i < 4; ++i) if (i < 3 || lane < 32) *ldsp<u32x4>(tail + i * 1024 + lane * 16) = (u32x4){0u, 0u, 0u, 0u};
    fence();
    u32x4 Lb[8], Ln[8]; unsigned tot = 0, direct = 0;
#pragma unroll
    for (int it = 0; it < 8; ++it) if (!OUT || it * 64 < nch) {
        u32x4 x, y; if (OUT) get(it, x, y);
        if (it == EARLY_AT) mid();
        unsigned h = 0, h0 = 0;
#pragma unroll
        for (int q = 0; q < 4; ++q) {
            const unsigned w0 = q < 2 ? x[2 * q] : y[2 * q - 4], w1 = q < 2 ? x[2 * q + 1] : y[2 * q - 3];
            const unsigned r0 = OUT ? pk_rsub(T2, w0 << 1) : R0[it][q], r1 = OUT ? pk_rsub(T2, w1 << 1) : R1[it][q];
            const unsigned ec = perm(pk_min_u16(r1, 0x07ff07ffu), pk_min_u16(r0, 0x07ff07ffu), 0x07050301u);
            Ln[it][q] = perm(r1, r0, 0x07050301u) + perm(0x01000000u, 0x00010207u, ec) - 0x03030303u;
            tot = __builtin_amdgcn_sad_u8(Ln[it][q], 0u, tot);
            if (OUT) { const unsigned b0 = perm(0u, 0x01000100u, ec); Lb[it][q] = perm(r1, r0, 0x06040200u) | b0; h0 |= b0 << q; h |= perm(0u, 0x01010000u, ec) << q; }
            else h |= perm(0u, 0x03020100u, ec) << (2 * q);
        }
        if (OUT) { direct += __builtin_popcount(h | h0); if (it * 64 + lane < nch) *ldsp<u16>(tail + (it * 64 + lane) * 2) = (u16)(h | (h >> 12)); }
        else { direct += __builtin_popcount((h | (h >> 1)) & 0x55555555u); *ldsp<unsigned>(tail + (it * 64 + lane) * 4) = h; }
    }
    const unsigned inc = scan_add(tot), nw = (last_lane(inc) + 33u) >> 5;
    const unsigned nesc = (OUT ? (unsigned)((nch + 63) >> 6) * 1024u : 8192u) - last_lane(scan_add(direct));
    if (nw > (unsigned)NW_CAP || nesc > (unsigned)ESC_CAP) return INFO_RAW;
    if (OUT) st_wt(dst, mk);
    unsigned pos = (tail + cls_bytes<OUT>(nch)) * 8u + 1u + inc - tot;
#pragma unroll
    for (int it = 0; it < 8; ++it) if (!OUT || it * 64 < nch) {
        if (OUT && it * 64 + lane < nch) st_wt(dst + 1024 + it * 1024, Lb[it]);
#pragma unroll
        for (int e = 0; e < 16; ++e) {
            pos += (Ln[it][e >> 2] >> (8 * (e & 3))) & 0xffu;
            asm volatile("ds_or_b32 %0, %1" :: "v"((pos >> 3) & ~3u), "v"(0x80000000u >> (pos & 31u)) : "memory");
        }
    }
    const unsigned info = (T >> 4) << 19 | nw << 10 | (OUT ? (unsigned)nch : 0u);
    const int rest = wire_bytes<OUT>(info) - cls_off<OUT>(nch);
    wait_lds();
#pragma unroll
    for (int i = 0; i < 4; ++i) if (i * 1024 < rest) { if (i * 1024 + lane * 16 < rest) st_wt(dst + cls_off<OUT>(nch) + i * 1024, *ldsp<u32x4>(tail + i * 1024 + lane * 16)); }
    return info;
}
__device__ __forceinline__ unsigned top_of(unsigned sum2, unsigned n) {
    return ((unsigned)__builtin_amdgcn_readfirstlane((int)((float)sum2 * __builtin_amdgcn_rcpf((float)n))) + 2u * (276u + 128u) + 8u) & 0xfff0u; }

template <class MID> __device__ __forceinline__ unsigned encode_in(const u32x4 (&v)[16], unsigned tail, char* dst, int lane, MID mid) {
    unsigned s = 0;
#pragma unroll
    for (int i = 0; i < 16; ++i) s = sum_u16(__builtin_amdgcn_alignbit(v[i][i & 3], v[i][i & 3], 31), s);
    return encode_values<false>([&](int it, u32x4& x, u32x4& y) { x = v[2 * it]; y = v[2 * it + 1]; }, mid, 512, top_of(last_lane(scan_add(s)), 2048u), v[0], tail, dst, lane);
}

__device__ __forceinline__ unsigned encode_out(const u32x4 (&W)[16], unsigned cw, unsigned tail, char* dst, int lane) {
    u32x4 mk; unsigned s = 0;
#pragma unroll
    for (int j = 0; j < 4; ++j) {
        unsigned t[2] = {0u, 0u};
#pragma unroll
        for (int d = 15; d >= 0; --d) { const unsigned w = W[j * 4 + (d >> 2)][d & 3]; t[d >> 3] = lshl_or<2>(t[d >> 3], pk_min_u16(w, 0x00010001u)); if (j == 0) s = sum_u16(w, s); }
        mk[j] = perm(t[1], t[0], 0x05040100u) | (perm(t[1], t[0], 0x07060302u) << 1);
    }
    unsigned end[8], run = cw;
    unsigned base[16];
#pragma unroll
    for (int k = 0; k < 8; ++k) {
        const unsigned m = mk[k >> 1] >> (16 * (k & 1));
        end[k] = scan_add(((unsigned)__builtin_popcount(m & 0xffu) | (unsigned)__builtin_popcount(m & 0xff00u) << 16) * 2u);
    }
#pragma unroll
    for (int k = 0; k < 8; ++k) { const unsigned t = last_lane(end[k]); base[2 * k] = run; run += t & 0xffffu; base[2 * k + 1] = run; run += t >> 16; }
    const unsigned n = (run - cw) >> 1;
    const int nch = (int)(n + 15u) >> 4;
    const unsigned T = top_of(4u * last_lane(scan_add(s)), max(base[4] - cw, 1u));
    {
        const unsigned pad = (((T - 0x200u) >> 1) & 0x7fffu) * 0x10001u, at = cw + (min(n, 8191u) >> 10) * 2048u + lane * 16;
        *ldsp<u32x4>(at) = (u32x4){pad, pad, pad, pad}; *ldsp<u32x4>(at + 1024) = (u32x4){pad, pad, pad, pad};
    }
    fence();
#pragma unroll
    for (int i = 0; i < 16; ++i) {
        unsigned a = base[i] + ((i & 1) ? end[i >> 1] >> 16 : end[i >> 1] & 0xffffu);
#pragma unroll
        for (int e = 7; e >= 0; --e) {
            const unsigned nz = (unsigned)__builtin_amdgcn_sbfe((int)mk[i >> 2], (i & 3) * 8 + e, 1);
            a = lshl_add<1>(nz, a);
            if (e & 1) asm volatile("ds_write_b16_d16_hi %0, %1" :: "v"(a & nz), "v"(W[i][e >> 1]) : "memory");
            else asm volatile("ds_write_b16 %0, %1" :: "v"(a & nz), "v"(W[i][e >> 1]) : "memory");
        }
    }
    wait_lds();
    return encode_values<true>([&](int it, u32x4& x, u32x4& y) { x = *ldsp<u32x4>(cw + (it * 64 + lane) * 32); y = *ldsp<u32x4>(cw + (it * 64 + lane) * 32 + 16); },
                               []() {}, nch, T, mk, tail, dst, lane);
}

template <int O> __device__ __forceinline__ void unary_step(unsigned& w, unsigned p, unsigned carry) {
    unsigned z; asm("v_ffbh_u32 %0, %1" : "=v"(z) : "v"(w));
    w <<= z & 31u; z += carry;
    asm volatile("ds_write_b8 %0, %1 offset:%2" :: "v"(min(p, w)), "v"(z + perm(0x03030303u, 0xfd040404u, min(z, 7u))), "n"(O));
    w <<= 1;
}
template <bool OUT, class PUT>
__device__ __forceinline__ void decode_values(PUT put, unsigned info, unsigned cw, unsigned esc, int lane DBG_ARGS) {
    const int nch = OUT ? (int)(info & 1023u) : 512;
    const unsigned T2 = ((info >> 19) & 0xfffu) * 0x100010u + (OUT ? 0x10001u : 0u);
    const bool five = !OUT || nch > 256, eight = !OUT || nch > 320;
    u32x4 L[8]; unsigned h[8];
#pragma unroll
    for (int it = 0; it < 8; ++it) if (it < 4 || (it < 5 ? five : eight)) {
        L[it] = *ldsp<u32x4>(cw + (OUT ? 1024 : 0) + (it * 64 + lane) * 16);
        h[it] = OUT ? *ldsp<u16>(cw + cls_off<OUT>(nch) + (it * 64 + lane) * 2) : *ldsp<unsigned>(cw + cls_off<OUT>(nch) + (it * 64 + lane) * 4);
    }
    {
        const unsigned str = cw + cls_off<OUT>(nch) + cls_bytes<OUT>(nch), nw = (info >> 10) & 511u, bits = nw * 32u - 2u;
        const unsigned lg = nw > 248u ? 9u : nw > 124u ? 8u : 7u, S = (bits + (1u << lg) - 1u) >> lg, msk = ((1u << S) - 1u) << (31u - S);
        unsigned at = esc + 4u, before = 1u;
        for (unsigned s0 = 0; s0 < (1u << lg); s0 += 128u) {
            unsigned x[2], a[2], l[2];
#pragma unroll
            for (int k = 0; k < 2; ++k) {
                const unsigned o = (s0 + lane * 2u + k) * S, i = min(o >> 5, nw - 1u);
                const unsigned d0 = *ldsp<unsigned>(str + i * 4u), d1 = *ldsp<unsigned>(str + i * 4u + 4u);
                a[k] = o + 1u;
                x[k] = __builtin_amdgcn_alignbit(d0, i + 1u < nw ? d1 : 0u, ~o) & msk;
                if (o >= bits) x[k] = 0u;
                l[k] = x[k] ? a[k] + 31u - (unsigned)__builtin_ctz(x[k]) : 0u;
            }
            const unsigned c0 = __builtin_popcount(x[0]), c = c0 + __builtin_popcount(x[1]), ci = scan_add(c), li = scan_max(max(l[0], l[1]));
            const unsigned prev = max(before, dpp0<0x138, 0xf>(li));
            unsigned p0 = at + ci - c, p1 = p0 + c0, w0 = x[0] << 1, w1 = x[1] << 1;
            unary_step<0>(w0, p0, a[0] - prev); unary_step<0>(w1, p1, a[1] - max(prev, l[0]));
            do {
                unary_step<1>(w0, p0, 0u); unary_step<1>(w1, p1, 0u); unary_step<2>(w0, p0, 0u); unary_step<2>(w1, p1, 0u); unary_step<3>(w0, p0, 0u); unary_step<3>(w1, p1, 0u);
                p0 += 3u; p1 += 3u;
            } while (__builtin_amdgcn_ballot_w64((w0 | w1) != 0u));
            at = min(at + last_lane(ci), esc + 4u + ESC_CAP); before = max(before, last_lane(li));
        }
    }
    SSTAMP(21);
    u32x4 C[8], S[8], P[8], D0[8], D1[8]; unsigned cnt = 0;
#pragma unroll
    for (int it = 0; it < 8; ++it) if (it < 4 || (it < 5 ? five : eight)) {
        if (OUT) { h[it] = (h[it] | (h[it] << 12)) << 1; if (it * 64 + lane >= nch) h[it] = ~0u; }
#pragma unroll
        for (int q = 0; q < 4; ++q) {
            C[it][q] = OUT ? (L[it][q] & 0x01010101u) | ((h[it] >> q) & 0x02020202u) : (h[it] >> (2 * q)) & 0x03030303u;
            const unsigned m = perm(0u, 0x000000ffu, C[it][q]), f = m & 0x01010101u, t = lshl_add<8>(f, f), u = lshl_add<16>(t, t);
            S[it][q] = ((u - f) & m) | (0x07060504u & ~m);
            P[it][q] = cnt; cnt += u >> 24;
        }
    }
    const unsigned p = esc + 4u + scan_add(cnt) - cnt;
    fence();
    auto fetch = [&](int it) {
#pragma unroll
        for (int q = 0; q < 4; ++q) { P[it][q] += p; const unsigned ad = P[it][q] & ~3u; D0[it][q] = *ldsp<unsigned>(ad); D1[it][q] = *ldsp<unsigned>(ad + 4u); }
    };
    auto finish = [&](int it) {
        u32x4 x, y;
#pragma unroll
        for (int q = 0; q < 4; ++q) {
            const unsigned E = perm(C[it][q], __builtin_amdgcn_alignbyte(D1[it][q], D0[it][q], P[it][q]), S[it][q]);
            const unsigned q0 = pk_rsub(T2, perm(E, L[it][q], 0x05010400u)), q1 = pk_rsub(T2, perm(E, L[it][q], 0x07030602u));
            const unsigned w0 = OUT ? pk_half(q0) : __builtin_amdgcn_alignbit(q0, q0, 1), w1 = OUT ? pk_half(q1) : __builtin_amdgcn_alignbit(q1, q1, 1);
            if (q < 2) { x[2 * q] = w0; x[2 * q + 1] = w1; } else { y[2 * q - 4] = w0; y[2 * q - 3] = w1; }
        }
        put(it, x, y);
    };
    fetch(0); fetch(1); fetch(2); fetch(3);
    if (OUT) {
        if (five) fetch(4);
        __builtin_amdgcn_sched_barrier(0);
        finish(0); finish(1); finish(2); finish(3);
        if (five) finish(4);
        if (eight) { fetch(5); fetch(6); fetch(7); __builtin_amdgcn_sched_barrier(0); finish(5); finish(6); finish(7); }
    } else {
#pragma unroll
        for (int it = 0; it < 8; ++it) { __builtin_amdgcn_sched_barrier(0); finish(it); if (it < 4) fetch(it + 4); }
    }
}

__device__ __forceinline__ void decode_in(unsigned info, unsigned cw, unsigned esc, char* dst, int lane DBG_ARGS) {
    decode_values<false>([&](int it, const u32x4& x, const u32x4& y) { st_wt(dst + it * 2048, x); st_wt(dst + it * 2048 + 1024, y); }, info, cw, esc, lane DBG_PASS);
}
__device__ __forceinline__ void make_table(int n) {
    unsigned s[4], c = 0;
#pragma unroll
    for (int e = 0; e < 4; ++e) { const bool nz = (n >> e) & 1; s[e] = nz ? 0x0100u + c * 0x0202u : 0x0c0cu; c += nz; }
    *ldsp<u32x2>(LDS_TAB + n * 8) = (u32x2){s[0] | s[1] << 16, s[2] | s[3] << 16};
}
__device__ __forceinline__ void decode_out(unsigned info, unsigned cw, unsigned esc, char* outp, int lane DBG_ARGS) {
    const u32x4 mk = *ldsp<u32x4>(cw + lane * 16);
    decode_values<true>([&](int it, const u32x4& x, const u32x4& y) { *ldsp<u32x4>(cw + (it * 64 + lane) * 32) = x; *ldsp<u32x4>(cw + (it * 64 + lane) * 32 + 16) = y; }, info, cw, esc, lane DBG_PASS);
    SSTAMP(22);
    unsigned nl[16], nh[16], cl[16], st[8], base[16], run = cw;
#pragma unroll
    for (int k = 0; k < 8; ++k) {
        unsigned c[2];
#pragma unroll
        for (int j = 0; j < 2; ++j) {
            const int i = 2 * k + j;
            nl[i] = (((mk[i >> 2] & 0x0f0f0f0fu) << 3) >> (8 * (i & 3))) & 0xffu; nh[i] = (((mk[i >> 2] & 0xf0f0f0f0u) >> 1) >> (8 * (i & 3))) & 0xffu;
            cl[i] = __builtin_popcount(nl[i]); c[j] = cl[i] + __builtin_popcount(nh[i]);
        }
        const unsigned own = (c[0] | c[1] << 16) * 2u, inc = scan_add(own), t = last_lane(inc);
        st[k] = inc - own;
        base[2 * k] = run; run += t & 0xffffu; base[2 * k + 1] = run; run += t >> 16;
    }
    fence();
    SSTAMP(23);
    unsigned at[32], d0[32], d1[32], d2[32]; u32x2 sel[32];
    auto fetch = [&](int i) {
        at[2 * i] = base[i] + ((i & 1) ? st[i >> 1] >> 16 : st[i >> 1] & 0xffffu); at[2 * i + 1] = at[2 * i] + cl[i] * 2u;
#pragma unroll
        for (int k = 2 * i; k < 2 * i + 2; ++k) {
            const unsigned ad = at[k] & ~3u;
            d0[k] = *ldsp<unsigned>(ad); d1[k] = *ldsp<unsigned>(ad + 4u); d2[k] = *ldsp<unsigned>(ad + 8u);
            sel[k] = *ldsp<u32x2>(LDS_TAB + ((k & 1) ? nh[i] : nl[i]));
        }
    };
#pragma unroll
    for (int i = 0; i < AHEAD; ++i) fetch(i);
#pragma unroll
    for (int i = 0; i < 16; ++i) {
        if (i + AHEAD < 16) fetch(i + AHEAD);
        __builtin_amdgcn_sched_barrier(0);
        u32x4 o;
#pragma unroll
        for (int j = 0; j < 2; ++j) {
            const int k = 2 * i + j;
            const unsigned t0 = __builtin_amdgcn_alignbyte(d1[k], d0[k], at[k]), t1 = __builtin_amdgcn_alignbyte(d2[k], d1[k], at[k]);
            o[2 * j] = perm(t1, t0, sel[k].x); o[2 * j + 1] = perm(t1, t0, sel[k].y);
        }
        st_wt(outp + OUT_OFF(i), o);
    }
}

__device__ __forceinline__ void in_job(const Args& a, char* myws, int dest, int g, int kt, int j, int lane, unsigned seen, unsigned tail DBG_ARGS) {
    const int panel = g * 8 + ((j & 15) >> 1), half = j & 1, rank = a.rank;
    char* pq = a.ws[dest == rank ? (rank + 1 + lane) & 7 : dest];
    u32x4 v[16];
    if (j < 16) {
        const int sb = lane * 16, swz = sb ^ (((sb >> 9) & 1) << 5);
        const char* src = a.x + (size_t)(panel * 256 + half * 128 + (swz >> 6)) * (KS * 2) + kt * 128 + (swz & 63);
#pragma unroll
        for (int i = 0; i < 16; ++i) v[i] = *(const u32x4*)(src + (i >> 1) * 16 * (KS * 2) + (i & 1) * 64);
        __builtin_amdgcn_sched_barrier(0); wait_vm0();
    } else {
        const char* src = a.w + (size_t)(kt * 64 + (lane & 3) * 8) * (GN * 2) + (size_t)(dest * NB + panel * 256 + half * 128 + (lane >> 2) * 8) * 2;
        u32x4 r[16];
#pragma unroll
        for (int i = 0; i < 16; ++i) r[i] = *(const u32x4*)(src + (size_t)((i >> 3) * 32 + (i & 7)) * (GN * 2));
        __builtin_amdgcn_sched_barrier(0); wait_vm0();
#pragma unroll
        for (int kh = 0; kh < 2; ++kh)
#pragma unroll
            for (int i = 0; i < 8; ++i)
#pragma unroll
                for (int d = 0; d < 4; ++d) v[kh * 8 + i][d] = __builtin_amdgcn_perm(r[kh * 8 + 2 * d + 1][i >> 1], r[kh * 8 + 2 * d][i >> 1], (i & 1) ? 0x07060302u : 0x05040100u);
    }
    asm volatile("" : "+v"(pq));
    asm volatile("" : "+v"(seen));
    STAMPD(29);
    const unsigned call = seen + 1u;
    const int pos = g * 256 + kt * 32 + j;
    if (j < 16 || dest != rank) {
        char* dst = myws + (j < 16 ? OFF_XW : OFF_WW + (size_t)dest * (256 * 16384)) + (size_t)(g * 128 + kt * 16 + (j & 15)) * 16384 + lane * 16;
        GAS u64* const note = (GAS u64*)(pq + OFF_Q) + rank * NQ + pos;
        const unsigned info = encode_in(v, tail, dst, lane, [&]() {
            if (EARLY(pos)) { wait_vm0(); if (lane < (j < 16 ? 7 : 1)) __hip_atomic_store(note + NPOS, (u64)call << 32, __ATOMIC_RELAXED, __HIP_MEMORY_SCOPE_SYSTEM); STAMPD(30); } });
        if (info & INFO_RAW) {
            wait_vm0();
#pragma unroll
            for (int i = 0; i < 16; ++i) st_wt(dst + i * 1024, v[i]);
        }
        wait_vm0();
        if (lane < (j < 16 ? 7 : 1)) __hip_atomic_store(note, ((u64)call << 32) | info, __ATOMIC_RELAXED, __HIP_MEMORY_SCOPE_SYSTEM);
        STAMPD(31);
    }
    if (dest == rank) {
        char* dst = myws + img_off(g, j, kt) + lane * 16;
#pragma unroll
        for (int i = 0; i < 16; ++i) st_wt(dst + i * 1024, v[i]);
        wait_vm0();
        if (lane == 0) __hip_atomic_store((GAS unsigned*)(myws + flag_off(g, j, kt)), call, __ATOMIC_RELAXED, __HIP_MEMORY_SCOPE_SYSTEM);
    }
}

__device__ __forceinline__ bool expired(unsigned* tmo, unsigned call, u64 t0) {
    if (__builtin_amdgcn_readfirstlane(__hip_atomic_load(tmo, __ATOMIC_RELAXED, __HIP_MEMORY_SCOPE_AGENT)) == call || now() - t0 > DEADLINE) {
        __hip_atomic_store(tmo, call, __ATOMIC_RELAXED, __HIP_MEMORY_SCOPE_AGENT);
        return true;
    }
    return false;
}

__device__ __forceinline__ void tile_of(int ti, int xcd, int& pm, int& pn) { pm = (ti >> 4) * 8 + (xcd & 1) * 4 + (ti & 3); pn = ((ti >> 3) & 1) * 8 + (xcd >> 1) * 2 + ((ti >> 2) & 1); }
__device__ __forceinline__ size_t out_off(int src, int pm, int pn, int w, int lane) { return (size_t)(pm * 256 + (w >> 2) * 64 + (lane >> 3)) * (GN * 2) + (size_t)(src * NB + pn * 256 + (w & 3) * 64) * 2 + (lane & 7) * 16; }

__global__ __launch_bounds__(512, 2) void gemm_ar_kernel(const Args a) {
    extern __shared__ __attribute__((aligned(16))) unsigned char shm[];
    LAS unsigned char* lds = (LAS unsigned char*)shm;
    const int tid = threadIdx.x, wid = __builtin_amdgcn_readfirstlane(tid >> 6), lane = tid & 63;
    const int b = blockIdx.x, xcd = b & 7, idx = b >> 3, rank = a.rank;
    char* const myws = a.myws;
    char* const myout = a.myout;
    const u64 t0 = now();
    STAMP(0);
    unsigned* const ctr = (unsigned*)(myws + OFF_CTR) + b * 32 + wid;
    unsigned* const tmo = (unsigned*)(myws + OFF_TMO);
    const unsigned seen = __hip_atomic_load(ctr, __ATOMIC_RELAXED, __HIP_MEMORY_SCOPE_AGENT);
    const unsigned cw = LDS_CW + wid * C_WAVE, tl = LDS_TL + wid * C_TAIL;

    const int pk = (idx - NG + 7 * 8 + xcd) % 7, peer = (rank + 1 + pk) & 7, ord = xcd * 2 + (idx >= NG + 7);
    if (idx >= NG) { if (tid < 2) *ldsp<unsigned>(LDS_CTL + tid * 4) = tid ? 0u : 8u; if (tid < 16) make_table(tid); wait_lds(); bar(); }
    {
        int q, qe, delay;
        if (idx < 8) { q = wid; qe = 4; delay = (wid || !(idx & 2) ? DLY0 : 0) + wid * DLY1; }
        else if (idx < 10 || idx >= NG) { q = 0; qe = 0; delay = 0; }
        else { q = wid * 64 + (idx - 10) * 8 + xcd; qe = 2048; delay = DLY0 + (ORD ? (q < 224 ? wid * DLY3 : (wid - 2) * DLY2) : wid < 2 ? 0 : (wid - 1) * DLY2); }
        for (int i = 0; i < delay; ++i) __builtin_amdgcn_s_sleep(8);
        for (; q < qe; q += 512) {
            int r;
            if (idx < 8) r = wid * 64 + (idx ^ 2) * 8 + xcd;
            else {
                const int kt = q < 896 ? q / 112 : 8 + (q - 896) / 144, rem = q < 896 ? q % 112 : (q - 896) % 144 - 32;
                r = kt * 32 + (rem < 0 ? rem + 32 : (rem % 7 + 1) * 512 + 16 + rem / 7);
            }
            in_job(a, myws, (rank + (r >> 9)) & 7, (r >> 8) & 1, (r >> 5) & 7, r & 31, lane, seen, tl DBG_PASS);
        }
    }
    const unsigned call = __builtin_amdgcn_readfirstlane(seen) + 1u;
    if (lane == 0) __hip_atomic_store(ctr, call, __ATOMIC_RELAXED, __HIP_MEMORY_SCOPE_AGENT);
    STAMP(1);

    if (idx >= NG) {
        const char* const pws = a.ws[peer];
        const u64* const queue = (const u64*)(myws + OFF_Q) + peer * NQ;
        unsigned* const tq = (unsigned*)(myws + OFF_TICK) + pk * 64;
        const unsigned tbase = (call - 1u) * NPOS - 128u;
        const unsigned ctl = LDS_CTL;
        unsigned pos = (((ORD ? 0x76543120u : 0x75634120u) >> (4 * wid)) & 7u) * 16 + ord;
        bool bad = false;
        while (pos < (unsigned)NPOS) {
            const int g = pos >> 8, kt = (pos >> 5) & 7, j = pos & 31;
            const char* const src = pws + (pos >= NIN ? OFF_OW + (size_t)(pos - NIN) * 16384
                                           : (j < 16 ? OFF_XW : OFF_WW + (size_t)rank * (256 * 16384)) + (size_t)(g * 128 + kt * 16 + (j & 15)) * 16384) + lane * 16;
            auto token = [&]() {
                unsigned my = wid;
                if (lane == 0 && pos >= 128u) asm volatile("ds_add_rtn_u32 %0, %1, %2\n\ts_waitcnt lgkmcnt(0)" : "+v"(my) : "v"(ctl), "v"(1u) : "memory");
                my = __builtin_amdgcn_readfirstlane(my);
                for (;;) {
                    unsigned rel;
                    asm volatile("ds_read_b32 %0, %1 offset:4\n\ts_waitcnt lgkmcnt(0)" : "=v"(rel) : "v"(ctl) : "memory");
                    if ((int)(__builtin_amdgcn_readfirstlane(rel) + TOK - my) > 0) break;
                    if (now() - t0 > DEADLINE) break;
                    __builtin_amdgcn_s_sleep(1);
                }
            };
            unsigned info;
            int o = 0;
            for (unsigned at = EARLY(pos) ? pos + NPOS : pos, spin = 1;; ++spin) {
                const u64 e = __hip_atomic_load(queue + at, __ATOMIC_RELAXED, __HIP_MEMORY_SCOPE_SYSTEM);
                info = __builtin_amdgcn_readfirstlane((unsigned)e);
                if (__builtin_amdgcn_readfirstlane((unsigned)(e >> 32)) >= call) {
                    if (at == pos) break;
                    STAMP1(2);
                    token();
                    for (; o < 8192; o += 1024) {
                        if (PEEK && o == 7168) { if (lane < 2) __builtin_amdgcn_global_load_lds((const unsigned*)(queue + pos) + lane, ldsp<unsigned>(tl), 4, 0, 17); wait_vm<WIN>(); }
                        else wait_vm<WIN - 1>();
                        __builtin_amdgcn_global_load_lds((const unsigned*)(src + o), ldsp<unsigned>(cw + o), 16, 0, 17);
                    }
                    if (PEEK) {
                        u32x2 pe;
                        asm volatile("s_waitcnt vmcnt(1)\n\tds_read_b64 %0, %1\n\ts_waitcnt lgkmcnt(0)" : "=v"(pe) : "v"(tl) : "memory");
                        info = __builtin_amdgcn_readfirstlane(pe.x);
                        if (__builtin_amdgcn_readfirstlane(pe.y) >= call) break;
                    }
                    at = pos;
                    continue;
                }
                if ((spin & 15u) == 0 && expired(tmo, call, t0)) { bad = true; break; }
                if (at == pos) __builtin_amdgcn_s_sleep(POLL_SLEEP); else __builtin_amdgcn_s_sleep(POLL_FAST);
            }
            if (bad) break;
            LAP(24); STAMP1(2);
            if ((info & 1023u) > 512u || ((info >> 10) & 511u) - 1u >= (unsigned)NW_CAP) info = INFO_RAW;
            if (info & INFO_RAW) o = 0;
            const int wire = (info & INFO_RAW) ? 16384 : pos >= NIN ? wire_bytes<true>(info) : wire_bytes<false>(info);
            if (!EARLY(pos)) token();
            LAP(25); STAMP1(7); STAMP(18);
            for (; o < wire; o += 1024) {
                if (o + lane * 16 < wire) __builtin_amdgcn_global_load_lds((const unsigned*)(src + o), ldsp<unsigned>(cw + o), 16, 0, 17);
                wait_vm<WIN - 1>();
            }
            wait_vm<REL>();
            if (lane == 0) asm volatile("ds_add_u32 %0, %1 offset:4" :: "v"(ctl), "v"(1u) : "memory");
            wait_vm0();
            LAP(26); STAMP1(3); STAMP(19);
            int t = 0;
            char* dst;
            if (pos >= NIN) { int pm, pn; tile_of((pos - NIN) >> 6, ((pos - NIN) >> 3) & 7, pm, pn); dst = myout + out_off(peer, pm, pn, pos & 7, lane); }
            else { t = slot_of(peer, rank, kt); dst = myws + img_off(g, j, t) + lane * 16; }
            if (info & INFO_RAW) {
                u32x4 v[16];
#pragma unroll
                for (int i = 0; i < 16; ++i) v[i] = *ldsp<u32x4>(cw + i * 1024 + lane * 16);
#pragma unroll
                for (int i = 0; i < 16; ++i) st_wt(dst + (pos >= NIN ? OUT_OFF(i) : (size_t)i * 1024), v[i]);
            } else if (pos >= NIN) decode_out(info, cw, tl, dst, lane DBG_PASS);
            else decode_in(info, cw, tl, dst, lane DBG_PASS);
            LAP(27); STAMP1(4); STAMP(20);
            unsigned nx = 0;
            if (lane == 0) nx = __hip_atomic_fetch_add(tq, 1u, __ATOMIC_RELAXED, __HIP_MEMORY_SCOPE_AGENT);
            wait_vm0();
            if (pos < NIN && lane == 0) __hip_atomic_store((unsigned*)(myws + flag_off(g, j, t)), call, __ATOMIC_RELAXED, __HIP_MEMORY_SCOPE_SYSTEM);
            LAP(28); STAMP(8 + (pos >> 8)); STAMP1(6);
            pos = __builtin_amdgcn_readfirstlane(nx) - tbase;
        }
        if (bad && lane == 0) *(GAS u16*)(myout + (size_t)rank * (NB * 2)) = 0x7fc0;
        STAMP(5);
        STAMP_FLUSH();
        return;
    }

    const int wr = wid >> 2, wc = wid & 3, fr = lane & 15, fq = lane >> 4;
    bool bad = false;
    const bool hm = idx < 4;
    unsigned voffA[2], voffB[2];
#pragma unroll
    for (int i = 0; i < 2; ++i) {
        int R, C; stage_rc(tid * 16 + i * 8192, R, C);
        voffA[i] = tid * 16 + (hm ? (idx & 1) * 8192 + i * 16384 : i * 8192);
        const int w_ = R >> 5, n_ = (R >> 4) & 1, q_ = (R & 15) >> 2, j_ = R & 3;
        voffB[i] = (w_ >> 1) * 16384 + (C >> 5) * 8192 + (4 * n_ + j_) * 1024 + ((w_ & 1) * 8 + q_) * 64 + ((C >> 3) & 3) * 16;
    }
    constexpr size_t kstep = 32768, hstepA = 16384, hstepB = 256;
    const unsigned ldsw = (unsigned)wid * 1024u;
    const int aoff = lds_byte(wr * 64 + fr, fq * 8), boff = lds_byte(wc * 32 + fr, fq * 8);
#define G_SA(b, h) (((b) * 2 + (h)) * HTB)
#define G_SB(b, h) ((4 + (b) * 2 + (h)) * HTB)
#define G_STAGE(bufoff, gbase, voff) do { _Pragma("unroll") for (int _i = 0; _i < 2; ++_i) \
        __builtin_amdgcn_global_load_lds((const unsigned*)((const char*)(gbase) + (voff)[_i]), (LAS unsigned*)(lds + (bufoff) + ldsw + _i * 8192), 16, 0, 17); } while (0)
#define G_LDA(dst, b, h) do { _Pragma("unroll") for (int m = 0; m < 4; ++m) _Pragma("unroll") for (int k = 0; k < 2; ++k) dst[m][k] = *(const LAS bf16x8*)(lds + G_SA(b, h) + aoff + m * 2048 + k * 1024); } while (0)
#define G_LDB(dst, b, h) do { _Pragma("unroll") for (int n = 0; n < 2; ++n) _Pragma("unroll") for (int k = 0; k < 2; ++k) dst[n][k] = *(const LAS bf16x8*)(lds + G_SB(b, h) + boff + n * 2048 + k * 1024); } while (0)
#define G_MMA(ai, bj, At, Bt) do { __builtin_amdgcn_s_setprio(1); _Pragma("unroll") for (int m = 0; m < 4; ++m) _Pragma("unroll") for (int n = 0; n < 2; ++n) _Pragma("unroll") for (int k = 0; k < 2; ++k) \
        acc[ai][bj][m][n] = __builtin_amdgcn_mfma_f32_16x16x32_bf16(Bt[n][k], At[m][k], acc[ai][bj][m][n], 0, 0, 0); __builtin_amdgcn_s_setprio(0); } while (0)
#define G_WAIT_V(n) asm volatile("s_waitcnt vmcnt(" #n ")" ::: "memory")
#define G_WAIT_L(n) asm volatile("s_waitcnt lgkmcnt(" #n ")" ::: "memory")
#define G_BAR __builtin_amdgcn_s_barrier()
#define G_SCHED __builtin_amdgcn_sched_barrier(0)
    for (int round = 0; round < 2; ++round) {
    const int ti = hm ? round * 8 + (idx >> 1) : round ? idx + 14 : idx < 10 ? idx - 2 : idx;
    int pm, pn; tile_of(ti, xcd, pm, pn);
    const char* const fX = myws + OFF_INF + (size_t)pm * 512 + lane * 8;
    const char* const fW = myws + OFF_INF + (size_t)(16 + pn) * 512 + lane * 8;
    auto poll = [&](int need) -> int {
        for (;;) {
            u32x2 fa, fb;
            asm volatile("global_load_dwordx2 %0, %2, off sc0 sc1\n\tglobal_load_dwordx2 %1, %3, off sc0 sc1\n\ts_waitcnt vmcnt(0)"
                         : "=&v"(fa), "=&v"(fb) : "v"(fX), "v"(fW) : "memory");
            const u64 miss = __builtin_amdgcn_ballot_w64(fa.x < call || fa.y < call || fb.x < call || fb.y < call);
            const int upto = miss ? __builtin_ctzll(miss) : 64;
            if (upto >= need) return upto;
            if (expired(tmo, call, t0)) { bad = true; return 64; }
            __builtin_amdgcn_s_sleep(4);
        }
    };
    f32x4 acc[2][2][4][2];
#pragma unroll
    for (int x = 0; x < 2; ++x)
#pragma unroll
        for (int y = 0; y < 2; ++y)
#pragma unroll
            for (int m = 0; m < 4; ++m)
#pragma unroll
                for (int n = 0; n < 2; ++n) acc[x][y][m][n] = (f32x4){0.f, 0.f, 0.f, 0.f};
    bf16x8 At[4][2], B0[2][2], B1[2][2];
    const char* const cA = myws + OFF_XI + (size_t)pm * 64 * kstep;
    const char* const cB = myws + OFF_WI + (size_t)pn * 64 * kstep;
    int ready = poll(2);
    STAMP1(2);
    wait_vm0();
    if (hm) {
        G_STAGE(0, cA, voffA); G_STAGE(HTB, cB, voffB); G_STAGE(2 * HTB, cB + hstepB, voffB);
        G_STAGE(3 * HTB, cA + kstep, voffA); G_STAGE(4 * HTB, cB + kstep, voffB); G_STAGE(5 * HTB, cB + kstep + hstepB, voffB);
        if (wr == 1) G_BAR;
        G_WAIT_V(6); G_BAR; G_BAR;
        unsigned cur = 0, nxt = 6 * HTB;
        for (int t = 0; t < 64; ++t) {
            if (t < 62 && ready < t + 3) ready = poll(t + 3);
            const size_t k2 = t < 62 ? (size_t)(t + 2) * kstep : 0;
            LAS unsigned char* const la = lds + cur + aoff; LAS unsigned char* const lb = lds + cur + boff;
#pragma unroll
            for (int n = 0; n < 2; ++n)
#pragma unroll
                for (int k = 0; k < 2; ++k) B0[n][k] = *(const LAS bf16x8*)(lb + HTB + n * 2048 + k * 1024);
#pragma unroll
            for (int m = 0; m < 4; ++m)
#pragma unroll
                for (int k = 0; k < 2; ++k) At[m][k] = *(const LAS bf16x8*)(la + m * 2048 + k * 1024);
            G_STAGE(nxt, cA + k2, voffA); G_STAGE(nxt + HTB, cB + k2, voffB);
            G_WAIT_V(10); G_BAR; G_WAIT_L(0); G_MMA(0, 0, At, B0); G_BAR; G_SCHED;
#pragma unroll
            for (int n = 0; n < 2; ++n)
#pragma unroll
                for (int k = 0; k < 2; ++k) B1[n][k] = *(const LAS bf16x8*)(lb + 2 * HTB + n * 2048 + k * 1024);
            G_STAGE(nxt + 2 * HTB, cB + k2 + hstepB, voffB);
            G_WAIT_V(8); G_BAR; G_WAIT_L(0); G_MMA(0, 1, At, B1); G_BAR; G_SCHED;
            nxt = cur; cur = cur == 6 * HTB ? 0 : cur + 3 * HTB;
        }
    } else {
    G_STAGE(G_SB(0, 0), cB, voffB); G_STAGE(G_SA(0, 0), cA, voffA); G_STAGE(G_SB(0, 1), cB + hstepB, voffB); G_STAGE(G_SA(0, 1), cA + hstepA, voffA);
    if (wr == 1) G_BAR;
    G_WAIT_V(4); G_BAR;
    G_STAGE(G_SB(1, 0), cB + kstep, voffB); G_STAGE(G_SA(1, 0), cA + kstep, voffA); G_STAGE(G_SB(1, 1), cB + hstepB + kstep, voffB);
    G_WAIT_V(6); G_BAR;
    for (int t = 0; t < 64; t += 2) {
        const bool last = (t == 62);
        if (!last && ready < t + 4) ready = poll(t + 4);
        const char* a1 = cA + (size_t)(t + 1) * kstep;
        const char* a2 = last ? cA : cA + (size_t)(t + 2) * kstep; const char* b2 = last ? cB : cB + (size_t)(t + 2) * kstep;
        const char* a3 = a2 + kstep; const char* b3 = b2 + kstep;
        G_LDB(B0, 0, 0); G_SCHED; G_LDA(At, 0, 0); G_STAGE(G_SA(1, 1), a1 + hstepA, voffA);
        G_WAIT_L(8); G_BAR; G_WAIT_L(0); G_MMA(0, 0, At, B0); G_BAR; G_SCHED;
        G_LDB(B1, 0, 1); G_STAGE(G_SB(0, 0), b2, voffB);
        G_BAR; G_WAIT_L(0); G_MMA(0, 1, At, B1); G_BAR;
        G_LDA(At, 0, 1); G_STAGE(G_SA(0, 0), a2, voffA);
        G_BAR; G_WAIT_L(0); G_MMA(1, 0, At, B0); G_BAR; G_SCHED;
        G_STAGE(G_SB(0, 1), b2 + hstepB, voffB);
        G_WAIT_V(6); G_BAR; G_MMA(1, 1, At, B1); G_BAR;
        G_LDB(B0, 1, 0); G_SCHED; G_LDA(At, 1, 0); G_STAGE(G_SA(0, 1), a2 + hstepA, voffA);
        G_WAIT_L(8); G_BAR; G_WAIT_L(0); G_MMA(0, 0, At, B0); G_BAR; G_SCHED;
        G_LDB(B1, 1, 1); G_STAGE(G_SB(1, 0), b3, voffB);
        G_BAR; G_WAIT_L(0); G_MMA(0, 1, At, B1); G_BAR;
        G_LDA(At, 1, 1); G_STAGE(G_SA(1, 0), a3, voffA);
        G_BAR; G_WAIT_L(0); G_MMA(1, 0, At, B0); G_BAR; G_SCHED;
        G_STAGE(G_SB(1, 1), b3 + hstepB, voffB);
        G_WAIT_V(6); G_BAR; G_MMA(1, 1, At, B1); G_BAR;
    }
    }
    G_WAIT_V(0);
    if (wr == 0) G_BAR;
    G_BAR;
    STAMP(round ? 6 : 3);

    int ln = lane;
    asm volatile("" : "+v"(ln) :: "memory");
    char* pq = a.ws[(rank + 1 + (ln >> 3)) & 7];
    if (bad) acc[0][0][0][0][0] = __builtin_nanf("");
    const unsigned cx = hm ? LDS_CW + ((idx & 1) * 4 + wc) * C_WAVE + wr * 8192 : cw;
#pragma unroll
    for (int ai = 0; ai < 2; ++ai) if (ai == 0 || !hm)
#pragma unroll
        for (int m = 0; m < 4; ++m)
#pragma unroll
            for (int bj = 0; bj < 2; ++bj) {
                const f32x4 v0 = acc[ai][bj][m][0], v1 = acc[ai][bj][m][1];
                u32x4 w; w[0] = relu_pk(cvt_pk_bf16(v0[0], v0[1])); w[1] = relu_pk(cvt_pk_bf16(v0[2], v0[3]));
                w[2] = relu_pk(cvt_pk_bf16(v1[0], v1[1])); w[3] = relu_pk(cvt_pk_bf16(v1[2], v1[3]));
                *ldsp<u32x4>(cx + (ai * 64 + m * 16 + (ln & 15)) * 128 + (((bj * 4 + (ln >> 4)) ^ (ln & 7)) << 4)) = w;
            }
    asm volatile("" ::: "memory");
    if (hm) { wait_lds(); bar(); }
    if (!hm || wr == (idx & 1)) {
    u32x4 W[16];
#pragma unroll
    for (int i = 0; i < 16; ++i) W[i] = *ldsp<u32x4>(cw + i * 1024 + (ln >> 3) * 128 + (((ln & 7) ^ (ln >> 3)) << 4));
    wait_lds(); wait_vm0();
    asm volatile("" : "+v"(pq));
    char* const outp = myout + out_off(rank, pm, pn, wid, ln);
#pragma unroll
    for (int i = 0; i < 16; ++i) st_wt(outp + OUT_OFF(i), W[i]);
    char* const slot = myws + OFF_OW + (size_t)((ti * 8 + xcd) * 8 + wid) * 16384 + ln * 16;
    const unsigned info = encode_out(W, cw, tl, slot, ln);
    if (info & INFO_RAW) {
#pragma unroll
        for (int i = 0; i < 16; ++i) st_wt(slot + i * 1024, W[i]);
    }
    if (ln == 0) *(LAS unsigned*)(lds + LDS_CTL + wid * 4) = info;
    }
    wait_lds();
    wait_vm0();
    bar();
    if (wid == 0 && ln < 56 && (!hm || ((ln >> 2) & 1) == (idx & 1))) {
        const unsigned inf = *(const LAS unsigned*)(lds + LDS_CTL + (ln & 7) * 4);
        __hip_atomic_store((GAS u64*)(pq + OFF_Q) + rank * NQ + NIN + (ti * 8 + xcd) * 8 + (ln & 7), ((u64)call << 32) | inf, __ATOMIC_RELAXED, __HIP_MEMORY_SCOPE_SYSTEM);
    }
    STAMP(round ? 7 : 4);
    }
    STAMP(5);
    STAMP_FLUSH();
}

extern "C" void kernel_launch(int rank, int nranks, ncclComm_t comm,
                              hipStream_t stream,
                              void* const* d_in, const int* in_sizes, int n_in,
                              void* d_out, int out_size,
                              const int* in_dtypes, int out_dtype,
                              void* const* d_out_all,
                              void* const* d_ws_all, size_t ws_bytes) {
    if (nranks != NR || (unsigned)rank >= (unsigned)NR || ws_bytes < WS_NEED || n_in < 2) { fprintf(stderr, "unsupported configuration\n"); return; }
    static bool attr[NR] = {};
    if (!attr[rank]) { HIP_CHECK(hipFuncSetAttribute((const void*)gemm_ar_kernel, hipFuncAttributeMaxDynamicSharedMemorySize, LDS_BYTES)); attr[rank] = true; }
    Args a;
    a.x = (const char*)d_in[0]; a.w = (const char*)d_in[1]; a.rank = rank;
    for (int p = 0; p < NR; ++p) a.ws[p] = (char*)d_ws_all[p];
    a.myws = a.ws[rank]; a.myout = (char*)d_out;
    hipLaunchKernelGGL(gemm_ar_kernel, dim3(256), dim3(512), LDS_BYTES, stream, a);
    HIP_CHECK(hipGetLastError());
}
```

```cpp
#include <hip/hip_runtime.h>
#include <rccl/rccl.h>
#include <cstdio>

#define HIP_CHECK(expr) do { hipError_t _e = (expr); if (_e != hipSuccess) { \
    fprintf(stderr, "HIP error %s at %s:%d\n", hipGetErrorString(_e), __FILE__, __LINE__); \
    return; } } while (0)

#define LAS __attribute__((address_space(3)))
#define GAS __attribute__((address_space(1)))
typedef unsigned short u16;
typedef unsigned long long u64;
typedef short bf16x8 __attribute__((ext_vector_type(8)));
typedef float f32x4 __attribute__((ext_vector_type(4)));
typedef unsigned u32x4 __attribute__((ext_vector_type(4)));
typedef unsigned u32x2 __attribute__((ext_vector_type(2)));

constexpr int NR = 8;
constexpr int GM = 4096, GN = 32768, KS = 512, NB = 4096;
constexpr int HTB = 16384;
constexpr int NG = 18;
constexpr int NIN = 512, NPOS = 2560;
constexpr int NQ = NPOS + 64;
constexpr unsigned LDS_TAB = 64, LDS_CW = 256, C_WAVE = 16384 + 128, LDS_TL = LDS_CW + 8 * C_WAVE, C_TAIL = 3584, ESC_CAP = 3560;
constexpr unsigned LDS_CTL = LDS_TL + 8 * C_TAIL + 64;
constexpr int LDS_BYTES = LDS_CTL + 64;
static_assert(LDS_TL >= 8 * HTB && LDS_CTL >= 9 * HTB && LDS_BYTES <= 163840, "LDS layout");
constexpr size_t OFF_CTR = 0;
constexpr size_t OFF_TMO = 32768;
constexpr size_t OFF_TICK = 32768 + 256;
constexpr size_t OFF_STAMP = 65536;
constexpr size_t OFF_INF = 512u << 10;
constexpr size_t OFF_Q = 1u << 20;
constexpr size_t OFF_XI = 4u << 20;
constexpr size_t OFF_WI = 36u << 20;
constexpr size_t OFF_XW = 68u << 20;
constexpr size_t OFF_WW = 72u << 20;
constexpr size_t OFF_OW = 104u << 20;
constexpr size_t WS_NEED = 136u << 20;
constexpr u64 DEADLINE = 50000000ull;

#define TOK 2
#define WIN 2
#define REL (WIN - 1)
#define DLY1 3
#define DLY2 5
#define POLL_SLEEP 6
#define DLY0 4
#define EARLY_AT 3
#define ORD 0
#define PEEK 1
#define EARLY(pos) (((pos) & (ORD ? ~63 : ~47)) == 0)
#define AHEAD 2
#define PREN 80
#define WIDE 24
#define DLY3 1
#define POLL_FAST 1
#define WIRE_ALIGN 64
struct Args { const char* x; const char* w; char* myws; char* myout; int rank; char* ws[NR]; };

__device__ __forceinline__ int lds_byte(int r, int c) { const int st = (r >> 4) * 2 + (c >> 5), rr = r & 15, cc = c & 31, ob = rr * 64 + cc * 2; return st * 1024 + (ob ^ (((ob >> 9) & 1) << 5)); }
__device__ __forceinline__ void stage_rc(int b, int& R, int& C) { const int st = b / 1024, sb = b % 1024, swz = sb ^ (((sb >> 9) & 1) << 5); R = (st >> 1) * 16 + swz / 64; C = (st & 1) * 32 + (swz % 64) / 2; }
__device__ __forceinline__ unsigned cvt_pk_bf16(float lo, float hi) { unsigned r; asm volatile("v_cvt_pk_bf16_f32 %0, %1, %2" : "=v"(r) : "v"(lo), "v"(hi)); return r; }
__device__ __forceinline__ unsigned relu_pk(unsigned v) { unsigned r; asm("v_pk_max_i16 %0, %1, 0" : "=v"(r) : "v"(v)); return r; }
__device__ __forceinline__ void st_wt(char* p, u32x4 v) { asm volatile("global_store_dwordx4 %0, %1, off sc0 sc1\n\ts_nop 1" :: "v"(p), "v"(v) : "memory"); }
__device__ __forceinline__ void wait_vm0() { __builtin_amdgcn_s_waitcnt(0x0F70); asm volatile("s_waitcnt vmcnt(0)" ::: "memory"); }
template <int N> __device__ __forceinline__ void wait_vm() { asm volatile("s_waitcnt vmcnt(%0)" :: "n"(N) : "memory"); }
__device__ __forceinline__ void bar() { asm volatile("" ::: "memory"); __builtin_amdgcn_s_barrier(); asm volatile("" ::: "memory"); }
__device__ __forceinline__ u64 now() { return __builtin_amdgcn_s_memrealtime(); }

#define STAMP(i) do { } while (0)
#define STAMP1(i) do { } while (0)
#define STAMPD(i) do { } while (0)
#define SSTAMP(i) do { } while (0)
#define DBG_ARGS
#define DBG_PASS
#define LAP(i) do { } while (0)
#define STAMP_FLUSH() do { } while (0)

__device__ __forceinline__ int slot_of(int s, int p, int kt) { return s == p ? kt : 8 + 7 * kt + ((s - p - 1) & 7); }
__device__ __forceinline__ size_t img_off(int g, int j, int t) { return (j < 16 ? OFF_XI : OFF_WI) + (size_t)((g * 8 + ((j & 15) >> 1)) * 64 + t) * 32768 + (j & 1) * 16384; }
__device__ __forceinline__ size_t flag_off(int g, int j, int t) { return OFF_INF + (size_t)((((j >> 4) * 16 + g * 8 + ((j & 15) >> 1)) * 64 + t) * 2 + (j & 1)) * 4; }

#define OUT_OFF(i) ((size_t)(((i) >> 3) * 128 + ((i) & 7) * 8) * (GN * 2))
constexpr unsigned INFO_RAW = 1u << 31;
constexpr int NW_CAP = 384;
template <bool OUT> __device__ __forceinline__ int cls_off(int nch) { return OUT ? 1024 + nch * 16 : 8192; }
template <bool OUT> __device__ __forceinline__ int cls_bytes(int nch) { return OUT ? (nch * 2 + 3) & ~3 : 2048; }
template <bool OUT> __device__ __forceinline__ int wire_bytes(unsigned info) {
    const int nch = info & 1023; return (cls_off<OUT>(nch) + cls_bytes<OUT>(nch) + (int)((info >> 10) & 511) * 4 + (WIRE_ALIGN - 1)) & ~(WIRE_ALIGN - 1); }
__device__ __forceinline__ unsigned pk_min_u16(unsigned x, unsigned y) { unsigned r; asm("v_pk_min_u16 %0, %1, %2" : "=v"(r) : "v"(x), "s"(y)); return r; }
__device__ __forceinline__ unsigned pk_rsub(unsigned s, unsigned y) { unsigned r; asm("v_pk_sub_u16 %0, %1, %2" : "=v"(r) : "s"(s), "v"(y)); return r; }
__device__ __forceinline__ unsigned pk_half(unsigned x) { unsigned r; asm("v_pk_lshrrev_b16 %0, 1, %1 op_sel_hi:[0,1]" : "=v"(r) : "v"(x)); return r; }
__device__ __forceinline__ unsigned sum_u16(unsigned x, unsigned acc) { unsigned r; asm("v_sad_u16 %0, %1, 0, %2" : "=v"(r) : "v"(x), "v"(acc)); return r; }
template <int S> __device__ __forceinline__ unsigned lshl_add(unsigned x, unsigned y) { unsigned r; asm("v_lshl_add_u32 %0, %1, %2, %3" : "=v"(r) : "v"(x), "n"(S), "v"(y)); return r; }
template <int S> __device__ __forceinline__ unsigned lshl_or(unsigned x, unsigned y) { unsigned r; asm("v_lshl_or_b32 %0, %1, %2, %3" : "=v"(r) : "v"(x), "n"(S), "v"(y)); return r; }
__device__ __forceinline__ unsigned perm(unsigned hi, unsigned lo, unsigned sel) { return __builtin_amdgcn_perm(hi, lo, sel); }
__device__ __forceinline__ void wait_lds() { asm volatile("s_waitcnt lgkmcnt(0)" ::: "memory"); }
__device__ __forceinline__ void fence() { asm volatile("" ::: "memory"); }
template <int C, int RM> __device__ __forceinline__ unsigned dpp0(unsigned x) { return (unsigned)__builtin_amdgcn_update_dpp(0, (int)x, C, RM, 0xf, false); }
__device__ __forceinline__ unsigned scan_add(unsigned x) {
    x += dpp0<0x111, 0xf>(x); x += dpp0<0x112, 0xf>(x); x += dpp0<0x114, 0xf>(x); x += dpp0<0x118, 0xf>(x); x += dpp0<0x142, 0xa>(x); x += dpp0<0x143, 0xc>(x); return x; }
__device__ __forceinline__ unsigned scan_max(unsigned x) {
    x = max(x, dpp0<0x111, 0xf>(x)); x = max(x, dpp0<0x112, 0xf>(x)); x = max(x, dpp0<0x114, 0xf>(x)); x = max(x, dpp0<0x118, 0xf>(x)); x = max(x, dpp0<0x142, 0xa>(x)); x = max(x, dpp0<0x143, 0xc>(x)); return x; }
__device__ __forceinline__ unsigned last_lane(unsigned x) { return (unsigned)__builtin_amdgcn_readlane((int)x, 63); }
template <class T> __device__ __forceinline__ LAS T* ldsp(unsigned a) { return (LAS T*)(size_t)a; }

template <bool OUT, class GET, class MID>
__device__ __forceinline__ unsigned encode_values(GET get, MID mid, int nch, unsigned T, const u32x4& mk, unsigned tail, char* dst, int lane) {
    const unsigned T2 = T * 0x10001u;
    u32x4 R0[8], R1[8];
    if (!OUT) {
#pragma unroll
        for (int it = 0; it < 8; ++it) {
            u32x4 x, y, lo; get(it, x, y);
#pragma unroll
            for (int q = 0; q < 4; ++q) {
                const unsigned w0 = q < 2 ? x[2 * q] : y[2 * q - 4], w1 = q < 2 ? x[2 * q + 1] : y[2 * q - 3];
                R0[it][q] = pk_rsub(T2, __builtin_amdgcn_alignbit(w0, w0, 31)); R1[it][q] = pk_rsub(T2, __builtin_amdgcn_alignbit(w1, w1, 31));
                lo[q] = perm(R1[it][q], R0[it][q], 0x06040200u);
            }
            st_wt(dst + it * 1024, lo);
        }
    }
#pragma unroll
    for (int i = 0; i < 4; ++i) if (i < 3 || lane < 32) *ldsp<u32x4>(tail + i * 1024 + lane * 16) = (u32x4){0u, 0u, 0u, 0u};
    fence();
    u32x4 Lb[8], Ln[8]; unsigned tot = 0, direct = 0;
#pragma unroll
    for (int it = 0; it < 8; ++it) if (!OUT || it * 64 < nch) {
        u32x4 x, y; if (OUT) get(it, x, y);
        if (it == EARLY_AT) mid();
        unsigned h = 0, h0 = 0;
#pragma unroll
        for (int q = 0; q < 4; ++q) {
            const unsigned w0 = q < 2 ? x[2 * q] : y[2 * q - 4], w1 = q < 2 ? x[2 * q + 1] : y[2 * q - 3];
            const unsigned r0 = OUT ? pk_rsub(T2, w0 << 1) : R0[it][q], r1 = OUT ? pk_rsub(T2, w1 << 1) : R1[it][q];
            const unsigned ec = perm(pk_min_u16(r1, 0x07ff07ffu), pk_min_u16(r0, 0x07ff07ffu), 0x07050301u);
            Ln[it][q] = perm(r1, r0, 0x07050301u) + perm(0x01000000u, 0x00010207u, ec) - 0x03030303u;
            tot = __builtin_amdgcn_sad_u8(Ln[it][q], 0u, tot);
            if (OUT) { const unsigned b0 = perm(0u, 0x01000100u, ec); Lb[it][q] = perm(r1, r0, 0x06040200u) | b0; h0 |= b0 << q; h |= perm(0u, 0x01010000u, ec) << q; }
            else h |= perm(0u, 0x03020100u, ec) << (2 * q);
        }
        if (OUT) { direct += __builtin_popcount(h | h0); if (it * 64 + lane < nch) *ldsp<u16>(tail + (it * 64 + lane) * 2) = (u16)(h | (h >> 12)); }
        else { direct += __builtin_popcount((h | (h >> 1)) & 0x55555555u); *ldsp<unsigned>(tail + (it * 64 + lane) * 4) = h; }
    }
    const unsigned inc = scan_add(tot), nw = (last_lane(inc) + 33u) >> 5;
    const unsigned nesc = (OUT ? (unsigned)((nch + 63) >> 6) * 1024u : 8192u) - last_lane(scan_add(direct));
    if (nw > (unsigned)NW_CAP || nesc > (unsigned)ESC_CAP) return INFO_RAW;
    if (OUT) st_wt(dst, mk);
    unsigned pos = (tail + cls_bytes<OUT>(nch)) * 8u + 1u + inc - tot;
#pragma unroll
    for (int it = 0; it < 8; ++it) if (!OUT || it * 64 < nch) {
        if (OUT && it * 64 + lane < nch) st_wt(dst + 1024 + it * 1024, Lb[it]);
#pragma unroll
        for (int e = 0; e < 16; ++e) {
            pos += (Ln[it][e >> 2] >> (8 * (e & 3))) & 0xffu;
            asm volatile("ds_or_b32 %0, %1" :: "v"((pos >> 3) & ~3u), "v"(0x80000000u >> (pos & 31u)) : "memory");
        }
    }
    const unsigned info = (T >> 4) << 19 | nw << 10 | (OUT ? (unsigned)nch : 0u);
    const int rest = wire_bytes<OUT>(info) - cls_off<OUT>(nch);
    wait_lds();
#pragma unroll
    for (int i = 0; i < 4; ++i) if (i * 1024 < rest) { if (i * 1024 + lane * 16 < rest) st_wt(dst + cls_off<OUT>(nch) + i * 1024, *ldsp<u32x4>(tail + i * 1024 + lane * 16)); }
    return info;
}
__device__ __forceinline__ unsigned top_of(unsigned sum2, unsigned n) {
    return ((unsigned)__builtin_amdgcn_readfirstlane((int)((float)sum2 * __builtin_amdgcn_rcpf((float)n))) + 2u * (276u + 128u) + 8u) & 0xfff0u; }

template <class MID> __device__ __forceinline__ unsigned encode_in(const u32x4 (&v)[16], unsigned tail, char* dst, int lane, MID mid) {
    unsigned s = 0;
#pragma unroll
    for (int i = 0; i < 16; ++i) s = sum_u16(__builtin_amdgcn_alignbit(v[i][i & 3], v[i][i & 3], 31), s);
    return encode_values<false>([&](int it, u32x4& x, u32x4& y) { x = v[2 * it]; y = v[2 * it + 1]; }, mid, 512, top_of(last_lane(scan_add(s)), 2048u), v[0], tail, dst, lane);
}

__device__ __forceinline__ unsigned encode_out(const u32x4 (&W)[16], unsigned cw, unsigned tail, char* dst, int lane) {
    u32x4 mk; unsigned s = 0;
#pragma unroll
    for (int j = 0; j < 4; ++j) {
        unsigned t[2] = {0u, 0u};
#pragma unroll
        for (int d = 15; d >= 0; --d) { const unsigned w = W[j * 4 + (d >> 2)][d & 3]; t[d >> 3] = lshl_or<2>(t[d >> 3], pk_min_u16(w, 0x00010001u)); if (j == 0) s = sum_u16(w, s); }
        mk[j] = perm(t[1], t[0], 0x05040100u) | (perm(t[1], t[0], 0x07060302u) << 1);
    }
    unsigned end[8], run = cw;
    unsigned base[16];
#pragma unroll
    for (int k = 0; k < 8; ++k) {
        const unsigned m = mk[k >> 1] >> (16 * (k & 1));
        end[k] = scan_add(((unsigned)__builtin_popcount(m & 0xffu) | (unsigned)__builtin_popcount(m & 0xff00u) << 16) * 2u);
    }
#pragma unroll
    for (int k = 0; k < 8; ++k) { const unsigned t = last_lane(end[k]); base[2 * k] = run; run += t & 0xffffu; base[2 * k + 1] = run; run += t >> 16; }
    const unsigned n = (run - cw) >> 1;
    const int nch = (int)(n + 15u) >> 4;
    const unsigned T = top_of(4u * last_lane(scan_add(s)), max(base[4] - cw, 1u));
    {
        const unsigned pad = (((T - 0x200u) >> 1) & 0x7fffu) * 0x10001u, at = cw + (min(n, 8191u) >> 10) * 2048u + lane * 16;
        *ldsp<u32x4>(at) = (u32x4){pad, pad, pad, pad}; *ldsp<u32x4>(at + 1024) = (u32x4){pad, pad, pad, pad};
    }
    fence();
#pragma unroll
    for (int i = 0; i < 16; ++i) {
        unsigned a = base[i] + ((i & 1) ? end[i >> 1] >> 16 : end[i >> 1] & 0xffffu);
#pragma unroll
        for (int e = 7; e >= 0; --e) {
            const unsigned nz = (unsigned)__builtin_amdgcn_sbfe((int)mk[i >> 2], (i & 3) * 8 + e, 1);
            a = lshl_add<1>(nz, a);
            if (e & 1) asm volatile("ds_write_b16_d16_hi %0, %1" :: "v"(a & nz), "v"(W[i][e >> 1]) : "memory");
            else asm volatile("ds_write_b16 %0, %1" :: "v"(a & nz), "v"(W[i][e >> 1]) : "memory");
        }
    }
    wait_lds();
    return encode_values<true>([&](int it, u32x4& x, u32x4& y) { x = *ldsp<u32x4>(cw + (it * 64 + lane) * 32); y = *ldsp<u32x4>(cw + (it * 64 + lane) * 32 + 16); },
                               []() {}, nch, T, mk, tail, dst, lane);
}

template <int O> __device__ __forceinline__ void unary_step(unsigned& w, unsigned p, unsigned carry) {
    unsigned z; asm("v_ffbh_u32 %0, %1" : "=v"(z) : "v"(w));
    w <<= z & 31u; z += carry;
    asm volatile("ds_write_b8 %0, %1 offset:%2" :: "v"(min(p, w)), "v"(z + perm(0x03030303u, 0xfd040404u, min(z, 7u))), "n"(O));
    w <<= 1;
}
template <bool OUT, class MID, class PUT>
__device__ __forceinline__ void decode_values(MID mid, PUT put, unsigned info, unsigned cw, unsigned esc, int lane DBG_ARGS) {
    const int nch = OUT ? (int)(info & 1023u) : 512;
    const unsigned T2 = ((info >> 19) & 0xfffu) * 0x100010u + (OUT ? 0x10001u : 0u);
    const bool five = !OUT || nch > 256, eight = !OUT || nch > 320;
    u32x4 L[8]; unsigned h[8];
#pragma unroll
    for (int it = 0; it < 8; ++it) if (it < 4 || (it < 5 ? five : eight)) {
        if (!OUT) L[it] = *ldsp<u32x4>(cw + (it * 64 + lane) * 16);
        h[it] = OUT ? *ldsp<u16>(cw + cls_off<OUT>(nch) + (it * 64 + lane) * 2) : *ldsp<unsigned>(cw + cls_off<OUT>(nch) + (it * 64 + lane) * 4);
    }
    {
        const unsigned str = cw + cls_off<OUT>(nch) + cls_bytes<OUT>(nch), nw = (info >> 10) & 511u, bits = nw * 32u - 2u;
        const unsigned lg = nw > 248u ? 9u : nw > 124u ? 8u : 7u, S = (bits + (1u << lg) - 1u) >> lg, msk = ((1u << S) - 1u) << (31u - S);
        unsigned at = esc + 4u, before = 1u;
        for (unsigned s0 = 0; s0 < (1u << lg); s0 += 128u) {
            unsigned x[2], a[2], l[2];
#pragma unroll
            for (int k = 0; k < 2; ++k) {
                const unsigned o = (s0 + lane * 2u + k) * S, i = min(o >> 5, nw - 1u);
                const unsigned d0 = *ldsp<unsigned>(str + i * 4u), d1 = *ldsp<unsigned>(str + i * 4u + 4u);
                a[k] = o + 1u;
                x[k] = __builtin_amdgcn_alignbit(d0, i + 1u < nw ? d1 : 0u, ~o) & msk;
                if (o >= bits) x[k] = 0u;
                l[k] = x[k] ? a[k] + 31u - (unsigned)__builtin_ctz(x[k]) : 0u;
            }
            const unsigned c0 = __builtin_popcount(x[0]), c = c0 + __builtin_popcount(x[1]), ci = scan_add(c), li = scan_max(max(l[0], l[1]));
            const unsigned prev = max(before, dpp0<0x138, 0xf>(li));
            unsigned p0 = at + ci - c, p1 = p0 + c0, w0 = x[0] << 1, w1 = x[1] << 1;
            unary_step<0>(w0, p0, a[0] - prev); unary_step<0>(w1, p1, a[1] - max(prev, l[0]));
            do {
                unary_step<1>(w0, p0, 0u); unary_step<1>(w1, p1, 0u); unary_step<2>(w0, p0, 0u); unary_step<2>(w1, p1, 0u); unary_step<3>(w0, p0, 0u); unary_step<3>(w1, p1, 0u);
                p0 += 3u; p1 += 3u;
            } while (__builtin_amdgcn_ballot_w64((w0 | w1) != 0u));
            at = min(at + last_lane(ci), esc + 4u + ESC_CAP); before = max(before, last_lane(li));
        }
    }
    mid();
    SSTAMP(21);
#pragma unroll
    for (int it = 0; it < 8; ++it) if (OUT && (it < 4 || (it < 5 ? five : eight))) L[it] = *ldsp<u32x4>(cw + 1024 + (it * 64 + lane) * 16);
    u32x4 C[8], S[8], P[8], D0[8], D1[8]; unsigned cnt = 0;
#pragma unroll
    for (int it = 0; it < 8; ++it) if (it < 4 || (it < 5 ? five : eight)) {
        if (OUT) { h[it] = (h[it] | (h[it] << 12)) << 1; if (it * 64 + lane >= nch) h[it] = ~0u; }
#pragma unroll
        for (int q = 0; q < 4; ++q) {
            C[it][q] = OUT ? (L[it][q] & 0x01010101u) | ((h[it] >> q) & 0x02020202u) : (h[it] >> (2 * q)) & 0x03030303u;
            const unsigned m = perm(0u, 0x000000ffu, C[it][q]), f = m & 0x01010101u, t = lshl_add<8>(f, f), u = lshl_add<16>(t, t);
            S[it][q] = ((u - f) & m) | (0x07060504u & ~m);
            P[it][q] = cnt; cnt += u >> 24;
        }
    }
    const unsigned p = esc + 4u + scan_add(cnt) - cnt;
    fence();
    auto fetch = [&](int it) {
#pragma unroll
        for (int q = 0; q < 4; ++q) { P[it][q] += p; const unsigned ad = P[it][q] & ~3u; D0[it][q] = *ldsp<unsigned>(ad); D1[it][q] = *ldsp<unsigned>(ad + 4u); }
    };
    auto finish = [&](int it) {
        u32x4 x, y;
#pragma unroll
        for (int q = 0; q < 4; ++q) {
            const unsigned E = perm(C[it][q], __builtin_amdgcn_alignbyte(D1[it][q], D0[it][q], P[it][q]), S[it][q]);
            const unsigned q0 = pk_rsub(T2, perm(E, L[it][q], 0x05010400u)), q1 = pk_rsub(T2, perm(E, L[it][q], 0x07030602u));
            const unsigned w0 = OUT ? pk_half(q0) : __builtin_amdgcn_alignbit(q0, q0, 1), w1 = OUT ? pk_half(q1) : __builtin_amdgcn_alignbit(q1, q1, 1);
            if (q < 2) { x[2 * q] = w0; x[2 * q + 1] = w1; } else { y[2 * q - 4] = w0; y[2 * q - 3] = w1; }
        }
        put(it, x, y);
    };
    fetch(0); fetch(1); fetch(2); fetch(3);
    if (OUT) {
        if (five) fetch(4);
        __builtin_amdgcn_sched_barrier(0);
        finish(0); finish(1); finish(2); finish(3);
        if (five) finish(4);
        if (eight) { fetch(5); fetch(6); fetch(7); __builtin_amdgcn_sched_barrier(0); finish(5); finish(6); finish(7); }
    } else {
#pragma unroll
        for (int it = 0; it < 8; ++it) { __builtin_amdgcn_sched_barrier(0); finish(it); if (it < 4) fetch(it + 4); }
    }
}

__device__ __forceinline__ void decode_in(unsigned info, unsigned cw, unsigned esc, char* dst, int lane DBG_ARGS) {
    decode_values<false>([]() {}, [&](int it, const u32x4& x, const u32x4& y) { st_wt(dst + it * 2048, x); st_wt(dst + it * 2048 + 1024, y); }, info, cw, esc, lane DBG_PASS);
}
__device__ __forceinline__ void make_table(int n) {
    unsigned s[4], c = 0;
#pragma unroll
    for (int e = 0; e < 4; ++e) { const bool nz = (n >> e) & 1; s[e] = nz ? 0x0100u + c * 0x0202u : 0x0c0cu; c += nz; }
    *ldsp<u32x2>(LDS_TAB + n * 8) = (u32x2){s[0] | s[1] << 16, s[2] | s[3] << 16};
}
template <class REST> __device__ __forceinline__ void decode_out(REST rest, unsigned info, unsigned cw, unsigned esc, char* outp, int lane DBG_ARGS) {
    const u32x4 mk = *ldsp<u32x4>(cw + lane * 16);
    unsigned pk[8];
    decode_values<true>([&]() {
        unsigned run = 0;
#pragma unroll
        for (int k = 0; k < 8; ++k) {
            const unsigned m = mk[k >> 1] >> (16 * (k & 1));
            const unsigned own = (__builtin_popcount(m & 0xffu) | __builtin_popcount(m & 0xff00u) << 16) * 2u, inc = scan_add(own), t = last_lane(inc);
            pk[k] = inc - own + (run | (run + (t & 0xffffu)) << 16);
            run += (t & 0xffffu) + (t >> 16);
        }
#pragma unroll
        for (int k = 0; k < 8; ++k) asm volatile("" : "+v"(pk[k]));
        rest();
    }, [&](int it, const u32x4& x, const u32x4& y) { *ldsp<u32x4>(cw + (it * 64 + lane) * 32) = x; *ldsp<u32x4>(cw + (it * 64 + lane) * 32 + 16) = y; }, info, cw, esc, lane DBG_PASS);
    SSTAMP(22);
    fence();
    SSTAMP(23);
    const u32x4 ml = (mk & 0x0f0f0f0fu) << 3, mh = (mk & 0xf0f0f0f0u) >> 1;
    unsigned at[32], d0[32], d1[32], d2[32]; u32x2 sel[32];
    auto fetch = [&](int i) {
        at[2 * i] = cw + ((i & 1) ? pk[i >> 1] >> 16 : pk[i >> 1] & 0xffffu); at[2 * i + 1] = at[2 * i] + __builtin_popcount(ml[i >> 2] & (0xffu << (8 * (i & 3)))) * 2u;
#pragma unroll
        for (int k = 2 * i; k < 2 * i + 2; ++k) {
            const unsigned ad = at[k] & ~3u;
            d0[k] = *ldsp<unsigned>(ad); d1[k] = *ldsp<unsigned>(ad + 4u); d2[k] = *ldsp<unsigned>(ad + 8u);
            sel[k] = *ldsp<u32x2>(LDS_TAB + ((((k & 1) ? mh : ml)[i >> 2] >> (8 * (i & 3))) & 0xffu));
        }
    };
#pragma unroll
    for (int i = 0; i < AHEAD; ++i) fetch(i);
#pragma unroll
    for (int i = 0; i < 16; ++i) {
        if (i + AHEAD < 16) fetch(i + AHEAD);
        __builtin_amdgcn_sched_barrier(0);
        u32x4 o;
#pragma unroll
        for (int j = 0; j < 2; ++j) {
            const int k = 2 * i + j;
            const unsigned t0 = __builtin_amdgcn_alignbyte(d1[k], d0[k], at[k]), t1 = __builtin_amdgcn_alignbyte(d2[k], d1[k], at[k]);
            o[2 * j] = perm(t1, t0, sel[k].x); o[2 * j + 1] = perm(t1, t0, sel[k].y);
        }
        st_wt(outp + OUT_OFF(i), o);
    }
}

__device__ __forceinline__ void in_job(const Args& a, char* myws, int dest, int g, int kt, int j, int lane, unsigned seen, unsigned tail DBG_ARGS) {
    const int panel = g * 8 + ((j & 15) >> 1), half = j & 1, rank = a.rank;
    char* pq = a.ws[dest == rank ? (rank + 1 + lane) & 7 : dest];
    u32x4 v[16];
    if (j < 16) {
        const int sb = lane * 16, swz = sb ^ (((sb >> 9) & 1) << 5);
        const char* src = a.x + (size_t)(panel * 256 + half * 128 + (swz >> 6)) * (KS * 2) + kt * 128 + (swz & 63);
#pragma unroll
        for (int i = 0; i < 16; ++i) v[i] = *(const u32x4*)(src + (i >> 1) * 16 * (KS * 2) + (i & 1) * 64);
        __builtin_amdgcn_sched_barrier(0); wait_vm0();
    } else {
        const char* src = a.w + (size_t)(kt * 64 + (lane & 3) * 8) * (GN * 2) + (size_t)(dest * NB + panel * 256 + half * 128 + (lane >> 2) * 8) * 2;
        u32x4 r[16];
#pragma unroll
        for (int i = 0; i < 16; ++i) r[i] = *(const u32x4*)(src + (size_t)((i >> 3) * 32 + (i & 7)) * (GN * 2));
        __builtin_amdgcn_sched_barrier(0); wait_vm0();
#pragma unroll
        for (int kh = 0; kh < 2; ++kh)
#pragma unroll
            for (int i = 0; i < 8; ++i)
#pragma unroll
                for (int d = 0; d < 4; ++d) v[kh * 8 + i][d] = __builtin_amdgcn_perm(r[kh * 8 + 2 * d + 1][i >> 1], r[kh * 8 + 2 * d][i >> 1], (i & 1) ? 0x07060302u : 0x05040100u);
    }
    asm volatile("" : "+v"(pq));
    asm volatile("" : "+v"(seen));
    STAMPD(29);
    const unsigned call = seen + 1u;
    const int pos = g * 256 + kt * 32 + j;
    if (j < 16 || dest != rank) {
        char* dst = myws + (j < 16 ? OFF_XW : OFF_WW + (size_t)dest * (256 * 16384)) + (size_t)(g * 128 + kt * 16 + (j & 15)) * 16384 + lane * 16;
        GAS u64* const note = (GAS u64*)(pq + OFF_Q) + rank * NQ + pos;
        const unsigned info = encode_in(v, tail, dst, lane, [&]() {
            if (EARLY(pos)) { wait_vm0(); if (lane < (j < 16 ? 7 : 1)) __hip_atomic_store(note + NPOS, (u64)call << 32, __ATOMIC_RELAXED, __HIP_MEMORY_SCOPE_SYSTEM); STAMPD(30); } });
        if (info & INFO_RAW) {
            wait_vm0();
#pragma unroll
            for (int i = 0; i < 16; ++i) st_wt(dst + i * 1024, v[i]);
        }
        wait_vm0();
        if (lane < (j < 16 ? 7 : 1)) __hip_atomic_store(note, ((u64)call << 32) | info, __ATOMIC_RELAXED, __HIP_MEMORY_SCOPE_SYSTEM);
        STAMPD(31);
    }
    if (dest == rank) {
        char* dst = myws + img_off(g, j, kt) + lane * 16;
#pragma unroll
        for (int i = 0; i < 16; ++i) st_wt(dst + i * 1024, v[i]);
        wait_vm0();
        if (lane == 0) __hip_atomic_store((GAS unsigned*)(myws + flag_off(g, j, kt)), call, __ATOMIC_RELAXED, __HIP_MEMORY_SCOPE_SYSTEM);
    }
}

__device__ __forceinline__ bool expired(unsigned* tmo, unsigned call, u64 t0) {
    if (__builtin_amdgcn_readfirstlane(__hip_atomic_load(tmo, __ATOMIC_RELAXED, __HIP_MEMORY_SCOPE_AGENT)) == call || now() - t0 > DEADLINE) {
        __hip_atomic_store(tmo, call, __ATOMIC_RELAXED, __HIP_MEMORY_SCOPE_AGENT);
        return true;
    }
    return false;
}

__device__ __forceinline__ void tile_of(int ti, int xcd, int& pm, int& pn) { pm = (ti >> 4) * 8 + (xcd & 1) * 4 + (ti & 3); pn = ((ti >> 3) & 1) * 8 + (xcd >> 1) * 2 + ((ti >> 2) & 1); }
__device__ __forceinline__ size_t out_off(int src, int pm, int pn, int w, int lane) { return (size_t)(pm * 256 + (w >> 2) * 64 + (lane >> 3)) * (GN * 2) + (size_t)(src * NB + pn * 256 + (w & 3) * 64) * 2 + (lane & 7) * 16; }

__global__ __launch_bounds__(512, 2) void gemm_ar_kernel(const Args a) {
    extern __shared__ __attribute__((aligned(16))) unsigned char shm[];
    LAS unsigned char* lds = (LAS unsigned char*)shm;
    const int tid = threadIdx.x, wid = __builtin_amdgcn_readfirstlane(tid >> 6), lane = tid & 63;
    const int b = blockIdx.x, xcd = b & 7, idx = b >> 3, rank = a.rank;
    char* const myws = a.myws;
    char* const myout = a.myout;
    const u64 t0 = now();
    STAMP(0);
    unsigned* const ctr = (unsigned*)(myws + OFF_CTR) + b * 32 + wid;
    unsigned* const tmo = (unsigned*)(myws + OFF_TMO);
    const unsigned seen = __hip_atomic_load(ctr, __ATOMIC_RELAXED, __HIP_MEMORY_SCOPE_AGENT);
    const unsigned cw = LDS_CW + wid * C_WAVE, tl = LDS_TL + wid * C_TAIL;

    const int pk = (idx - NG + 7 * 8 + xcd) % 7, peer = (rank + 1 + pk) & 7, ord = xcd * 2 + (idx >= NG + 7);
    if (idx >= NG) { if (tid < 2) *ldsp<unsigned>(LDS_CTL + tid * 4) = tid ? 0u : 8u; if (tid < 16) make_table(tid); wait_lds(); bar(); }
    {
        int q, qe, delay;
        if (idx < 8) { q = wid; qe = 4; delay = (wid || !(idx & 2) ? DLY0 : 0) + wid * DLY1; }
        else if (idx < 10 || idx >= NG) { q = 0; qe = 0; delay = 0; }
        else { q = wid * 64 + (idx - 10) * 8 + xcd; qe = 2048; delay = DLY0 + (ORD ? (q < 224 ? wid * DLY3 : (wid - 2) * DLY2) : wid < 2 ? 0 : (wid - 1) * DLY2); }
        for (int i = 0; i < delay; ++i) __builtin_amdgcn_s_sleep(8);
        for (; q < qe; q += 512) {
            int r;
            if (idx < 8) r = wid * 64 + (idx ^ 2) * 8 + xcd;
            else {
                const int kt = q < 896 ? q / 112 : 8 + (q - 896) / 144, rem = q < 896 ? q % 112 : (q - 896) % 144 - 32;
                r = kt * 32 + (rem < 0 ? rem + 32 : (rem % 7 + 1) * 512 + 16 + rem / 7);
            }
            in_job(a, myws, (rank + (r >> 9)) & 7, (r >> 8) & 1, (r >> 5) & 7, r & 31, lane, seen, tl DBG_PASS);
        }
    }
    const unsigned call = __builtin_amdgcn_readfirstlane(seen) + 1u;
    if (lane == 0) __hip_atomic_store(ctr, call, __ATOMIC_RELAXED, __HIP_MEMORY_SCOPE_AGENT);
    STAMP(1);

    if (idx >= NG) {
        const char* const pws = a.ws[peer];
        const u64* const queue = (const u64*)(myws + OFF_Q) + peer * NQ;
        unsigned* const tq = (unsigned*)(myws + OFF_TICK) + pk * 64;
        const unsigned tbase = (call - 1u) * NPOS - 128u;
        const unsigned ctl = LDS_CTL;
        unsigned pos = (((ORD ? 0x76543120u : 0x75634120u) >> (4 * wid)) & 7u) * 16 + ord;
        bool bad = false;
        while (pos < (unsigned)NPOS) {
            const int g = pos >> 8, kt = (pos >> 5) & 7, j = pos & 31;
            const char* const src = pws + (pos >= NIN ? OFF_OW + (size_t)(pos - NIN) * 16384
                                           : (j < 16 ? OFF_XW : OFF_WW + (size_t)rank * (256 * 16384)) + (size_t)(g * 128 + kt * 16 + (j & 15)) * 16384) + lane * 16;
            unsigned my = wid;
            auto token = [&]() {
                for (;;) {
                    unsigned rel;
                    asm volatile("ds_read_b32 %0, %1 offset:4\n\ts_waitcnt lgkmcnt(0)" : "=v"(rel) : "v"(ctl) : "memory");
                    if ((int)(__builtin_amdgcn_readfirstlane(rel) + TOK - my) > 0) break;
                    if (now() - t0 > DEADLINE) break;
                    __builtin_amdgcn_s_sleep(1);
                }
            };
            unsigned info;
            int o = 0;
            for (unsigned at = EARLY(pos) ? pos + NPOS : pos, spin = 1;; ++spin) {
                const u64 e = __hip_atomic_load(queue + at, __ATOMIC_RELAXED, __HIP_MEMORY_SCOPE_SYSTEM);
                info = __builtin_amdgcn_readfirstlane((unsigned)e);
                if (__builtin_amdgcn_readfirstlane((unsigned)(e >> 32)) >= call) {
                    if (at == pos) break;
                    STAMP1(2);
                    token();
                    for (; o < 8192; o += 1024) {
                        if (PEEK && o == 7168) { if (lane < 2) __builtin_amdgcn_global_load_lds((const unsigned*)(queue + pos) + lane, ldsp<unsigned>(tl), 4, 0, 17); wait_vm<WIN>(); }
                        else wait_vm<WIN - 1>();
                        __builtin_amdgcn_global_load_lds((const unsigned*)(src + o), ldsp<unsigned>(cw + o), 16, 0, 17);
                    }
                    if (PEEK) {
                        u32x2 pe;
                        asm volatile("s_waitcnt vmcnt(1)\n\tds_read_b64 %0, %1\n\ts_waitcnt lgkmcnt(0)" : "=v"(pe) : "v"(tl) : "memory");
                        info = __builtin_amdgcn_readfirstlane(pe.x);
                        if (__builtin_amdgcn_readfirstlane(pe.y) >= call) break;
                    }
                    at = pos;
                    continue;
                }
                if ((spin & 15u) == 0 && expired(tmo, call, t0)) { bad = true; break; }
                if (at == pos) __builtin_amdgcn_s_sleep(POLL_SLEEP); else __builtin_amdgcn_s_sleep(POLL_FAST);
            }
            if (bad) break;
            LAP(24); STAMP1(2);
            if ((info & 1023u) > 512u || ((info >> 10) & 511u) - 1u >= (unsigned)NW_CAP) info = INFO_RAW;
            if (info & INFO_RAW) o = 0;
            const int wire = (info & INFO_RAW) ? 16384 : pos >= NIN ? wire_bytes<true>(info) : wire_bytes<false>(info);
            if (lane == 0 && pos >= 128u) asm volatile("ds_add_rtn_u32 %0, %1, %2\n\ts_waitcnt lgkmcnt(0)" : "+v"(my) : "v"(ctl), "v"(1u) : "memory");
            my = __builtin_amdgcn_readfirstlane(my);
            auto pull = [&](int end) {
                if (!EARLY(pos)) token();
                LAP(25); STAMP1(7); STAMP(18);
                for (; o < end; o += 1024) {
                    if (o + lane * 16 < end) __builtin_amdgcn_global_load_lds((const unsigned*)(src + o), ldsp<unsigned>(cw + o), 16, 0, 17);
                    if (pos < (unsigned)(NPOS - WIDE)) wait_vm<WIN - 1>();
                }
                wait_vm<REL>();
                if (lane == 0) asm volatile("ds_add_u32 %0, %1 offset:4" :: "v"(ctl), "v"(1u) : "memory");
                wait_vm0();
                LAP(26); STAMP1(3); STAMP(19);
            };
            const bool pre = pos >= (unsigned)(NPOS - PREN) && !(info & INFO_RAW);
            const int cut = cls_off<true>(info & 1023u) & ~127;
            if (pre) {
                __builtin_amdgcn_global_load_lds((const unsigned*)src, ldsp<unsigned>(cw), 16, 0, 17);
                for (int r = cut; r < wire; r += 1024) if (r + lane * 16 < wire) __builtin_amdgcn_global_load_lds((const unsigned*)(src + r), ldsp<unsigned>(cw + r), 16, 0, 17);
                wait_vm0();
                o = 1024;
            } else pull(wire);
            int t = 0;
            char* dst;
            if (pos >= NIN) { int pm, pn; tile_of((pos - NIN) >> 6, ((pos - NIN) >> 3) & 7, pm, pn); dst = myout + out_off(peer, pm, pn, pos & 7, lane); }
            else { t = slot_of(peer, rank, kt); dst = myws + img_off(g, j, t) + lane * 16; }
            if (info & INFO_RAW) {
                u32x4 v[16];
#pragma unroll
                for (int i = 0; i < 16; ++i) v[i] = *ldsp<u32x4>(cw + i * 1024 + lane * 16);
#pragma unroll
                for (int i = 0; i < 16; ++i) st_wt(dst + (pos >= NIN ? OUT_OFF(i) : (size_t)i * 1024), v[i]);
            } else if (pos >= NIN) decode_out([&]() { if (pre) pull(cut); }, info, cw, tl, dst, lane DBG_PASS);
            else decode_in(info, cw, tl, dst, lane DBG_PASS);
            LAP(27); STAMP1(4); STAMP(20);
            unsigned nx = 0;
            if (lane == 0) nx = __hip_atomic_fetch_add(tq, 1u, __ATOMIC_RELAXED, __HIP_MEMORY_SCOPE_AGENT);
            wait_vm0();
            if (pos < NIN && lane == 0) __hip_atomic_store((unsigned*)(myws + flag_off(g, j, t)), call, __ATOMIC_RELAXED, __HIP_MEMORY_SCOPE_SYSTEM);
            LAP(28); STAMP(8 + (pos >> 8)); STAMP1(6);
            pos = __builtin_amdgcn_readfirstlane(nx) - tbase;
        }
        if (bad && lane == 0) *(GAS u16*)(myout + (size_t)rank * (NB * 2)) = 0x7fc0;
        STAMP(5);
        STAMP_FLUSH();
        return;
    }

    const int wr = wid >> 2, wc = wid & 3, fr = lane & 15, fq = lane >> 4;
    bool bad = false;
    const bool hm = idx < 4;
    unsigned voffA[2], voffB[2];
#pragma unroll
    for (int i = 0; i < 2; ++i) {
        int R, C; stage_rc(tid * 16 + i * 8192, R, C);
        voffA[i] = tid * 16 + (hm ? (idx & 1) * 8192 + i * 16384 : i * 8192);
        const int w_ = R >> 5, n_ = (R >> 4) & 1, q_ = (R & 15) >> 2, j_ = R & 3;
        voffB[i] = (w_ >> 1) * 16384 + (C >> 5) * 8192 + (4 * n_ + j_) * 1024 + ((w_ & 1) * 8 + q_) * 64 + ((C >> 3) & 3) * 16;
    }
    constexpr size_t kstep = 32768, hstepA = 16384, hstepB = 256;
    const unsigned ldsw = (unsigned)wid * 1024u;
    const int aoff = lds_byte(wr * 64 + fr, fq * 8), boff = lds_byte(wc * 32 + fr, fq * 8);
#define G_SA(b, h) (((b) * 2 + (h)) * HTB)
#define G_SB(b, h) ((4 + (b) * 2 + (h)) * HTB)
#define G_STAGE(bufoff, gbase, voff) do { _Pragma("unroll") for (int _i = 0; _i < 2; ++_i) \
        __builtin_amdgcn_global_load_lds((const unsigned*)((const char*)(gbase) + (voff)[_i]), (LAS unsigned*)(lds + (bufoff) + ldsw + _i * 8192), 16, 0, 17); } while (0)
#define G_LDA(dst, b, h) do { _Pragma("unroll") for (int m = 0; m < 4; ++m) _Pragma("unroll") for (int k = 0; k < 2; ++k) dst[m][k] = *(const LAS bf16x8*)(lds + G_SA(b, h) + aoff + m * 2048 + k * 1024); } while (0)
#define G_LDB(dst, b, h) do { _Pragma("unroll") for (int n = 0; n < 2; ++n) _Pragma("unroll") for (int k = 0; k < 2; ++k) dst[n][k] = *(const LAS bf16x8*)(lds + G_SB(b, h) + boff + n * 2048 + k * 1024); } while (0)
#define G_MMA(ai, bj, At, Bt) do { __builtin_amdgcn_s_setprio(1); _Pragma("unroll") for (int m = 0; m < 4; ++m) _Pragma("unroll") for (int n = 0; n < 2; ++n) _Pragma("unroll") for (int k = 0; k < 2; ++k) \
        acc[ai][bj][m][n] = __builtin_amdgcn_mfma_f32_16x16x32_bf16(Bt[n][k], At[m][k], acc[ai][bj][m][n], 0, 0, 0); __builtin_amdgcn_s_setprio(0); } while (0)
#define G_WAIT_V(n) asm volatile("s_waitcnt vmcnt(" #n ")" ::: "memory")
#define G_WAIT_L(n) asm volatile("s_waitcnt lgkmcnt(" #n ")" ::: "memory")
#define G_BAR __builtin_amdgcn_s_barrier()
#define G_SCHED __builtin_amdgcn_sched_barrier(0)
    for (int round = 0; round < 2; ++round) {
    const int ti = hm ? round * 8 + (idx >> 1) : round ? idx + 14 : idx < 10 ? idx - 2 : idx;
    int pm, pn; tile_of(ti, xcd, pm, pn);
    const char* const fX = myws + OFF_INF + (size_t)pm * 512 + lane * 8;
    const char* const fW = myws + OFF_INF + (size_t)(16 + pn) * 512 + lane * 8;
    auto poll = [&](int need) -> int {
        for (;;) {
            u32x2 fa, fb;
            asm volatile("global_load_dwordx2 %0, %2, off sc0 sc1\n\tglobal_load_dwordx2 %1, %3, off sc0 sc1\n\ts_waitcnt vmcnt(0)"
                         : "=&v"(fa), "=&v"(fb) : "v"(fX), "v"(fW) : "memory");
            const u64 miss = __builtin_amdgcn_ballot_w64(fa.x < call || fa.y < call || fb.x < call || fb.y < call);
            const int upto = miss ? __builtin_ctzll(miss) : 64;
            if (upto >= need) return upto;
            if (expired(tmo, call, t0)) { bad = true; return 64; }
            __builtin_amdgcn_s_sleep(4);
        }
    };
    f32x4 acc[2][2][4][2];
#pragma unroll
    for (int x = 0; x < 2; ++x)
#pragma unroll
        for (int y = 0; y < 2; ++y)
#pragma unroll
            for (int m = 0; m < 4; ++m)
#pragma unroll
                for (int n = 0; n < 2; ++n) acc[x][y][m][n] = (f32x4){0.f, 0.f, 0.f, 0.f};
    bf16x8 At[4][2], B0[2][2], B1[2][2];
    const char* const cA = myws + OFF_XI + (size_t)pm * 64 * kstep;
    const char* const cB = myws + OFF_WI + (size_t)pn * 64 * kstep;
    int ready = poll(2);
    STAMP1(2);
    wait_vm0();
    if (hm) {
        G_STAGE(0, cA, voffA); G_STAGE(HTB, cB, voffB); G_STAGE(2 * HTB, cB + hstepB, voffB);
        G_STAGE(3 * HTB, cA + kstep, voffA); G_STAGE(4 * HTB, cB + kstep, voffB); G_STAGE(5 * HTB, cB + kstep + hstepB, voffB);
        if (wr == 1) G_BAR;
        G_WAIT_V(6); G_BAR; G_BAR;
        unsigned cur = 0, nxt = 6 * HTB;
        for (int t = 0; t < 64; ++t) {
            if (t < 62 && ready < t + 3) ready = poll(t + 3);
            const size_t k2 = t < 62 ? (size_t)(t + 2) * kstep : 0;
            LAS unsigned char* const la = lds + cur + aoff; LAS unsigned char* const lb = lds + cur + boff;
#pragma unroll
            for (int n = 0; n < 2; ++n)
#pragma unroll
                for (int k = 0; k < 2; ++k) B0[n][k] = *(const LAS bf16x8*)(lb + HTB + n * 2048 + k * 1024);
#pragma unroll
            for (int m = 0; m < 4; ++m)
#pragma unroll
                for (int k = 0; k < 2; ++k) At[m][k] = *(const LAS bf16x8*)(la + m * 2048 + k * 1024);
            G_STAGE(nxt, cA + k2, voffA); G_STAGE(nxt + HTB, cB + k2, voffB);
            G_WAIT_V(10); G_BAR; G_WAIT_L(0); G_MMA(0, 0, At, B0); G_BAR; G_SCHED;
#pragma unroll
            for (int n = 0; n < 2; ++n)
#pragma unroll
                for (int k = 0; k < 2; ++k) B1[n][k] = *(const LAS bf16x8*)(lb + 2 * HTB + n * 2048 + k * 1024);
            G_STAGE(nxt + 2 * HTB, cB + k2 + hstepB, voffB);
            G_WAIT_V(8); G_BAR; G_WAIT_L(0); G_MMA(0, 1, At, B1); G_BAR; G_SCHED;
            nxt = cur; cur = cur == 6 * HTB ? 0 : cur + 3 * HTB;
        }
    } else {
    G_STAGE(G_SB(0, 0), cB, voffB); G_STAGE(G_SA(0, 0), cA, voffA); G_STAGE(G_SB(0, 1), cB + hstepB, voffB); G_STAGE(G_SA(0, 1), cA + hstepA, voffA);
    if (wr == 1) G_BAR;
    G_WAIT_V(4); G_BAR;
    G_STAGE(G_SB(1, 0), cB + kstep, voffB); G_STAGE(G_SA(1, 0), cA + kstep, voffA); G_STAGE(G_SB(1, 1), cB + hstepB + kstep, voffB);
    G_WAIT_V(6); G_BAR;
    for (int t = 0; t < 64; t += 2) {
        const bool last = (t == 62);
        if (!last && ready < t + 4) ready = poll(t + 4);
        const char* a1 = cA + (size_t)(t + 1) * kstep;
        const char* a2 = last ? cA : cA + (size_t)(t + 2) * kstep; const char* b2 = last ? cB : cB + (size_t)(t + 2) * kstep;
        const char* a3 = a2 + kstep; const char* b3 = b2 + kstep;
        G_LDB(B0, 0, 0); G_SCHED; G_LDA(At, 0, 0); G_STAGE(G_SA(1, 1), a1 + hstepA, voffA);
        G_WAIT_L(8); G_BAR; G_WAIT_L(0); G_MMA(0, 0, At, B0); G_BAR; G_SCHED;
        G_LDB(B1, 0, 1); G_STAGE(G_SB(0, 0), b2, voffB);
        G_BAR; G_WAIT_L(0); G_MMA(0, 1, At, B1); G_BAR;
        G_LDA(At, 0, 1); G_STAGE(G_SA(0, 0), a2, voffA);
        G_BAR; G_WAIT_L(0); G_MMA(1, 0, At, B0); G_BAR; G_SCHED;
        G_STAGE(G_SB(0, 1), b2 + hstepB, voffB);
        G_WAIT_V(6); G_BAR; G_MMA(1, 1, At, B1); G_BAR;
        G_LDB(B0, 1, 0); G_SCHED; G_LDA(At, 1, 0); G_STAGE(G_SA(0, 1), a2 + hstepA, voffA);
        G_WAIT_L(8); G_BAR; G_WAIT_L(0); G_MMA(0, 0, At, B0); G_BAR; G_SCHED;
        G_LDB(B1, 1, 1); G_STAGE(G_SB(1, 0), b3, voffB);
        G_BAR; G_WAIT_L(0); G_MMA(0, 1, At, B1); G_BAR;
        G_LDA(At, 1, 1); G_STAGE(G_SA(1, 0), a3, voffA);
        G_BAR; G_WAIT_L(0); G_MMA(1, 0, At, B0); G_BAR; G_SCHED;
        G_STAGE(G_SB(1, 1), b3 + hstepB, voffB);
        G_WAIT_V(6); G_BAR; G_MMA(1, 1, At, B1); G_BAR;
    }
    }
    G_WAIT_V(0);
    if (wr == 0) G_BAR;
    G_BAR;
    STAMP(round ? 6 : 3);

    int ln = lane;
    asm volatile("" : "+v"(ln) :: "memory");
    char* pq = a.ws[(rank + 1 + (ln >> 3)) & 7];
    if (bad) acc[0][0][0][0][0] = __builtin_nanf("");
    const unsigned cx = hm ? LDS_CW + ((idx & 1) * 4 + wc) * C_WAVE + wr * 8192 : cw;
#pragma unroll
    for (int ai = 0; ai < 2; ++ai) if (ai == 0 || !hm)
#pragma unroll
        for (int m = 0; m < 4; ++m)
#pragma unroll
            for (int bj = 0; bj < 2; ++bj) {
                const f32x4 v0 = acc[ai][bj][m][0], v1 = acc[ai][bj][m][1];
                u32x4 w; w[0] = relu_pk(cvt_pk_bf16(v0[0], v0[1])); w[1] = relu_pk(cvt_pk_bf16(v0[2], v0[3]));
                w[2] = relu_pk(cvt_pk_bf16(v1[0], v1[1])); w[3] = relu_pk(cvt_pk_bf16(v1[2], v1[3]));
                *ldsp<u32x4>(cx + (ai * 64 + m * 16 + (ln & 15)) * 128 + (((bj * 4 + (ln >> 4)) ^ (ln & 7)) << 4)) = w;
            }
    asm volatile("" ::: "memory");
    if (hm) { wait_lds(); bar(); }
    if (!hm || wr == (idx & 1)) {
    u32x4 W[16];
#pragma unroll
    for (int i = 0; i < 16; ++i) W[i] = *ldsp<u32x4>(cw + i * 1024 + (ln >> 3) * 128 + (((ln & 7) ^ (ln >> 3)) << 4));
    wait_lds(); wait_vm0();
    asm volatile("" : "+v"(pq));
    char* const outp = myout + out_off(rank, pm, pn, wid, ln);
#pragma unroll
    for (int i = 0; i < 16; ++i) st_wt(outp + OUT_OFF(i), W[i]);
    char* const slot = myws + OFF_OW + (size_t)((ti * 8 + xcd) * 8 + wid) * 16384 + ln * 16;
    const unsigned info = encode_out(W, cw, tl, slot, ln);
    if (info & INFO_RAW) {
#pragma unroll
        for (int i = 0; i < 16; ++i) st_wt(slot + i * 1024, W[i]);
    }
    if (ln == 0) *(LAS unsigned*)(lds + LDS_CTL + wid * 4) = info;
    }
    wait_lds();
    wait_vm0();
    bar();
    if (wid == 0 && ln < 56 && (!hm || ((ln >> 2) & 1) == (idx & 1))) {
        const unsigned inf = *(const LAS unsigned*)(lds + LDS_CTL + (ln & 7) * 4);
        __hip_atomic_store((GAS u64*)(pq + OFF_Q) + rank * NQ + NIN + (ti * 8 + xcd) * 8 + (ln & 7), ((u64)call << 32) | inf, __ATOMIC_RELAXED, __HIP_MEMORY_SCOPE_SYSTEM);
    }
    STAMP(round ? 7 : 4);
    }
    STAMP(5);
    STAMP_FLUSH();
}

extern "C" void kernel_launch(int rank, int nranks, ncclComm_t comm,
                              hipStream_t stream,
                              void* const* d_in, const int* in_sizes, int n_in,
                              void* d_out, int out_size,
                              const int* in_dtypes, int out_dtype,
                              void* const* d_out_all,
                              void* const* d_ws_all, size_t ws_bytes) {
    if (nranks != NR || (unsigned)rank >= (unsigned)NR || ws_bytes < WS_NEED || n_in < 2) { fprintf(stderr, "unsupported configuration\n"); return; }
    static bool attr[NR] = {};
    if (!attr[rank]) { HIP_CHECK(hipFuncSetAttribute((const void*)gemm_ar_kernel, hipFuncAttributeMaxDynamicSharedMemorySize, LDS_BYTES)); attr[rank] = true; }
    Args a;
    a.x = (const char*)d_in[0]; a.w = (const char*)d_in[1]; a.rank = rank;
    for (int p = 0; p < NR; ++p) a.ws[p] = (char*)d_ws_all[p];
    a.myws = a.ws[rank]; a.myout = (char*)d_out;
    hipLaunchKernelGGL(gemm_ar_kernel, dim3(256), dim3(512), LDS_BYTES, stream, a);
    HIP_CHECK(hipGetLastError());
}
```
